```python
import numpy as np

kernel_name = "gpu_gemm_a2a_m4096_k4096_n4096_bf16_reluq8"


ABSMAX_RTOL = 0.02

COMPUTE_DTYPE = "bfloat16"
EPILOGUE = "relu_quant_int8"
IN_QUANT = None
OUT_QUANT = "int8"
IN_DTYPES = {"x": "bfloat16", "w_mat": "bfloat16"}
WORLD_SIZE = 8

def _to_bf16(x):
    u = np.ascontiguousarray(x, dtype=np.float32).view(np.uint32)
    return ((u + 0x7FFF + ((u >> 16) & 1)) & 0xFFFF0000).view(np.float32)

def setup_inputs():

    x = _to_bf16(np.random.randn(4096, 4096).astype(np.float32))
    w_mat = _to_bf16((np.random.randn(4096, 4096) / np.sqrt(4096)).astype(np.float32))
    return {"x": x, "w_mat": w_mat}

def _quant_dequant(y, amax):
    scale = amax / 127.0
    q = np.clip(np.round(y / scale), -127.0, 127.0)
    return (q * scale).astype(np.float32)
def _epilogue(y):
    y = np.maximum(y, 0.0)
    return _quant_dequant(y, np.abs(y).max())

def reference(x, w_mat):
    y = (x @ w_mat).astype(np.float32)
    return _epilogue(y)
```

```cpp
#include <hip/hip_runtime.h>
#include <rccl/rccl.h>
#include <cstdio>

typedef unsigned u32;
typedef unsigned long long u64;
typedef u32 u32x4 __attribute__((ext_vector_type(4)));
typedef u32 u32x2 __attribute__((ext_vector_type(2)));
typedef float f32x4 __attribute__((ext_vector_type(4)));
typedef __bf16 bf16x4 __attribute__((ext_vector_type(4)));
typedef __bf16 bf16x8 __attribute__((ext_vector_type(8)));
typedef __bf16 bf16x2 __attribute__((ext_vector_type(2)));
#define LAS __attribute__((address_space(3)))

#define CFG_BN 128
#define CFG_P16 0
#define CFG_XOR 0

constexpr int NR = 8, MS = 512, KD = 4096, ND = 4096, NCOL = 512;
constexpr int BM = 128, BN = CFG_BN, BK = 64, KS = BN / 64;
constexpr int NF = BN / 32;
constexpr int NSTEP = KD / KS / BK;
constexpr int NS = (BN == 128) ? 4 : 3;
constexpr int XS = BM * BK * 2, WS_ = BK * BN * 2;
constexpr int XRING = 0, WRING = NS * XS, SMALL = NS * (XS + WS_);
constexpr int LDS_BYTES = SMALL + 256;
constexpr int XI = XS / 4096, WI = WS_ / 4096, LI = XI + WI;
constexpr int PI = CFG_P16 ? 4 : 8;
constexpr u64 DEADLINE = 100000000ull;

constexpr size_t WS_CTR = 0;
constexpr size_t WS_TMO = 32768;
constexpr size_t WS_DONE = 36864;
constexpr size_t WS_AMAX = 65536;
constexpr size_t WS_XF = 131072;
constexpr size_t WS_TS = 262144;
constexpr size_t WS_EX = 1u << 20;
constexpr size_t WS_END = WS_EX + (size_t)256 * (KS - 1) * 4 * PI * 1024;

template <int N> __device__ __forceinline__ void wait_vm() { asm volatile("s_waitcnt vmcnt(%0)" ::"i"(N) : "memory"); }
__device__ __forceinline__ void st16_sys(void* p, u32x4 v) { asm volatile("global_store_dwordx4 %0, %1, off sc0 sc1\n\ts_nop 1" ::"v"(p), "v"(v) : "memory"); }
__device__ __forceinline__ void st8_sys(void* p, u32x2 v) { asm volatile("global_store_dwordx2 %0, %1, off sc0 sc1\n\ts_nop 1" ::"v"(p), "v"(v) : "memory"); }
__device__ __forceinline__ void st4_sys(void* p, u32 v) { asm volatile("global_store_dword %0, %1, off sc0 sc1\n\ts_nop 1" ::"v"(p), "v"(v) : "memory"); }
__device__ __forceinline__ u32 ld4_sys(const void* p) { u32 v; asm volatile("global_load_dword %0, %1, off sc0 sc1\n\ts_waitcnt vmcnt(0)" : "=&v"(v) : "v"(p) : "memory"); return v; }
__device__ __forceinline__ u64 now() { return __builtin_amdgcn_s_memrealtime(); }

#define TS(i) do {} while (0)

__global__ __launch_bounds__(512, 2) void gemm_a2a(const char* __restrict__ x, const char* __restrict__ w,
                                                   char* const* out_all, char* const* ws_all, char* ws, int rank) {
  extern __shared__ __attribute__((aligned(16))) char smem[];
  const int b = blockIdx.x, lane = threadIdx.x & 63;
  const int wave = __builtin_amdgcn_readfirstlane(threadIdx.x >> 6);
  const int xcd = b & 7, idx = b >> 3;
  const int ks = xcd % KS, mt = idx & 3, ntl = idx >> 2, nt = (xcd / KS) * 8 + ntl;
  const int kx = (mt ^ ntl) & CFG_XOR;
  TS(0);
  if (wave >= 4) {
    const int lw = wave - 4;
    auto rx = __builtin_amdgcn_make_buffer_rsrc((void*)x, 0, MS * KD * 2, 0x00020000);
    auto rw = __builtin_amdgcn_make_buffer_rsrc((void*)w, 0, KD * ND * 2, 0x00020000);
    int vx[2];
#pragma unroll
    for (int h = 0; h < 2; h++) { const int r = (lane >> 3) + 8 * h; vx[h] = (lane >> 3) * 8192 + (((lane & 7) ^ ((r >> 1) & 7)) << 4); }
    constexpr int LPR = BN / 8, RPI = 64 / LPR;
    int vw[WI];
#pragma unroll
    for (int i = 0; i < WI; i++) { const int k = (lw * WI + i) * RPI + lane / LPR; vw[i] = (lane / LPR) * 8192 + (((lane % LPR) ^ (k & 11)) << 4); }
    const int sx0 = (mt * BM + lw * XI * 8) * 8192 + ks * (KD / KS) * 2;
    const int sw0 = (ks * (KD / KS) + lw * WI * RPI) * 8192 + nt * BN * 2;
    auto issue = [&](int s, int slot) {
      const int sp = s ^ kx;
#pragma unroll
      for (int i = 0; i < XI; i++)
        __builtin_amdgcn_raw_ptr_buffer_load_lds(rx, (LAS void*)(smem + XRING + slot * XS + (lw * XI + i) * 1024), 16, vx[i & 1], sx0 + i * 8 * 8192 + sp * (BK * 2), 0, 0);
#pragma unroll
      for (int i = 0; i < WI; i++)
        __builtin_amdgcn_raw_ptr_buffer_load_lds(rw, (LAS void*)(smem + WRING + slot * WS_ + (lw * WI + i) * 1024), 16, vw[i], sw0 + (i * RPI + sp * BK) * 8192, 0, 0);
    };
#pragma unroll
    for (int s = 0; s < NS; s++) issue(s, s);
    wait_vm<(NS - 1) * LI>();
    __builtin_amdgcn_s_barrier();
    int slot = 0;
    for (int j = 1; j <= NSTEP - NS; j++) {
      wait_vm<(NS - 2) * LI>();
      __builtin_amdgcn_s_barrier();
      issue(j - 1 + NS, slot);
      slot = slot + 1 == NS ? 0 : slot + 1;
    }
    if (NS == 4) { wait_vm<2 * LI>(); __builtin_amdgcn_s_barrier(); }
    wait_vm<LI>(); __builtin_amdgcn_s_barrier();
    wait_vm<0>(); __builtin_amdgcn_s_barrier();
    TS(1);
    return;
  }
  const int wm = wave >> 1, wn = wave & 1, g = lane >> 4, li = lane & 15;
  u32* ctr = (u32*)(ws + WS_CTR + b * 128);
  u32* tmo = (u32*)(ws + WS_TMO);
  const u32 call = __builtin_amdgcn_readfirstlane(ld4_sys(ctr)) + 1;
  char* peer_ws = ws_all[lane & 7];
  f32x4 acc[4][NF];
#pragma unroll
  for (int m = 0; m < 4; m++)
#pragma unroll
    for (int n = 0; n < NF; n++) acc[m][n] = f32x4{0.f, 0.f, 0.f, 0.f};
  u32 ax[2], aw[4];
#pragma unroll
  for (int kk = 0; kk < 2; kk++) ax[kk] = (u32)(size_t)smem + XRING + (32 * wm + li) * 128 + (((4 * kk + g) ^ ((li >> 1) & 7)) << 4);
  {
    const int q = li >> 2, p = li & 3;
#pragma unroll
    for (int n = 0; n < 4; n++)
      aw[n] = (u32)(size_t)smem + WRING + (8 * g + q) * (BN * 2) + (((8 * wn + 2 * p + (n >> 1)) ^ q ^ ((g & 1) << 3)) << 4) + (((n ^ p) & 1) << 3);
  }
  bf16x8 fx[2][4], fw[2][NF];
  auto rd = [&](int buf, int slot, int kk) {
#pragma unroll
    for (int n = 0; n < NF; n++) {
      const u32 a = aw[n & 3] + slot * WS_ + kk * 32 * BN * 2 + (n >> 2) * 256;
      const bf16x4 lo = __builtin_amdgcn_ds_read_tr16_b64_v4bf16((LAS bf16x4*)a);
      const bf16x4 hi = __builtin_amdgcn_ds_read_tr16_b64_v4bf16((LAS bf16x4*)(a + 4 * BN * 2));
      fw[buf][n] = __builtin_shufflevector(lo, hi, 0, 1, 2, 3, 4, 5, 6, 7);
    }
#pragma unroll
    for (int m = 0; m < 4; m++) fx[buf][m] = *(LAS bf16x8*)(ax[kk] + slot * XS + ((m & 1) * 16 + (m >> 1) * 64) * 128);
  };
  auto mm = [&](int buf) {
#pragma unroll
    for (int n = 0; n < NF; n++)
#pragma unroll
      for (int m = 0; m < 4; m++) acc[m][n] = __builtin_amdgcn_mfma_f32_16x16x32_bf16(fw[buf][n], fx[buf][m], acc[m][n], 0, 0, 0);
  };
  auto mix = [&]() {
#pragma unroll
    for (int i = 0; i < 2 * NF + 4; i++) { __builtin_amdgcn_sched_group_barrier(0x008, 1, 0); __builtin_amdgcn_sched_group_barrier(0x100, 1, 0); }
    __builtin_amdgcn_sched_group_barrier(0x008, 4 * NF - (2 * NF + 4), 0);
  };
  __builtin_amdgcn_s_barrier();
  __builtin_amdgcn_sched_barrier(0);
  rd(0, 0, 0);
  for (int j0 = 0; j0 < NSTEP; j0 += NS) {
#pragma unroll
    for (int u = 0; u < NS; u++) {
      if (NSTEP % NS != 0 && j0 + u >= NSTEP) break;
      rd(1, u, 1); mm(0); mix();
      __builtin_amdgcn_sched_barrier(0);
      if (j0 + u + 1 < NSTEP) {
        asm volatile("s_waitcnt lgkmcnt(0)" ::: "memory");
        __builtin_amdgcn_s_barrier();
        __builtin_amdgcn_sched_barrier(0);
        rd(0, (u + 1) % NS, 0); mm(1); mix();
        __builtin_amdgcn_sched_barrier(0);
      } else {
        mm(1);
      }
    }
  }
  TS(1);
  bool bad = false;
  const u64 t0 = now();
  f32x4 y[2][4];
  {
    char* ex = ws + WS_EX;
#pragma unroll
    for (int o = 0; o < KS; o++) {
      if (o == ks) {
#pragma unroll
        for (int m = 0; m < 2; m++)
#pragma unroll
          for (int n = 0; n < 4; n++) y[m][n] = acc[2 * (o & 1) + m][4 * (o >> 1) + n];
      } else {
        const int d = b - ks + o;
        char* dst = ex + ((size_t)(d * (KS - 1) + ((o ^ ks) - 1)) * PI * 4 + wave) * 1024 + lane * 16;
#pragma unroll
        for (int i = 0; i < PI; i++) {
          u32x4 v;
          if (CFG_P16) {
            const f32x4 a = acc[2 * (o & 1) + (i >> 1)][4 * (o >> 1) + 2 * (i & 1)], c = acc[2 * (o & 1) + (i >> 1)][4 * (o >> 1) + 2 * (i & 1) + 1];
            v[0] = __builtin_bit_cast(u32, bf16x2{(__bf16)a[0], (__bf16)a[1]}); v[1] = __builtin_bit_cast(u32, bf16x2{(__bf16)a[2], (__bf16)a[3]});
            v[2] = __builtin_bit_cast(u32, bf16x2{(__bf16)c[0], (__bf16)c[1]}); v[3] = __builtin_bit_cast(u32, bf16x2{(__bf16)c[2], (__bf16)c[3]});
          } else {
            v = __builtin_bit_cast(u32x4, acc[2 * (o & 1) + (i >> 2)][4 * (o >> 1) + (i & 3)]);
          }
          st16_sys(dst + i * 4096, v);
        }
      }
    }
    wait_vm<0>();
    TS(2);
    if (lane < KS && lane != ks) st4_sys(ws + WS_XF + ((b - ks + lane) * 4 + wave) * 128 + ks * 4, call);
    if (wave == 0 && lane == 63) st4_sys(ctr, call);
    const char* pf = lane < KS ? ws + WS_XF + (b * 4 + wave) * 128 + lane * 4 : (const char*)tmo;
    while (true) {
      const u32 f = ld4_sys(pf);
      if (__builtin_amdgcn_readlane(f, 63) == call || now() - t0 > DEADLINE) { bad = true; break; }
      if (__all(lane >= KS || lane == ks || f >= call)) break;
      __builtin_amdgcn_s_sleep(1);
    }
    TS(3);
#pragma unroll
    for (int j = 0; j < KS - 1; j++) {
      const char* src = ex + ((size_t)(b * (KS - 1) + j) * PI * 4 + wave) * 1024 + lane * 16;
      u32x4 r[PI];
      if (CFG_P16) {
        asm volatile("global_load_dwordx4 %0, %4, off sc0 sc1\n\tglobal_load_dwordx4 %1, %5, off sc0 sc1\n\t"
                     "global_load_dwordx4 %2, %6, off sc0 sc1\n\tglobal_load_dwordx4 %3, %7, off sc0 sc1\n\ts_waitcnt vmcnt(0)"
                     : "=&v"(r[0]), "=&v"(r[1]), "=&v"(r[2]), "=&v"(r[3])
                     : "v"(src), "v"(src + 4096), "v"(src + 8192), "v"(src + 12288) : "memory");
#pragma unroll
        for (int i = 0; i < 4; i++)
#pragma unroll
          for (int e = 0; e < 4; e++) {
            y[i >> 1][2 * (i & 1) + (e >> 1)][2 * (e & 1)] += __builtin_bit_cast(float, r[i][e] << 16);
            y[i >> 1][2 * (i & 1) + (e >> 1)][2 * (e & 1) + 1] += __builtin_bit_cast(float, r[i][e] & 0xffff0000u);
          }
      } else {
        asm volatile("global_load_dwordx4 %0, %8, off sc0 sc1\n\tglobal_load_dwordx4 %1, %9, off sc0 sc1\n\t"
                     "global_load_dwordx4 %2, %10, off sc0 sc1\n\tglobal_load_dwordx4 %3, %11, off sc0 sc1\n\t"
                     "global_load_dwordx4 %4, %12, off sc0 sc1\n\tglobal_load_dwordx4 %5, %13, off sc0 sc1\n\t"
                     "global_load_dwordx4 %6, %14, off sc0 sc1\n\tglobal_load_dwordx4 %7, %15, off sc0 sc1\n\ts_waitcnt vmcnt(0)"
                     : "=&v"(r[0]), "=&v"(r[1]), "=&v"(r[2]), "=&v"(r[3]), "=&v"(r[4 % PI]), "=&v"(r[5 % PI]), "=&v"(r[6 % PI]), "=&v"(r[7 % PI])
                     : "v"(src), "v"(src + 4096), "v"(src + 8192), "v"(src + 12288), "v"(src + 16384), "v"(src + 20480), "v"(src + 24576), "v"(src + 28672) : "memory");
#pragma unroll
        for (int i = 0; i < PI; i++) y[(i >> 2) & 1][i & 3] += __builtin_bit_cast(f32x4, r[i]);
      }
    }
  }
  TS(4);
  float mx = 0.f;
#pragma unroll
  for (int m = 0; m < 2; m++)
#pragma unroll
    for (int n = 0; n < 4; n++)
#pragma unroll
      for (int e = 0; e < 4; e++) { y[m][n][e] = fmaxf(y[m][n][e], 0.f); mx = fmaxf(mx, y[m][n][e]); }
#pragma unroll
  for (int o = 32; o; o >>= 1) mx = fmaxf(mx, __shfl_xor(mx, o));
  LAS u32* sm = (LAS u32*)(smem + SMALL);
  if (lane == 0) sm[wave] = __builtin_bit_cast(u32, mx);
  asm volatile("s_waitcnt lgkmcnt(0)" ::: "memory");
  __builtin_amdgcn_s_barrier();
  const size_t gran = WS_AMAX + (size_t)(call & 1) * 16384;
  if (wave == 0) {
    const u32x4 v = *(LAS u32x4*)sm;
    const u32 m4 = max(max(v[0], v[1]), max(v[2], v[3]));
    if (lane < NR) st8_sys(peer_ws + gran + (rank * 256 + b) * 8, u32x2{m4, call});
  }
  TS(5);
  u32 gm;
  {
    const char* pg = ws + gran + wave * 4096 + lane * 16;
    while (true) {
      u32x4 r0, r1, r2, r3; u32 tv;
      asm volatile("global_load_dwordx4 %0, %5, off sc0 sc1\n\tglobal_load_dwordx4 %1, %5, off offset:1024 sc0 sc1\n\t"
                   "global_load_dwordx4 %2, %5, off offset:2048 sc0 sc1\n\tglobal_load_dwordx4 %3, %5, off offset:3072 sc0 sc1\n\t"
                   "global_load_dword %4, %6, off sc0 sc1\n\ts_waitcnt vmcnt(0)"
                   : "=&v"(r0), "=&v"(r1), "=&v"(r2), "=&v"(r3), "=&v"(tv) : "v"(pg), "v"(tmo) : "memory");
      const u32 lo = min(min(min(r0[1], r0[3]), min(r1[1], r1[3])), min(min(r2[1], r2[3]), min(r3[1], r3[3])));
      gm = max(max(max(r0[0], r0[2]), max(r1[0], r1[2])), max(max(r2[0], r2[2]), max(r3[0], r3[2])));
      if (__builtin_amdgcn_readfirstlane(tv) == call || now() - t0 > DEADLINE) { bad = true; break; }
      if (__all(lo >= call)) break;
      __builtin_amdgcn_s_sleep(1);
    }
  }
  if (bad && lane == 0) st4_sys(tmo, call);
#pragma unroll
  for (int o = 32; o; o >>= 1) gm = max(gm, (u32)__shfl_xor((int)gm, o));
  if (lane == 0) sm[4 + wave] = bad ? 0xffffffffu : gm;
  asm volatile("s_waitcnt lgkmcnt(0)" ::: "memory");
  __builtin_amdgcn_s_barrier();
  {
    const u32x4 v = *(LAS u32x4*)(sm + 4);
    gm = max(max(v[0], v[1]), max(v[2], v[3]));
  }
  TS(6);
  bad = gm == 0xffffffffu;
  const float amax = __builtin_bit_cast(float, gm);
  const float scale = amax / 127.0f;
  const float inv = scale > 0.f ? 1.0f / scale : 0.f;
  const int col0 = nt * BN + (ks >> 1) * 128 + wn * 64;
  const u32 dest = (u32)col0 / NCOL;
  char* outp = out_all[dest & 7] + (size_t)(rank * MS + mt * BM + (ks & 1) * 64 + wm * 32 + li) * NCOL + (col0 % NCOL) + g * 16;
#pragma unroll
  for (int m = 0; m < 2; m++) {
    u32 d[4];
#pragma unroll
    for (int n = 0; n < 4; n++) {
      u32 q[4];
#pragma unroll
      for (int e = 0; e < 4; e++) q[e] = __builtin_bit_cast(u32, fminf(__builtin_fmaf(y[m][n][e], inv, 8388608.f), 8388735.f));
      d[n] = __builtin_amdgcn_perm(__builtin_amdgcn_perm(q[3], q[2], 0x0c0c0400u), __builtin_amdgcn_perm(q[1], q[0], 0x0c0c0400u), 0x05040100u);
    }
    u32x4 v = (g & 1) ? u32x4{d[1], d[0], d[3], d[2]} : u32x4{d[0], d[1], d[2], d[3]};
    if (bad) v = u32x4{0x80808080u, 0x80808080u, 0x80808080u, 0x80808080u};
    st16_sys(outp + m * 16 * NCOL, v);
  }
  wait_vm<0>();
  TS(7);
  const int slot32 = ((col0 % NCOL) / 128) * 8 + mt * 2 + (ks & 1);
  if (lane == (int)dest) st4_sys(peer_ws + WS_DONE + ((rank * 32 + slot32) * 4 + wave) * 4, call);
  if (b == 0) {
    const char* pd = ws + WS_DONE + wave * 1024 + lane * 16;
    while (true) {
      u32x4 r; u32 tv;
      asm volatile("global_load_dwordx4 %0, %2, off sc0 sc1\n\tglobal_load_dword %1, %3, off sc0 sc1\n\ts_waitcnt vmcnt(0)"
                   : "=&v"(r), "=&v"(tv) : "v"(pd), "v"(tmo) : "memory");
      if (__all(min(min(r[0], r[1]), min(r[2], r[3])) >= call)) break;
      if (__builtin_amdgcn_readfirstlane(tv) == call || now() - t0 > DEADLINE) {
        if (lane == 0) { st4_sys(tmo, call); st4_sys(out_all[rank & 7] + (size_t)rank * MS * NCOL, 0x80808080u); }
        break;
      }
      __builtin_amdgcn_s_sleep(1);
    }
  }
  TS(8);
}

extern "C" void kernel_launch(int rank, int nranks, ncclComm_t comm, hipStream_t stream, void* const* d_in, const int* in_sizes, int n_in,
                              void* d_out, int out_size, const int* in_dtypes, int out_dtype, void* const* d_out_all,
                              void* const* d_ws_all, size_t ws_bytes) {
  if (nranks != NR || (unsigned)rank >= (unsigned)NR || n_in < 2 || in_sizes[0] != MS * KD || in_sizes[1] != KD * ND ||
      out_size != NR * MS * NCOL || ws_bytes < WS_END) {
    fprintf(stderr, "unexpected problem shape\n");
    return;
  }
  hipError_t e = hipFuncSetAttribute((const void*)gemm_a2a, hipFuncAttributeMaxDynamicSharedMemorySize, LDS_BYTES);
  if (e != hipSuccess) { fprintf(stderr, "HIP error %s\n", hipGetErrorString(e)); return; }
  gemm_a2a<<<256, 512, LDS_BYTES, stream>>>((const char*)d_in[0], (const char*)d_in[1], (char* const*)d_out_all,
                                             (char* const*)d_ws_all, (char*)d_ws_all[rank], rank);
}
```

```cpp
#define CFG_XOR 3
#define CFG_P16 1
#define CFG_NAGG 8
#define CFG_REP 8
#include <hip/hip_runtime.h>
#include <rccl/rccl.h>
#include <cstdio>

typedef unsigned u32;
typedef unsigned long long u64;
typedef u32 u32x4 __attribute__((ext_vector_type(4)));
typedef u32 u32x2 __attribute__((ext_vector_type(2)));
typedef float f32x4 __attribute__((ext_vector_type(4)));
typedef __bf16 bf16x4 __attribute__((ext_vector_type(4)));
typedef __bf16 bf16x8 __attribute__((ext_vector_type(8)));
typedef __bf16 bf16x2 __attribute__((ext_vector_type(2)));
#define LAS __attribute__((address_space(3)))

#define CFG_BN 128

#define CFG_SLP 1
constexpr int NAGG = CFG_NAGG, REP = CFG_REP;
constexpr int NR = 8, MS = 512, KD = 4096, ND = 4096, NCOL = 512;
constexpr int BM = 128, BN = CFG_BN, BK = 64, KS = BN / 64;
constexpr int NF = BN / 32;
constexpr int NSTEP = KD / KS / BK;
constexpr int NS = (BN == 128) ? 4 : 3;
constexpr int XS = BM * BK * 2, WS_ = BK * BN * 2;
constexpr int XRING = 0, WRING = NS * XS, SMALL = NS * (XS + WS_);
constexpr int LDS_BYTES = SMALL + 256;
constexpr int XI = XS / 4096, WI = WS_ / 4096, LI = XI + WI;
constexpr int PI = CFG_P16 ? 4 : 8;
constexpr u64 DEADLINE = 100000000ull;

constexpr size_t WS_CTR = 0;
constexpr size_t WS_TMO = 32768 + 4;
constexpr size_t WS_DONE = 65536;
constexpr size_t WS_LMAX = 98304;
constexpr size_t WS_XF = 131072;
constexpr size_t WS_TS = 262144;
constexpr size_t WS_GMAX = 786432;
constexpr size_t WS_EX = 1u << 20;
constexpr size_t WS_END = WS_EX + (size_t)256 * (KS - 1) * 4 * PI * 1024;

template <int N> __device__ __forceinline__ void wait_vm() { asm volatile("s_waitcnt vmcnt(%0)" ::"i"(N) : "memory"); }
__device__ __forceinline__ void st16_sys(void* p, u32x4 v) { asm volatile("global_store_dwordx4 %0, %1, off sc0 sc1\n\ts_nop 1" ::"v"(p), "v"(v) : "memory"); }
__device__ __forceinline__ void st8_sys(void* p, u32x2 v) { asm volatile("global_store_dwordx2 %0, %1, off sc0 sc1\n\ts_nop 1" ::"v"(p), "v"(v) : "memory"); }
__device__ __forceinline__ void st4_sys(void* p, u32 v) { asm volatile("global_store_dword %0, %1, off sc0 sc1\n\ts_nop 1" ::"v"(p), "v"(v) : "memory"); }
__device__ __forceinline__ u32 ld4_sys(const void* p) { u32 v; asm volatile("global_load_dword %0, %1, off sc0 sc1\n\ts_waitcnt vmcnt(0)" : "=&v"(v) : "v"(p) : "memory"); return v; }
__device__ __forceinline__ u64 now() { return __builtin_amdgcn_s_memrealtime(); }

__device__ __forceinline__ u32 wave_max(u32 v) {
  v = max(v, (u32)__builtin_amdgcn_update_dpp(0, (int)v, 0xb1, 0xf, 0xf, true));
  v = max(v, (u32)__builtin_amdgcn_update_dpp(0, (int)v, 0x4e, 0xf, 0xf, true));
  v = max(v, (u32)__builtin_amdgcn_update_dpp(0, (int)v, 0x141, 0xf, 0xf, true));
  v = max(v, (u32)__builtin_amdgcn_update_dpp(0, (int)v, 0x140, 0xf, 0xf, true));
  return max(max((u32)__builtin_amdgcn_readlane(v, 0), (u32)__builtin_amdgcn_readlane(v, 16)),
             max((u32)__builtin_amdgcn_readlane(v, 32), (u32)__builtin_amdgcn_readlane(v, 48)));
}
__device__ __forceinline__ u32x2 ld8_sys(const void* p) { u32x2 v; asm volatile("global_load_dwordx2 %0, %1, off sc0 sc1\n\ts_waitcnt vmcnt(0)" : "=&v"(v) : "v"(p) : "memory"); return v; }

__global__ __launch_bounds__(512, 2) void gemm_a2a(const char* __restrict__ x, const char* __restrict__ w,
                                                   char* const* out_all, char* const* ws_all, char* ws, int rank) {
  extern __shared__ __attribute__((aligned(16))) char smem[];
  const int b = blockIdx.x, lane = threadIdx.x & 63;
  const int wave = __builtin_amdgcn_readfirstlane(threadIdx.x >> 6);
  const int xcd = b & 7, idx = b >> 3;
  const int ks = xcd % KS, mt = idx & 3, ntl = idx >> 2, nt = (xcd / KS) * 8 + ntl;
  const int kx = (mt ^ ntl) & CFG_XOR;
  if (wave >= 4) {
    const int lw = wave - 4;
    auto rx = __builtin_amdgcn_make_buffer_rsrc((void*)x, 0, MS * KD * 2, 0x00020000);
    auto rw = __builtin_amdgcn_make_buffer_rsrc((void*)w, 0, KD * ND * 2, 0x00020000);
    int vx[2];
#pragma unroll
    for (int h = 0; h < 2; h++) { const int r = (lane >> 3) + 8 * h; vx[h] = (lane >> 3) * 8192 + (((lane & 7) ^ ((r >> 1) & 7)) << 4); }
    constexpr int LPR = BN / 8, RPI = 64 / LPR;
    int vw[WI];
#pragma unroll
    for (int i = 0; i < WI; i++) { const int k = (lw * WI + i) * RPI + lane / LPR; vw[i] = (lane / LPR) * 8192 + (((lane % LPR) ^ (k & 11)) << 4); }
    const int sx0 = (mt * BM + lw * XI * 8) * 8192 + ks * (KD / KS) * 2;
    const int sw0 = (ks * (KD / KS) + lw * WI * RPI) * 8192 + nt * BN * 2;
    auto issue = [&](int s, int slot) {
      const int sp = s ^ kx;
#pragma unroll
      for (int i = 0; i < XI; i++)
        __builtin_amdgcn_raw_ptr_buffer_load_lds(rx, (LAS void*)(smem + XRING + slot * XS + (lw * XI + i) * 1024), 16, vx[i & 1], sx0 + i * 8 * 8192 + sp * (BK * 2), 0, 0);
#pragma unroll
      for (int i = 0; i < WI; i++)
        __builtin_amdgcn_raw_ptr_buffer_load_lds(rw, (LAS void*)(smem + WRING + slot * WS_ + (lw * WI + i) * 1024), 16, vw[i], sw0 + (i * RPI + sp * BK) * 8192, 0, 0);
    };
    issue(0, 0); issue(1, 1);
    wait_vm<LI>();
    __builtin_amdgcn_s_barrier();
#pragma unroll
    for (int s = 2; s < NS; s++) issue(s, s);
    int slot = 0;
#pragma unroll 1
    for (int j = 1; j <= NSTEP - NS; j++) {
      wait_vm<(NS - 2) * LI>();
      __builtin_amdgcn_s_barrier();
      issue(j - 1 + NS, slot);
      slot = slot + 1 == NS ? 0 : slot + 1;
    }
    if (NS == 4) { wait_vm<2 * LI>(); __builtin_amdgcn_s_barrier(); }
    wait_vm<LI>(); __builtin_amdgcn_s_barrier();
    wait_vm<0>(); __builtin_amdgcn_s_barrier();
    if (lw == 0 && b < NAGG) {
      const u32 call = ((LAS u32*)(smem + SMALL))[2];
      const u64 t0 = now();
      char* pw = ws_all[lane & 7];
      u32* tmo = (u32*)(ws + WS_TMO);
      constexpr int PER = 256 / NAGG, LPL = PER > 64 ? PER / 64 : 1;
      u32 gm = 0;
      bool bad = false;
      const char* pg = ws + WS_LMAX + (size_t)(((lane * LPL) % PER) * NAGG + b) * 128;
      while (true) {
        u32x2 r[4]; u32 tv;
        if (LPL == 4) {
          asm volatile("global_load_dwordx2 %0, %5, off sc0 sc1\n\tglobal_load_dwordx2 %1, %5, off offset:128 sc0 sc1\n\t"
                       "global_load_dwordx2 %2, %5, off offset:256 sc0 sc1\n\tglobal_load_dwordx2 %3, %5, off offset:384 sc0 sc1\n\t"
                       "global_load_dword %4, %6, off sc0 sc1\n\ts_waitcnt vmcnt(0)"
                       : "=&v"(r[0]), "=&v"(r[1]), "=&v"(r[2]), "=&v"(r[3]), "=&v"(tv) : "v"(pg), "v"(tmo) : "memory");
        } else {
          asm volatile("global_load_dwordx2 %0, %2, off sc0 sc1\n\tglobal_load_dword %1, %3, off sc0 sc1\n\ts_waitcnt vmcnt(0)"
                       : "=&v"(r[0]), "=&v"(tv) : "v"(pg), "v"(tmo) : "memory");
          r[1] = r[2] = r[3] = r[0];
        }
        gm = max(max(r[0][0], r[1][0]), max(r[2][0], r[3][0]));
        if (__all(min(min(r[0][1], r[1][1]), min(r[2][1], r[3][1])) >= call)) break;
        if (__builtin_amdgcn_readfirstlane(tv) == call || now() - t0 > DEADLINE) { bad = true; break; }
      }
      gm = wave_max(gm);
      if (bad) { if (lane == 0) st4_sys(tmo, call); gm = 0xffffffffu; }
      if (lane < NR * REP) st8_sys(pw + WS_GMAX + (size_t)(((call & 1) * REP + (lane >> 3)) * (NR * NAGG) + rank * NAGG + b) * 128, u32x2{gm, call});
    }
    return;
  }
  const int wm = wave >> 1, wn = wave & 1, g = lane >> 4, li = lane & 15;
  u32* ctr = (u32*)(ws + WS_CTR + b * 128);
  u32* tmo = (u32*)(ws + WS_TMO);
  const u32 call = __builtin_amdgcn_readfirstlane(ld4_sys(ctr)) + 1;
  char* peer_ws = ws_all[lane & 7];
  LAS u32* sm = (LAS u32*)(smem + SMALL);
  if (wave == 0 && lane < 3) sm[lane] = lane == 2 ? call : 0;
  f32x4 acc[4][NF];
#pragma unroll
  for (int m = 0; m < 4; m++)
#pragma unroll
    for (int n = 0; n < NF; n++) acc[m][n] = f32x4{0.f, 0.f, 0.f, 0.f};
  u32 ax[2], aw[4];
#pragma unroll
  for (int kk = 0; kk < 2; kk++) ax[kk] = (u32)(size_t)smem + XRING + (32 * wm + li) * 128 + (((4 * kk + g) ^ ((li >> 1) & 7)) << 4);
  {
    const int q = li >> 2, p = li & 3;
#pragma unroll
    for (int n = 0; n < 4; n++)
      aw[n] = (u32)(size_t)smem + WRING + (8 * g + q) * (BN * 2) + (((8 * wn + 2 * p + (n >> 1)) ^ q ^ ((g & 1) << 3)) << 4) + (((n ^ p) & 1) << 3);
  }
  bf16x8 fx[2][4], fw[2][NF];
  auto rd = [&](int buf, int slot, int kk) {
#pragma unroll
    for (int n = 0; n < NF; n++) {
      const u32 a = aw[n & 3] + slot * WS_ + kk * 32 * BN * 2 + (n >> 2) * 256;
      const bf16x4 lo = __builtin_amdgcn_ds_read_tr16_b64_v4bf16((LAS bf16x4*)(size_t)a);
      const bf16x4 hi = __builtin_amdgcn_ds_read_tr16_b64_v4bf16((LAS bf16x4*)(size_t)(a + 4 * BN * 2));
      fw[buf][n] = __builtin_shufflevector(lo, hi, 0, 1, 2, 3, 4, 5, 6, 7);
    }
#pragma unroll
    for (int m = 0; m < 4; m++) fx[buf][m] = *(LAS bf16x8*)(size_t)(ax[kk] + slot * XS + ((m & 1) * 16 + (m >> 1) * 64) * 128);
  };
  auto mm = [&](int buf) {
#pragma unroll
    for (int n = 0; n < NF; n++)
#pragma unroll
      for (int m = 0; m < 4; m++) {
        acc[m][n] = __builtin_amdgcn_mfma_f32_16x16x32_bf16(fw[buf][n], fx[buf][m], acc[m][n], 0, 0, 0);
      }
  };
  auto mix = [&]() {
#pragma unroll
    for (int i = 0; i < 2 * NF + 4; i++) { __builtin_amdgcn_sched_group_barrier(0x008, 1, 0); __builtin_amdgcn_sched_group_barrier(0x100, 1, 0); }
    __builtin_amdgcn_sched_group_barrier(0x008, 4 * NF - (2 * NF + 4), 0);
  };
  asm volatile("s_waitcnt lgkmcnt(0)" ::: "memory");
  __builtin_amdgcn_s_barrier();
  __builtin_amdgcn_sched_barrier(0);
  rd(0, 0, 0);
  for (int j0 = 0; j0 < NSTEP; j0 += NS) {
#pragma unroll
    for (int u = 0; u < NS; u++) {
      if (NSTEP % NS != 0 && j0 + u >= NSTEP) break;
      rd(1, u, 1); mm(0); mix();
      __builtin_amdgcn_sched_barrier(0);
      if (j0 + u + 1 < NSTEP) {
        asm volatile("s_waitcnt lgkmcnt(0)" ::: "memory");
        __builtin_amdgcn_s_barrier();
        __builtin_amdgcn_sched_barrier(0);
        rd(0, (u + 1) % NS, 0); mm(1); mix();
        __builtin_amdgcn_sched_barrier(0);
      } else {
        mm(1);
      }
    }
  }
  bool bad = false;
  const u64 t0 = now();
  f32x4 y[2][4];
  {
    char* ex = ws + WS_EX;
#pragma unroll
    for (int o = 0; o < KS; o++) {
      if (o == ks) {
#pragma unroll
        for (int m = 0; m < 2; m++)
#pragma unroll
          for (int n = 0; n < 4; n++) y[m][n] = acc[2 * (o & 1) + m][4 * (o >> 1) + n];
      } else {
        const int d = b - ks + o;
        char* dst = ex + ((size_t)(d * (KS - 1) + ((o ^ ks) - 1)) * PI * 4 + wave) * 1024 + lane * 16;
#pragma unroll
        for (int i = 0; i < PI; i++) {
          u32x4 v;
          if (CFG_P16) {
            const f32x4 a = acc[2 * (o & 1) + (i >> 1)][4 * (o >> 1) + 2 * (i & 1)], c = acc[2 * (o & 1) + (i >> 1)][4 * (o >> 1) + 2 * (i & 1) + 1];
            v[0] = __builtin_bit_cast(u32, bf16x2{(__bf16)a[0], (__bf16)a[1]}); v[1] = __builtin_bit_cast(u32, bf16x2{(__bf16)a[2], (__bf16)a[3]});
            v[2] = __builtin_bit_cast(u32, bf16x2{(__bf16)c[0], (__bf16)c[1]}); v[3] = __builtin_bit_cast(u32, bf16x2{(__bf16)c[2], (__bf16)c[3]});
          } else {
            v = __builtin_bit_cast(u32x4, acc[2 * (o & 1) + (i >> 2)][4 * (o >> 1) + (i & 3)]);
          }
          st16_sys(dst + i * 4096, v);
        }
      }
    }
    wait_vm<0>();
    if (lane < KS && lane != ks) st4_sys(ws + WS_XF + ((b - ks + lane) * 4 + wave) * 128 + ks * 4, call);
    if (wave == 0 && lane == 63) st4_sys(ctr, call);
    const char* pf = lane < KS ? ws + WS_XF + (b * 4 + wave) * 128 + lane * 4 : (const char*)tmo;
    while (true) {
      const u32 f = ld4_sys(pf);
      if (__builtin_amdgcn_readlane(f, 63) == call || now() - t0 > DEADLINE) { bad = true; break; }
      if (__all(lane >= KS || lane == ks || f >= call)) break;
      __builtin_amdgcn_s_sleep(1);
    }
#pragma unroll
    for (int j = 0; j < KS - 1; j++) {
      const char* src = ex + ((size_t)(b * (KS - 1) + j) * PI * 4 + wave) * 1024 + lane * 16;
      u32x4 r[PI];
      if (CFG_P16) {
        asm volatile("global_load_dwordx4 %0, %4, off sc0 sc1\n\tglobal_load_dwordx4 %1, %5, off sc0 sc1\n\t"
                     "global_load_dwordx4 %2, %6, off sc0 sc1\n\tglobal_load_dwordx4 %3, %7, off sc0 sc1\n\ts_waitcnt vmcnt(0)"
                     : "=&v"(r[0]), "=&v"(r[1]), "=&v"(r[2]), "=&v"(r[3])
                     : "v"(src), "v"(src + 4096), "v"(src + 8192), "v"(src + 12288) : "memory");
#pragma unroll
        for (int i = 0; i < 4; i++)
#pragma unroll
          for (int e = 0; e < 4; e++) {
            y[i >> 1][2 * (i & 1) + (e >> 1)][2 * (e & 1)] += __builtin_bit_cast(float, r[i][e] << 16);
            y[i >> 1][2 * (i & 1) + (e >> 1)][2 * (e & 1) + 1] += __builtin_bit_cast(float, r[i][e] & 0xffff0000u);
          }
      } else {
        asm volatile("global_load_dwordx4 %0, %8, off sc0 sc1\n\tglobal_load_dwordx4 %1, %9, off sc0 sc1\n\t"
                     "global_load_dwordx4 %2, %10, off sc0 sc1\n\tglobal_load_dwordx4 %3, %11, off sc0 sc1\n\t"
                     "global_load_dwordx4 %4, %12, off sc0 sc1\n\tglobal_load_dwordx4 %5, %13, off sc0 sc1\n\t"
                     "global_load_dwordx4 %6, %14, off sc0 sc1\n\tglobal_load_dwordx4 %7, %15, off sc0 sc1\n\ts_waitcnt vmcnt(0)"
                     : "=&v"(r[0]), "=&v"(r[1]), "=&v"(r[2]), "=&v"(r[3]), "=&v"(r[4 % PI]), "=&v"(r[5 % PI]), "=&v"(r[6 % PI]), "=&v"(r[7 % PI])
                     : "v"(src), "v"(src + 4096), "v"(src + 8192), "v"(src + 12288), "v"(src + 16384), "v"(src + 20480), "v"(src + 24576), "v"(src + 28672) : "memory");
#pragma unroll
        for (int i = 0; i < PI; i++) y[(i >> 2) & 1][i & 3] += __builtin_bit_cast(f32x4, r[i]);
      }
    }
  }
  float mx = 0.f;
#pragma unroll
  for (int m = 0; m < 2; m++)
#pragma unroll
    for (int n = 0; n < 4; n++)
#pragma unroll
      for (int e = 0; e < 4; e++) { y[m][n][e] = fmaxf(y[m][n][e], 0.f); mx = fmaxf(mx, y[m][n][e]); }
  {
    u32 wmx = wave_max(__builtin_bit_cast(u32, mx));
    if (bad) wmx = 0xffffffffu;
    if (lane == 0) {
      u32 tk, m4;
      asm volatile("ds_max_u32 %2, %3\n\tds_add_rtn_u32 %0, %2, %4 offset:4\n\tds_read_b32 %1, %2\n\ts_waitcnt lgkmcnt(0)"
                   : "=&v"(tk), "=&v"(m4) : "v"((u32)(size_t)sm), "v"(wmx), "v"(1u) : "memory");
      if (tk == 3) st8_sys(ws + WS_LMAX + b * 128, u32x2{m4, call});
    }
  }
  u32 gm;
  {
    const char* pg = lane < NR * NAGG ? ws + WS_GMAX + (size_t)(((call & 1) * REP + b % REP) * (NR * NAGG) + lane) * 128 : (const char*)tmo - 4;
    if (NAGG == 8) pg = ws + WS_GMAX + (size_t)(((call & 1) * REP + b % REP) * (NR * NAGG) + lane) * 128;
    while (true) {
      const u32x2 r = ld8_sys(pg);
      gm = lane < NR * NAGG ? r[0] : 0;
      if (__all(lane >= NR * NAGG || r[1] >= call)) break;
      if ((NAGG < 8 && __builtin_amdgcn_readlane(r[1], 63) == call) || now() - t0 > DEADLINE) { bad = true; break; }
      __builtin_amdgcn_s_sleep(CFG_SLP);
    }
  }
  if (bad && lane == 0) st4_sys(tmo, call);
  gm = bad ? 0xffffffffu : wave_max(gm);
  bad = gm == 0xffffffffu;
  const float amax = __builtin_bit_cast(float, gm);
  const float scale = amax / 127.0f;
  const float inv = scale > 0.f ? 1.0f / scale : 0.f;
  const int col0 = nt * BN + (ks >> 1) * 128 + wn * 64;
  const u32 dest = (u32)col0 / NCOL;
  char* outp = out_all[dest & 7] + (size_t)(rank * MS + mt * BM + (ks & 1) * 64 + wm * 32 + li) * NCOL + (col0 % NCOL) + g * 16;
#pragma unroll
  for (int m = 0; m < 2; m++) {
    u32 d[4];
#pragma unroll
    for (int n = 0; n < 4; n++) {
      u32 q[4];
#pragma unroll
      for (int e = 0; e < 4; e++) q[e] = __builtin_bit_cast(u32, fminf(__builtin_fmaf(y[m][n][e], inv, 8388608.f), 8388735.f));
      d[n] = __builtin_amdgcn_perm(__builtin_amdgcn_perm(q[3], q[2], 0x0c0c0400u), __builtin_amdgcn_perm(q[1], q[0], 0x0c0c0400u), 0x05040100u);
    }
    u32x4 v = (g & 1) ? u32x4{d[1], d[0], d[3], d[2]} : u32x4{d[0], d[1], d[2], d[3]};
    if (bad) v = u32x4{0x80808080u, 0x80808080u, 0x80808080u, 0x80808080u};
    st16_sys(outp + m * 16 * NCOL, v);
  }
  wait_vm<0>();
  const int slot32 = ((col0 % NCOL) / 128) * 8 + mt * 2 + (ks & 1);
  if (lane == (int)dest) st4_sys(peer_ws + WS_DONE + (rank * 32 + slot32) * 128 + wave * 4, call);
  {
    const char* pd = lane == 0 ? ws + WS_DONE + b * 128 + wave * 4 : (const char*)tmo;
    while (true) {
      const u32 f = ld4_sys(pd);
      if (__builtin_amdgcn_readfirstlane(f) >= call) break;
      if (__builtin_amdgcn_readlane(f, 63) == call || now() - t0 > DEADLINE) {
        if (lane == 0) { st4_sys(tmo, call); st4_sys(out_all[rank & 7] + (size_t)rank * MS * NCOL, 0x80808080u); }
        break;
      }
      __builtin_amdgcn_s_sleep(CFG_SLP);
    }
  }
}

static bool launch_setup() {
  hipError_t e = hipFuncSetAttribute((const void*)gemm_a2a, hipFuncAttributeMaxDynamicSharedMemorySize, LDS_BYTES);
  if (e != hipSuccess) { fprintf(stderr, "HIP error %s\n", hipGetErrorString(e)); return false; }
  return true;
}
static void launch_impl(int rank, hipStream_t stream, const void* x, const void* w, char* const* out_dev, char* const* ws_dev,
                        char* const* out_host, char* const* ws_host) {
  gemm_a2a<<<256, 512, LDS_BYTES, stream>>>((const char*)x, (const char*)w, out_dev, ws_dev, ws_host[rank], rank);
}
extern "C" void kernel_launch(int rank, int nranks, ncclComm_t comm, hipStream_t stream, void* const* d_in, const int* in_sizes, int n_in,
                              void* d_out, int out_size, const int* in_dtypes, int out_dtype, void* const* d_out_all,
                              void* const* d_ws_all, size_t ws_bytes) {
  if (nranks != NR || (unsigned)rank >= (unsigned)NR || n_in < 2 || in_sizes[0] != MS * KD || in_sizes[1] != KD * ND ||
      out_size != NR * MS * NCOL || ws_bytes < WS_END) {
    fprintf(stderr, "unexpected problem shape\n");
    return;
  }
  if (!launch_setup()) return;
  launch_impl(rank, stream, d_in[0], d_in[1], (char* const*)d_out_all, (char* const*)d_ws_all, (char* const*)d_out_all, (char* const*)d_ws_all);
}
```

```cpp
#define CFG_PDLY 275
#include <hip/hip_runtime.h>
#include <rccl/rccl.h>
#include <cstdio>

typedef unsigned u32;
typedef unsigned long long u64;
typedef u32 u32x4 __attribute__((ext_vector_type(4)));
typedef u32 u32x2 __attribute__((ext_vector_type(2)));
typedef float f32x4 __attribute__((ext_vector_type(4)));
typedef __bf16 bf16x4 __attribute__((ext_vector_type(4)));
typedef __bf16 bf16x8 __attribute__((ext_vector_type(8)));
typedef __bf16 bf16x2 __attribute__((ext_vector_type(2)));
#define LAS __attribute__((address_space(3)))

#define CFG_XOR 3
#define CFG_PSLP 2
#define CFG_RF 1
#define CFG_EXP 0
#define CFG_P1 0
#define CFG_OST 1
#define CFG_REP 8
constexpr int NAGG = 8, REP = CFG_REP;
constexpr int NR = 8, MS = 512, KD = 4096, ND = 4096, NCOL = 512;
constexpr int NSTEP = KD / 2 / 64, NS = 4, BLK = 16384;
constexpr int XRING = 0, WRING = NS * BLK, SMALL = 2 * NS * BLK, LDS_BYTES = SMALL + 256;
constexpr u64 DEADLINE = 100000000ull;

constexpr size_t WS_CTR = 0;
constexpr size_t WS_TMO = 32768 + 4;
constexpr size_t WS_LMAX = 98304;
constexpr size_t WS_XF = 131072;
constexpr size_t WS_GMAX = 524288;
constexpr size_t WS_TS = 786432;
constexpr size_t WS_EX = 1u << 20;
constexpr size_t WS_STG = WS_EX + (size_t)256 * 16384;
constexpr size_t WS_MSK = WS_STG + (size_t)1024 * 2048;
constexpr size_t WS_END = WS_MSK + (size_t)2 * 1024 * 512;

template <int N> __device__ __forceinline__ void wait_vm() { asm volatile("s_waitcnt vmcnt(%0)" ::"i"(N) : "memory"); }
__device__ __forceinline__ void st16_sys(void* p, u32x4 v) { asm volatile("global_store_dwordx4 %0, %1, off sc0 sc1\n\ts_nop 1" ::"v"(p), "v"(v) : "memory"); }
template <int O> __device__ __forceinline__ void st16_so(const char* base, u32 off, u32x4 v) {
  asm volatile("global_store_dwordx4 %0, %1, %2 offset:%3 sc0 sc1\n\ts_nop 1" ::"v"(off), "v"(v), "s"(base), "i"(O) : "memory");
}
__device__ __forceinline__ void st8_sys(void* p, u32x2 v) { asm volatile("global_store_dwordx2 %0, %1, off sc0 sc1\n\ts_nop 1" ::"v"(p), "v"(v) : "memory"); }
__device__ __forceinline__ void st4_sys(void* p, u32 v) { asm volatile("global_store_dword %0, %1, off sc0 sc1\n\ts_nop 1" ::"v"(p), "v"(v) : "memory"); }
__device__ __forceinline__ u32 ld4_sys(const void* p) { u32 v; asm volatile("global_load_dword %0, %1, off sc0 sc1\n\ts_waitcnt vmcnt(0)" : "=&v"(v) : "v"(p) : "memory"); return v; }
__device__ __forceinline__ u32x2 ld8_sys(const void* p) { u32x2 v; asm volatile("global_load_dwordx2 %0, %1, off sc0 sc1\n\ts_waitcnt vmcnt(0)" : "=&v"(v) : "v"(p) : "memory"); return v; }
__device__ __forceinline__ u64 now() { return __builtin_amdgcn_s_memrealtime(); }
__device__ __forceinline__ u32 vm_peek() { u32 v; asm volatile("s_getreg_b32 %0, hwreg(HW_REG_IB_STS, 0, 4)" : "=s"(v) :: "memory"); return v; }
__device__ __forceinline__ char* uni(char* p) {
  const u64 v = (u64)p;
  return (char*)(((u64)(u32)__builtin_amdgcn_readfirstlane((u32)(v >> 32)) << 32) | (u32)__builtin_amdgcn_readfirstlane((u32)v));
}
__device__ __forceinline__ u32 clk() { return (u32)__builtin_amdgcn_s_memtime(); }
__device__ __forceinline__ u32 pk2(float a, float b) { return __builtin_bit_cast(u32, bf16x2{(__bf16)a, (__bf16)b}); }
__device__ __forceinline__ u32x4 pk8(f32x4 a, f32x4 c) { return u32x4{pk2(a[0], a[1]), pk2(a[2], a[3]), pk2(c[0], c[1]), pk2(c[2], c[3])}; }
__device__ __forceinline__ float lo16(u32 v) { return __builtin_bit_cast(float, v << 16); }
__device__ __forceinline__ float hi16(u32 v) { return __builtin_bit_cast(float, v & 0xffff0000u); }

__device__ __forceinline__ u32 wave_max(u32 v) {
  v = max(v, (u32)__builtin_amdgcn_update_dpp(0, (int)v, 0xb1, 0xf, 0xf, true));
  v = max(v, (u32)__builtin_amdgcn_update_dpp(0, (int)v, 0x4e, 0xf, 0xf, true));
  v = max(v, (u32)__builtin_amdgcn_update_dpp(0, (int)v, 0x141, 0xf, 0xf, true));
  v = max(v, (u32)__builtin_amdgcn_update_dpp(0, (int)v, 0x140, 0xf, 0xf, true));
  return max(max((u32)__builtin_amdgcn_readlane(v, 0), (u32)__builtin_amdgcn_readlane(v, 16)),
             max((u32)__builtin_amdgcn_readlane(v, 32), (u32)__builtin_amdgcn_readlane(v, 48)));
}
__device__ __forceinline__ void wg_max(u32 wmx, LAS u32* sm, char* ws, int b, int lane, u32 call) {
  if (lane == 0) {
    u32 tk, m4;
    asm volatile("ds_max_u32 %2, %3\n\tds_add_rtn_u32 %0, %2, %4 offset:4\n\tds_read_b32 %1, %2\n\ts_waitcnt lgkmcnt(0)"
                 : "=&v"(tk), "=&v"(m4) : "v"((u32)(size_t)sm), "v"(wmx), "v"(1u) : "memory");
    if (tk == 3) st8_sys(ws + WS_LMAX + b * 128, u32x2{m4, call});
  }
}
__device__ __forceinline__ u32 wait_gmax(char* ws, int b, int lane, u32 call, u64 t0, bool bad) {
  const char* pg = ws + WS_GMAX + (size_t)(((call & 1) * REP + b % REP) * (NR * NAGG) + lane) * 128;
  u32 gm;
  while (true) {
    const u32x2 r = ld8_sys(pg);
    gm = r[0];
    if (__all(r[1] >= call)) break;
    if (now() - t0 > DEADLINE) { bad = true; break; }
    __builtin_amdgcn_s_sleep(1);
  }
  if (bad && lane == 0) st4_sys(ws + WS_TMO, call);
  return bad ? 0xffffffffu : wave_max(gm);
}
__device__ __forceinline__ u32x4 quant16(const f32x4* y, float inv, u32 tag) {
  u32x4 d;
#pragma unroll
  for (int n = 0; n < 4; n++) {
    u32 q[4];
#pragma unroll
    for (int e = 0; e < 4; e++) q[e] = __builtin_bit_cast(u32, fminf(__builtin_fmaf(y[n][e], inv, 8388608.f), 8388735.f));
    d[n] = (__builtin_amdgcn_perm(__builtin_amdgcn_perm(q[3], q[2], 0x0c0c0400u), __builtin_amdgcn_perm(q[1], q[0], 0x0c0c0400u), 0x05040100u) & 0x7f7f7f7fu) | tag;
  }
  return d;
}
__device__ __forceinline__ u32 spread4(u32 nb) { return __builtin_amdgcn_ubfe(nb, 0, 4) * 0x204081u & 0x01010101u; }
__device__ __forceinline__ u32 wave_scan(u32 v) {
  v += (u32)__builtin_amdgcn_update_dpp(0, (int)v, 0x111, 0xf, 0xf, false);
  v += (u32)__builtin_amdgcn_update_dpp(0, (int)v, 0x112, 0xf, 0xf, false);
  v += (u32)__builtin_amdgcn_update_dpp(0, (int)v, 0x114, 0xf, 0xf, false);
  v += (u32)__builtin_amdgcn_update_dpp(0, (int)v, 0x118, 0xf, 0xf, false);
  v += (u32)__builtin_amdgcn_update_dpp(0, (int)v, 0x142, 0xa, 0xf, false);
  v += (u32)__builtin_amdgcn_update_dpp(0, (int)v, 0x143, 0xc, 0xf, false);
  return v;
}

template <int STR> __device__ __forceinline__ void ld64_sys(u32x4* r, const char* base, u32 off) {
  asm volatile("global_load_dwordx4 %0, %4, %5 sc0 sc1\n\tglobal_load_dwordx4 %1, %4, %6 sc0 sc1\n\t"
               "global_load_dwordx4 %2, %4, %7 sc0 sc1\n\tglobal_load_dwordx4 %3, %4, %8 sc0 sc1\n\ts_waitcnt vmcnt(0)"
               : "=&v"(r[0]), "=&v"(r[1]), "=&v"(r[2]), "=&v"(r[3])
               : "v"(off), "s"(base), "s"(base + STR), "s"(base + 2 * STR), "s"(base + 3 * STR) : "memory");
}
__device__ __forceinline__ u32x2 lds_ld8(LAS u32* p) { u32x2 v; asm volatile("ds_read_b64 %0, %1\n\ts_waitcnt lgkmcnt(0)" : "=&v"(v) : "v"((u32)(size_t)p) : "memory"); return v; }

__global__ __launch_bounds__(512, 2) void gemm_a2a(const char* __restrict__ x, const char* __restrict__ w,
                                                   char* const* out_all, char* const* ws_all, char* ws, char* out, int rank) {
  extern __shared__ __attribute__((aligned(16))) char smem[];
  const int b = blockIdx.x, lane = threadIdx.x & 63;
  const int wave = __builtin_amdgcn_readfirstlane(threadIdx.x >> 6);
  const int xcd = b & 7, idx = b >> 3;
  const int ks = xcd & 1, mt = idx & 3, ntl = idx >> 2, nt = (xcd >> 1) * 8 + ntl;
  const int g = lane >> 4, li = lane & 15;
  const int dest = nt >> 2, item = rank * 32 + (nt & 3) * 8 + mt * 2 + ks;
  char* const mine = out + (size_t)((b >> 5) * MS + (b & 7) * 64) * NCOL + ((b >> 3) & 3) * 128;
  LAS u32* sm = (LAS u32*)(smem + SMALL);
  u32* tmo = (u32*)(ws + WS_TMO);
  auto get_gmax = [&](u32 call, u64 t0, bool bad, auto&& idle) {
    u32 gm;
    while (true) {
      const u32x2 v = lds_ld8(sm + 6);
      gm = __builtin_amdgcn_readfirstlane(v[0]);
      if (__builtin_amdgcn_readfirstlane(v[1]) == call) break;
      if (now() - t0 > 2 * DEADLINE) { bad = true; break; }
      idle();
    }
    if (bad && lane == 0) st4_sys(tmo, call);
    return bad ? 0xffffffffu : gm;
  };
  auto recv = [&](int h, u32 call, u64 t0) {
    bool bad = get_gmax(call, t0, false, [&]() { __builtin_amdgcn_s_sleep(2); }) == 0xffffffffu;
    t0 = now();
    const u32 tag = spread4(call) << 7;
    const char* pm = ws + WS_MSK + (size_t)(((call & 1) * 256 + b) * 4 + h) * 512 + lane * 8;
    u32 km = 0;
    while (!bad) {
      const u32x2 r = ld8_sys(pm);
      km = ~r[0];
      if (__all(r[1] >= call)) break;
      if (now() - t0 > DEADLINE) bad = true;
      __builtin_amdgcn_s_sleep(2);
    }
    const u32 cnt = __builtin_popcount(km), inc = wave_scan(cnt);
    const int len = (__builtin_amdgcn_readlane(inc, 63) + 15) >> 4;
    const u32 lb = (u32)(size_t)smem + WRING + h * 4096;
    u32 ad[8], sel[8], keep[8];
#pragma unroll
    for (int k = 0; k < 8; k++) {
      const u32 sh = 4 * (k ^ (g & 1));
      const u32 o = inc - cnt + __builtin_popcount(km & ((1u << sh) - 1)), sp = spread4(km >> sh);
      ad[k] = lb + (o & ~3u);
      sel[k] = (sp << 8) + (sp << 16) + (sp << 24) + (o & 3) * 0x01010101u;
      keep[k] = sp * 127;
      asm volatile("" : "+v"(ad[k]), "+v"(sel[k]), "+v"(keep[k]));
    }
    if (CFG_PDLY) { while (now() - t0 < CFG_PDLY) __builtin_amdgcn_s_sleep(1); }
    const char* stg = ws + WS_STG + (size_t)(b * 4 + h) * 2048;
    const u32 o0 = max(min(lane, len - 1), 0) * 16, o1 = max(min(lane + 64, len - 1), 0) * 16;
    u32x4 r0, r1;
    if (CFG_P1) {
      while (!bad && len > 0) {
        const u32 f = __builtin_amdgcn_readfirstlane(ld4_sys(stg + (CFG_P1 == 1 ? (len - 1) * 16 : 0)));
        if (((f ^ tag) & 0x80808080u) == 0) break;
        if (now() - t0 > DEADLINE) bad = true;
        __builtin_amdgcn_s_sleep(CFG_PSLP);
      }
    }
    while (!bad && len > 0) {
      asm volatile("global_load_dwordx4 %0, %2, %4 sc0 sc1\n\tglobal_load_dwordx4 %1, %3, %4 sc0 sc1\n\ts_waitcnt vmcnt(0)"
                   : "=&v"(r0), "=&v"(r1) : "v"(o0), "v"(o1), "s"(stg) : "memory");
      const u32x4 e = (r0 ^ tag) | (r1 ^ tag);
      if (__all(((e[0] | e[1] | e[2] | e[3]) & 0x80808080u) == 0)) break;
      if (now() - t0 > DEADLINE) bad = true;
      __builtin_amdgcn_s_sleep(CFG_PSLP);
    }
    *(LAS u32x4*)(size_t)(lb + o0) = r0;
    *(LAS u32x4*)(size_t)(lb + o1) = r1;
    asm volatile("" ::: "memory");
    u32 d[8];
#pragma unroll
    for (int k = 0; k < 8; k++) {
      const LAS u32* q = (const LAS u32*)(size_t)ad[k];
      d[k] = __builtin_amdgcn_perm(q[1], q[0], sel[k]) & keep[k];
    }
    if (bad) {
#pragma unroll
      for (int k = 0; k < 8; k++) { d[k] = 0x80808080u; asm volatile("" : "+v"(d[k])); }
    }
    const char* op = mine + (h >> 1) * 32 * NCOL + (h & 1) * 64;
#pragma unroll
    for (int m = 0; m < 2; m++) {
      const u32x4 v = u32x4{d[4 * m], d[4 * m + 1], d[4 * m + 2], d[4 * m + 3]};
      if (CFG_OST) st16_so<0>(op + m * 16 * NCOL, li * NCOL + g * 16, v);
      else *(u32x4*)(op + m * 16 * NCOL + li * NCOL + g * 16) = v;
    }
    if (CFG_RF == 0) {
      if (lane >= len) st16_so<0>(stg, lane * 16, u32x4{tag, tag, tag, tag});
      if (lane + 64 >= len) st16_so<1024>(stg, lane * 16, u32x4{tag, tag, tag, tag});
    } else {
      const int cell = (lane >> 3) * 64 + (call & 7) * 8 + (lane & 7);
      if (lane < 16 && cell >= len) st16_so<0>(stg, cell * 16, u32x4{tag, tag, tag, tag});
    }
    if (bad && lane == 0) st4_sys(tmo, call);
  };
  if (wave >= 6) {
#pragma unroll 1
    for (int nb = 0; nb < NSTEP; nb++) __builtin_amdgcn_s_barrier();
    const u32 call = __builtin_amdgcn_readfirstlane(sm[2]);
    const u64 t0 = now();
    if (wave == 6) {
      const u32 gm = wait_gmax(ws, b, lane, call, t0, false);
      if (lane == 0) *(LAS u32x2*)(sm + 6) = u32x2{gm, call};
    }
    recv(wave - 4, call, t0);
    return;
  }
  if (wave >= 4) {
    const int lw = wave - 4;
    const int kx = (mt ^ ntl) & CFG_XOR;
    auto rx = __builtin_amdgcn_make_buffer_rsrc((void*)x, 0, MS * KD * 2, 0x00020000);
    auto rw = __builtin_amdgcn_make_buffer_rsrc((void*)w, 0, KD * ND * 2, 0x00020000);
    int vx[2], vw[2];
#pragma unroll
    for (int h = 0; h < 2; h++) {
      vx[h] = (lane >> 3) * 8192 + (((lane & 7) ^ ((((lane >> 3) + 8 * h) >> 1) & 7)) << 4);
      vw[h] = g * 8192 + ((li ^ g ^ (8 * h)) << 4);
    }
    const int sx0 = (mt * 128 + (lw ^ ks) * 64) * 8192 + ks * KD;
    const int sw0 = (ks * (KD / 2) + lw * 32) * 8192 + nt * 256;
    auto issue = [&](int s, int slot) {
      const int sp = s ^ kx;
      LAS char* l = (LAS char*)(smem + slot * BLK + lw * 8192);
#pragma unroll
      for (int i = 0; i < 8; i++) __builtin_amdgcn_raw_ptr_buffer_load_lds(rx, l + XRING + i * 1024, 16, vx[i & 1], sx0 + i * 8 * 8192 + sp * 128, 0, 0);
#pragma unroll
      for (int i = 0; i < 8; i++) __builtin_amdgcn_raw_ptr_buffer_load_lds(rw, l + WRING + i * 1024, 16, vw[(i >> 1) & 1], sw0 + (sp * 64 + i * 4) * 8192, 0, 0);
    };
    issue(0, 0); issue(1, 1);
    wait_vm<16>();
    __builtin_amdgcn_s_barrier();
    issue(2, 2); issue(3, 3);
#pragma unroll 1
    for (int j = 1; j <= NSTEP - NS; j++) {
      wait_vm<32>();
      __builtin_amdgcn_s_barrier();
      issue(j - 1 + NS, (j - 1) & 3);
    }
    wait_vm<32>(); __builtin_amdgcn_s_barrier();
    wait_vm<16>(); __builtin_amdgcn_s_barrier();
    wait_vm<0>(); __builtin_amdgcn_s_barrier();
    const u32 call = __builtin_amdgcn_readfirstlane(sm[2]);
    const u64 t0 = now();
    bool bad = false;
    if (lw == 1 && b < NAGG) {
      char* pw = ws_all[lane & 7];
      u32 gm = 0;
      const char* pg = lane < 32 ? ws + WS_LMAX + (size_t)(lane * NAGG + b) * 128 : (const char*)tmo - 4;
      while (true) {
        const u32x2 r = ld8_sys(pg);
        gm = lane < 32 ? r[0] : 0;
        if (__all(lane >= 32 || r[1] >= call)) break;
        if (__builtin_amdgcn_readlane(r[1], 63) == call || now() - t0 > DEADLINE) { bad = true; break; }
      }
      gm = bad ? 0xffffffffu : wave_max(gm);
      if (lane < NR * REP) st8_sys(pw + WS_GMAX + (size_t)(((call & 1) * REP + (lane >> 3)) * (NR * NAGG) + rank * NAGG + b) * 128, u32x2{gm, call});
    }
    recv(lw, call, t0);
    return;
  }
  const int wm = wave >> 1, wn = wave & 1;
  u32* ctr = (u32*)(ws + WS_CTR + b * 128);
  const u32 call = __builtin_amdgcn_readfirstlane(ld4_sys(ctr)) + 1;
  char* const pw = uni(ws_all[dest & 7]);
  if (wave == 0 && lane < 8) sm[lane] = lane == 2 ? call : 0;
  f32x4 acc[4][4];
#pragma unroll
  for (int i = 0; i < 16; i++) acc[i >> 2][i & 3] = f32x4{0.f, 0.f, 0.f, 0.f};
  u32 ax[2], aw[4];
#pragma unroll
  for (int kk = 0; kk < 2; kk++) ax[kk] = (u32)(size_t)smem + XRING + (32 * wm + li) * 128 + (((4 * kk + g) ^ ((li >> 1) & 7)) << 4);
  {
    const int q = li >> 2, p = li & 3;
#pragma unroll
    for (int n = 0; n < 4; n++)
      aw[n] = (u32)(size_t)smem + WRING + (8 * g + q) * 256 + (((8 * wn + 2 * p + (n >> 1)) ^ q ^ ((g & 1) << 3)) << 4) + (((n ^ p) & 1) << 3);
  }
#pragma unroll
  for (int n = 0; n < 4; n++) asm volatile("" : "+v"(aw[n]));
  bf16x8 fx[2][4], fw[2][4];
  auto rdw = [&](int buf, int n, u32 a) {
    const bf16x4 lo = __builtin_amdgcn_ds_read_tr16_b64_v4bf16((LAS bf16x4*)(size_t)a);
    const bf16x4 hi = __builtin_amdgcn_ds_read_tr16_b64_v4bf16((LAS bf16x4*)(size_t)(a + 1024));
    fw[buf][n] = __builtin_shufflevector(lo, hi, 0, 1, 2, 3, 4, 5, 6, 7);
  };
  auto rd = [&](int buf, int slot, int kk) {
    rdw(buf, 0, aw[0] + slot * BLK + kk * 8192);
#pragma unroll
    for (int m = 0; m < 4; m++) fx[buf][m] = *(LAS bf16x8*)(size_t)(ax[kk] + slot * BLK + ((m & 1) * 16 + (m >> 1) * 64) * 128);
#pragma unroll
    for (int n = 1; n < 4; n++) rdw(buf, n, aw[n] + slot * BLK + kk * 8192);
  };
  auto mm = [&](int buf) {
#pragma unroll
    for (int n = 0; n < 4; n++)
#pragma unroll
      for (int i = 0; i < 4; i++) {
        const int m = (n & 1) ? 3 - i : i;
        acc[m][n] = __builtin_amdgcn_mfma_f32_16x16x32_bf16(fw[buf][n], fx[buf][m], acc[m][n], 0, 0, 0);
      }
  };
  auto mix = [&]() {
#pragma unroll
    for (int i = 0; i < 12; i++) { __builtin_amdgcn_sched_group_barrier(0x008, 1, 0); __builtin_amdgcn_sched_group_barrier(0x100, 1, 0); }
    __builtin_amdgcn_sched_group_barrier(0x008, 4, 0);
    __builtin_amdgcn_sched_barrier(0);
  };
  auto sync = [&]() {
    __builtin_amdgcn_s_waitcnt(0xc07f);
    asm volatile("" ::: "memory");
    __builtin_amdgcn_s_barrier();
    __builtin_amdgcn_sched_barrier(0);
  };
  auto step = [&](int u, bool last) {
    __builtin_amdgcn_s_waitcnt(0xc67f);
    __builtin_amdgcn_sched_barrier(0);
    rd(1, u, 1); mm(0); mix();
    if (!last) { sync(); rd(0, (u + 1) % NS, 0); mm(1); mix(); }
    else mm(1);
  };
  sync();
  rd(0, 0, 0);
  __builtin_amdgcn_sched_barrier(0);
#pragma unroll 1
  for (int j0 = 0; j0 < NSTEP - NS; j0 += NS) {
#pragma unroll
    for (int u = 0; u < NS; u++) step(u, false);
  }
#pragma unroll
  for (int u = 0; u < NS; u++) step(u, u == NS - 1);
  bool bad = false;
  const u64 t0 = now();
  {
    const char* dst = ws + WS_EX + (size_t)((b ^ 1) * 4 + wave) * 4096;
    auto pk = [&](int m, int n) { return pk8(acc[m][n], acc[m][n + 1]); };
    st16_so<0>(dst, lane * 16, pk(2, 0));
    st16_so<1024>(dst, lane * 16, pk(2, 2));
    st16_so<2048>(dst, lane * 16, pk(3, 0));
    st16_so<3072>(dst, lane * 16, pk(3, 2));
    const char* src = ws + WS_EX + (size_t)(b * 4 + wave) * 4096;
    u32x4 r[4];
    {
      wait_vm<0>();
      if (lane == 0) st4_sys(ws + WS_XF + ((b ^ 1) * 4 + wave) * 128, call);
      if (wave == 0 && lane == 63) st4_sys(ctr, call);
      const char* pf = lane == 0 ? ws + WS_XF + (b * 4 + wave) * 128 : (const char*)tmo;
      while (true) {
        const u32 f = ld4_sys(pf);
        if (__builtin_amdgcn_readfirstlane(f) >= call) break;
        if (__builtin_amdgcn_readlane(f, 63) == call || now() - t0 > DEADLINE) { bad = true; break; }
        __builtin_amdgcn_s_sleep(1);
      }
      ld64_sys<1024>(r, src, lane * 16);
    }
#pragma unroll
    for (int i = 0; i < 4; i++)
#pragma unroll
      for (int e = 0; e < 4; e++) {
        acc[i >> 1][2 * (i & 1) + (e >> 1)][2 * (e & 1)] += lo16(r[i][e]);
        acc[i >> 1][2 * (i & 1) + (e >> 1)][2 * (e & 1) + 1] += hi16(r[i][e]);
      }
  }
  float mx = 0.f;
#pragma unroll
  for (int i = 0; i < 8; i++)
#pragma unroll
    for (int e = 0; e < 4; e++) mx = fmaxf(mx, acc[i >> 2][i & 3][e]);
  wg_max(bad ? 0xffffffffu : wave_max(__builtin_bit_cast(u32, mx)), sm, ws, b, lane, call);
  __builtin_amdgcn_sched_barrier(0);
  u32 neg = 0;
#pragma unroll
  for (int j = 31; j >= 0; j--) {
    const float v = acc[j >> 4][(j >> 2) & 3][j & 3];
    neg = __builtin_amdgcn_alignbit(neg, __builtin_bit_cast(u32, v), 31);
  }
  if (CFG_EXP == 0 || call <= 2) st8_sys(pw + WS_MSK + (size_t)(((call & 1) * 256 + item) * 4 + wave) * 512 + lane * 8, u32x2{neg, CFG_EXP ? ~0u : call});
  if (CFG_EXP == 2) st4_sys(pw + WS_END + (size_t)(((call & 1) * 256 + item) * 4 + wave) * 512 + lane * 4, neg);
  if (CFG_EXP == 3) { st4_sys(pw + WS_END + (size_t)(((call & 1) * 256 + item) * 4 + wave) * 512 + lane * 4, neg); if (lane < 16) st4_sys(pw + WS_END + (size_t)(((call & 1) * 256 + item) * 4 + wave) * 512 + 256 + lane * 4, neg); }
  if (CFG_EXP == 4 && lane < 16) st16_sys(pw + WS_END + (size_t)(((call & 1) * 256 + item) * 4 + wave) * 512 + lane * 16, u32x4{neg, neg * 3, neg * 5, neg * 7});
  const u32 cnt = __builtin_popcount(~neg), inc = wave_scan(cnt);
  const int len = (__builtin_amdgcn_readlane(inc, 63) + 15) >> 4;
  const u32 zb = (u32)(size_t)smem + XRING + 512 + wave * 8704;
  {
    u32 p = zb + (inc - cnt) * 4;
    const u32 skip = zb - 512 + lane * 4;
#pragma unroll
    for (int j = 0; j < 32; j++) {
      const float v = acc[j >> 4][(j >> 2) & 3][j & 3];
      u32 t = (u32)(__builtin_bit_cast(int, v) >> 31);
      asm("" : "+v"(t));
      *(LAS float*)(size_t)(((t & skip) | (~t & p)) + 4 * j) = v;
      p += t << 2;
      asm("" : "+v"(p));
    }
  }
  asm volatile("" ::: "memory");
  f32x4 y[2][4];
#pragma unroll
  for (int i = 0; i < 8; i++) y[i >> 2][i & 3] = *(LAS f32x4*)(size_t)(zb + (i >> 2) * 4096 + lane * 64 + (i & 3) * 16);
  const u32 gm = get_gmax(call, t0, bad, [&]() {});
  if (gm != 0xffffffffu) {
    const float scale = __builtin_bit_cast(float, gm) / 127.0f;
    const float inv = scale > 0.f ? 1.0f / scale : 0.f;
    const u32 tag = spread4(call) << 7;
    const char* dst = pw + WS_STG + (size_t)(item * 4 + wave) * 2048;
    if (lane < len) st16_so<0>(dst, lane * 16, quant16(y[0], inv, tag));
    if (lane + 64 < len) st16_so<1024>(dst, lane * 16, quant16(y[1], inv, tag));
  }
}

static bool launch_setup() {
  hipError_t e = hipFuncSetAttribute((const void*)gemm_a2a, hipFuncAttributeMaxDynamicSharedMemorySize, LDS_BYTES);
  if (e != hipSuccess) { fprintf(stderr, "HIP error %s\n", hipGetErrorString(e)); return false; }
  return true;
}
static void launch_impl(int rank, hipStream_t stream, const void* x, const void* w, char* const* out_dev, char* const* ws_dev,
                        char* const* out_host, char* const* ws_host) {
  gemm_a2a<<<256, 512, LDS_BYTES, stream>>>((const char*)x, (const char*)w, out_dev, ws_dev, ws_host[rank], out_host[rank], rank);
}
extern "C" void kernel_launch(int rank, int nranks, ncclComm_t comm, hipStream_t stream, void* const* d_in, const int* in_sizes, int n_in,
                              void* d_out, int out_size, const int* in_dtypes, int out_dtype, void* const* d_out_all,
                              void* const* d_ws_all, size_t ws_bytes) {
  if (nranks != NR || (unsigned)rank >= (unsigned)NR || n_in < 2 || in_sizes[0] != MS * KD || in_sizes[1] != KD * ND ||
      out_size != NR * MS * NCOL || ws_bytes < WS_END) {
    fprintf(stderr, "unexpected problem shape\n");
    return;
  }
  if (!launch_setup()) return;
  launch_impl(rank, stream, d_in[0], d_in[1], (char* const*)d_out_all, (char* const*)d_ws_all, (char* const*)d_out_all, (char* const*)d_ws_all);
}
```

```cpp
#define CFG_NE 12
#define CFG_PDLY 185
#define CFG_EDLY 220
#define CFG_EX 16
#include <hip/hip_runtime.h>
#include <rccl/rccl.h>
#include <cstdio>

typedef unsigned u32;
typedef unsigned long long u64;
typedef u32 u32x4 __attribute__((ext_vector_type(4)));
typedef u32 u32x2 __attribute__((ext_vector_type(2)));
typedef float f32x4 __attribute__((ext_vector_type(4)));
typedef __bf16 bf16x4 __attribute__((ext_vector_type(4)));
typedef __bf16 bf16x8 __attribute__((ext_vector_type(8)));
typedef __bf16 bf16x2 __attribute__((ext_vector_type(2)));
#define LAS __attribute__((address_space(3)))

#define CFG_PSLP 2
#define CFG_PH 0
#define CFG_PX 0
#define CFG_KSF 0
#define CFG_REP 8
#define CFG_EOFF 0
#define CFG_AM 1
#define CFG_MADD 1
#define CFG_M4 1
constexpr int NAGG = 8, REP = CFG_REP;
constexpr int NR = 8, MS = 512, KD = 4096, ND = 4096, NCOL = 512;
constexpr int NSTEP = KD / 2 / 64, NS = 4, BLK = 16384;
constexpr int XRING = 0, WRING = NS * BLK, SMALL = 2 * NS * BLK, LDS_BYTES = SMALL + 256;
constexpr u64 DEADLINE = 100000000ull;

constexpr size_t WS_CTR = 0;
constexpr size_t WS_TMO = 32768 + 4;
constexpr size_t WS_LMAX = 98304;
constexpr size_t WS_XF = 131072;
constexpr size_t WS_GMAX = 524288;
constexpr size_t WS_TS = 786432;
constexpr size_t WS_EX = 1u << 20;
constexpr size_t WS_STG = WS_EX + (size_t)256 * 16384;
#define CFG_LAY 0
constexpr u32 PLANE = 1u << 20, SSTR = CFG_LAY == 0 ? 1024 : CFG_LAY == 1 ? 2048 : 4096;
__device__ __forceinline__ u32 cell_off(u32 c) { return CFG_LAY == 0 ? (c >> 6) * PLANE + (c & 63) * 16 : CFG_LAY == 1 ? (c >> 7) * 2 * PLANE + (c & 127) * 16 : c * 16; }
constexpr size_t WS_MSK = WS_STG + (size_t)2 * 1024 * 4096;
constexpr size_t WS_END = WS_MSK + (size_t)2 * 1024 * 512;

template <int N> __device__ __forceinline__ void wait_vm() { asm volatile("s_waitcnt vmcnt(%0)" ::"i"(N) : "memory"); }
__device__ __forceinline__ void st16_sys(void* p, u32x4 v) { asm volatile("global_store_dwordx4 %0, %1, off sc0 sc1\n\ts_nop 1" ::"v"(p), "v"(v) : "memory"); }
template <int O> __device__ __forceinline__ void st16_so(const char* base, u32 off, u32x4 v) {
  asm volatile("global_store_dwordx4 %0, %1, %2 offset:%3 sc0 sc1\n\ts_nop 1" ::"v"(off), "v"(v), "s"(base), "i"(O) : "memory");
}
__device__ __forceinline__ void st8_sys(void* p, u32x2 v) { asm volatile("global_store_dwordx2 %0, %1, off sc0 sc1\n\ts_nop 1" ::"v"(p), "v"(v) : "memory"); }
__device__ __forceinline__ void st4_sys(void* p, u32 v) { asm volatile("global_store_dword %0, %1, off sc0 sc1\n\ts_nop 1" ::"v"(p), "v"(v) : "memory"); }
__device__ __forceinline__ u32 ld4_sys(const void* p) { u32 v; asm volatile("global_load_dword %0, %1, off sc0 sc1\n\ts_waitcnt vmcnt(0)" : "=&v"(v) : "v"(p) : "memory"); return v; }
__device__ __forceinline__ u32x2 ld8_sys(const void* p) { u32x2 v; asm volatile("global_load_dwordx2 %0, %1, off sc0 sc1\n\ts_waitcnt vmcnt(0)" : "=&v"(v) : "v"(p) : "memory"); return v; }
__device__ __forceinline__ u64 now() { return __builtin_amdgcn_s_memrealtime(); }
__device__ __forceinline__ u32 vm_peek() { u32 v; asm volatile("s_getreg_b32 %0, hwreg(HW_REG_IB_STS, 0, 4)" : "=s"(v) :: "memory"); return v; }
__device__ __forceinline__ char* uni(char* p) {
  const u64 v = (u64)p;
  return (char*)(((u64)(u32)__builtin_amdgcn_readfirstlane((u32)(v >> 32)) << 32) | (u32)__builtin_amdgcn_readfirstlane((u32)v));
}
__device__ __forceinline__ u32 clk() { return (u32)__builtin_amdgcn_s_memtime(); }
__device__ __forceinline__ u32 pk2(float a, float b) { return __builtin_bit_cast(u32, bf16x2{(__bf16)a, (__bf16)b}); }
__device__ __forceinline__ u32x4 pk8(f32x4 a, f32x4 c) { return u32x4{pk2(a[0], a[1]), pk2(a[2], a[3]), pk2(c[0], c[1]), pk2(c[2], c[3])}; }
__device__ __forceinline__ float lo16(u32 v) { return __builtin_bit_cast(float, v << 16); }
__device__ __forceinline__ float hi16(u32 v) { return __builtin_bit_cast(float, v & 0xffff0000u); }

__device__ __forceinline__ u32 wave_max(u32 v) {
  v = max(v, (u32)__builtin_amdgcn_update_dpp(0, (int)v, 0xb1, 0xf, 0xf, true));
  v = max(v, (u32)__builtin_amdgcn_update_dpp(0, (int)v, 0x4e, 0xf, 0xf, true));
  v = max(v, (u32)__builtin_amdgcn_update_dpp(0, (int)v, 0x141, 0xf, 0xf, true));
  v = max(v, (u32)__builtin_amdgcn_update_dpp(0, (int)v, 0x140, 0xf, 0xf, true));
  return max(max((u32)__builtin_amdgcn_readlane(v, 0), (u32)__builtin_amdgcn_readlane(v, 16)),
             max((u32)__builtin_amdgcn_readlane(v, 32), (u32)__builtin_amdgcn_readlane(v, 48)));
}
__device__ __forceinline__ void wg_max(u32 wmx, LAS u32* sm, char* ws, int b, int lane, u32 call) {
  if (lane == 0) {
    u32 tk, m4;
    asm volatile("ds_max_u32 %2, %3\n\tds_add_rtn_u32 %0, %2, %4 offset:4\n\tds_read_b32 %1, %2\n\ts_waitcnt lgkmcnt(0)"
                 : "=&v"(tk), "=&v"(m4) : "v"((u32)(size_t)sm), "v"(wmx), "v"(1u) : "memory");
    if (tk == 3) st8_sys(ws + WS_LMAX + b * 128, u32x2{m4, call});
  }
}
__device__ __forceinline__ u32 wait_gmax(char* ws, int b, int lane, u32 call, u64 t0, bool bad) {
  const char* pg = ws + WS_GMAX + (size_t)(((call & 1) * REP + b % REP) * (NR * NAGG) + lane) * 128;
  u32 gm;
  while (true) {
    const u32x2 r = ld8_sys(pg);
    gm = r[0];
    if (__all(r[1] >= call)) break;
    if (now() - t0 > DEADLINE) { bad = true; break; }
    __builtin_amdgcn_s_sleep(1);
  }
  if (bad && lane == 0) st4_sys(ws + WS_TMO, call);
  return bad ? 0xffffffffu : wave_max(gm);
}
__device__ __forceinline__ float inv_scale(u32 gm) {
  const float scale = __builtin_bit_cast(float, gm) / 127.0f;
  return scale > 0.f ? 1.0f / scale : 0.f;
}
__device__ __forceinline__ u32 quant4(f32x4 y, float inv) {
  u32 q[4];
#pragma unroll
  for (int e = 0; e < 4; e++) q[e] = __builtin_bit_cast(u32, fminf(__builtin_fmaf(y[e], inv, 8388608.f), 8388735.f));
  return __builtin_amdgcn_perm(__builtin_amdgcn_perm(q[3], q[2], 0x0c0c0400u), __builtin_amdgcn_perm(q[1], q[0], 0x0c0c0400u), 0x05040100u);
}
__device__ __forceinline__ u32 spread4(u32 nb) { return __builtin_amdgcn_ubfe(nb, 0, 4) * 0x204081u & 0x01010101u; }
__device__ __forceinline__ u32x2 tags(u32 use, bool wide) {
  const u32 t = spread4(use) << 7;
  return wide ? u32x2{(use & 1) << 15 | (use & 2) << 30, (use & 4) << 13 | (use & 8) << 28} : u32x2{t, t};
}
__device__ __forceinline__ u32 wave_scan(u32 v) {
  v += (u32)__builtin_amdgcn_update_dpp(0, (int)v, 0x111, 0xf, 0xf, false);
  v += (u32)__builtin_amdgcn_update_dpp(0, (int)v, 0x112, 0xf, 0xf, false);
  v += (u32)__builtin_amdgcn_update_dpp(0, (int)v, 0x114, 0xf, 0xf, false);
  v += (u32)__builtin_amdgcn_update_dpp(0, (int)v, 0x118, 0xf, 0xf, false);
  v += (u32)__builtin_amdgcn_update_dpp(0, (int)v, 0x142, 0xa, 0xf, false);
  v += (u32)__builtin_amdgcn_update_dpp(0, (int)v, 0x143, 0xc, 0xf, false);
  return v;
}

template <int STR> __device__ __forceinline__ void ld64_sys(u32x4* r, const char* base, u32 off) {
  asm volatile("global_load_dwordx4 %0, %4, %5 sc0 sc1\n\tglobal_load_dwordx4 %1, %4, %6 sc0 sc1\n\t"
               "global_load_dwordx4 %2, %4, %7 sc0 sc1\n\tglobal_load_dwordx4 %3, %4, %8 sc0 sc1\n\ts_waitcnt vmcnt(0)"
               : "=&v"(r[0]), "=&v"(r[1]), "=&v"(r[2]), "=&v"(r[3])
               : "v"(off), "s"(base), "s"(base + STR), "s"(base + 2 * STR), "s"(base + 3 * STR) : "memory");
}
__device__ __forceinline__ u32x2 lds_ld8(LAS u32* p) { u32x2 v; asm volatile("ds_read_b64 %0, %1\n\ts_waitcnt lgkmcnt(0)" : "=&v"(v) : "v"((u32)(size_t)p) : "memory"); return v; }

__global__ __launch_bounds__(512, 2) void gemm_a2a(const char* __restrict__ x, const char* __restrict__ w,
                                                   char* const* out_all, char* const* ws_all, char* ws, char* out, int rank) {
  extern __shared__ __attribute__((aligned(16))) char smem[];
  const int b = blockIdx.x, lane = threadIdx.x & 63;
  const int wave = __builtin_amdgcn_readfirstlane(threadIdx.x >> 6);
  const int xcd = b & 7, idx = b >> 3;
  const int id = idx ^ CFG_EX;
  const int ks = (xcd ^ CFG_KSF) & 1, mt = (id >> 1) & 3, ntl = (id & 1) * 4 + (id >> 3), nt = (xcd >> 1) * 8 + ntl;
  const int g = lane >> 4, li = lane & 15;
  const int dest = nt >> 2, item = rank * 32 + (id & ~1) + ks, mitem = xcd * 32 + id;
  const bool early = id < CFG_NE;
  char* const mine = out + (size_t)(xcd * MS + (id & 7) * 64) * NCOL + (id >> 3) * 128;
  LAS u32* sm = (LAS u32*)(smem + SMALL);
  u32* tmo = (u32*)(ws + WS_TMO);
  auto get_gmax = [&](u32 call, u64 t0, bool bad, auto&& idle) {
    u32 gm;
    while (true) {
      const u32x2 v = lds_ld8(sm + 6);
      gm = __builtin_amdgcn_readfirstlane(v[0]);
      if (__builtin_amdgcn_readfirstlane(v[1]) == call) break;
      if (now() - t0 > 2 * DEADLINE) { bad = true; break; }
      idle();
    }
    if (bad && lane == 0) st4_sys(tmo, call);
    return bad ? 0xffffffffu : gm;
  };
  auto recv = [&](int h, u32 call, u64 t0) {
    const u32 gm = get_gmax(call, t0, false, [&]() { __builtin_amdgcn_s_sleep(2); });
    bool bad = gm == 0xffffffffu;
    t0 = now();
    const char* pm = ws + WS_MSK + (size_t)(((call & 1) * 256 + mitem) * 4 + h) * 512 + (CFG_M4 ? (lane & 31) * 16 + (lane >> 5) * 8 : lane * 8);
    u32 km = 0;
    while (!bad) {
      const u32x2 r = ld8_sys(pm);
      km = ~r[0];
      if (__all(r[1] >= call)) break;
      if (now() - t0 > DEADLINE) bad = true;
      __builtin_amdgcn_s_sleep(2);
    }
    const u32 cnt = __builtin_popcount(km), inc = wave_scan(cnt);
    const int len = early ? (__builtin_amdgcn_readlane(inc, 63) + 7) >> 3 : (__builtin_amdgcn_readlane(inc, 63) + 15) >> 4;
    const u32 lb = (u32)(size_t)smem + WRING + h * 4096;
    u32 ad[8], sel[8], keep[8];
#pragma unroll
    for (int k = 0; k < 8; k++) {
      const u32 sh = 4 * (k ^ (g & 1));
      const u32 o = inc - cnt + __builtin_popcount(km & ((1u << sh) - 1)), sp = spread4(km >> sh);
      ad[k] = lb + (o & ~3u);
      sel[k] = (sp << 8) + (sp << 16) + (sp << 24) + (o & 3) * 0x01010101u;
      keep[k] = sp * 127;
      asm volatile("" : "+v"(ad[k]), "+v"(sel[k]), "+v"(keep[k]));
    }
    const char* stg = ws + WS_STG + (size_t)(call & 1) * 4 * PLANE + (mitem * 4 + h) * SSTR;
    const u32 use = (call + 1) >> 1;
    const u32x2 tg = tags(use, early);
    const u32x4 tag = u32x4{tg[0], tg[1], tg[0], tg[1]};
    auto cl = [&](int i) { return (u32)max(min(lane + 64 * i, len - 1), 0); };
    auto off = [&](int i) { return cell_off(cl(i)); };
    if (early) {
      u32x4 r[4];
      while (!bad && len > 0) {
        asm volatile("global_load_dwordx4 %0, %4, %8 sc0 sc1\n\tglobal_load_dwordx4 %1, %5, %8 sc0 sc1\n\t"
                     "global_load_dwordx4 %2, %6, %8 sc0 sc1\n\tglobal_load_dwordx4 %3, %7, %8 sc0 sc1\n\ts_waitcnt vmcnt(0)"
                     : "=&v"(r[0]), "=&v"(r[1]), "=&v"(r[2]), "=&v"(r[3]) : "v"(off(0)), "v"(off(1)), "v"(off(2)), "v"(off(3)), "s"(stg) : "memory");
        const u32x4 e = (r[0] ^ tag) | (r[1] ^ tag) | (r[2] ^ tag) | (r[3] ^ tag);
        if (__all(((e[0] | e[1] | e[2] | e[3]) & 0x80008000u) == 0)) break;
        if (now() - t0 > DEADLINE) bad = true;
        __builtin_amdgcn_s_sleep(CFG_PSLP);
      }
      const float inv = inv_scale(gm);
#pragma unroll
      for (int i = 0; i < 4; i++) {
        if (i * 64 >= len) break;
        f32x4 y[2];
#pragma unroll
        for (int e = 0; e < 4; e++) {
          const u32 v = r[i][e];
          y[e >> 1][2 * (e & 1)] = __builtin_fabsf(lo16(v));
          y[e >> 1][2 * (e & 1) + 1] = __builtin_bit_cast(float, v & 0x7fff0000u);
        }
        *(LAS u32x2*)(size_t)(lb + cl(i) * 8) = u32x2{quant4(y[0], inv), quant4(y[1], inv)};
      }
    } else {
      while (now() - t0 < CFG_PDLY) __builtin_amdgcn_s_sleep(1);
      u32x4 r0, r1;
      while (!bad && len > 0) {
        asm volatile("global_load_dwordx4 %0, %2, %4 sc0 sc1\n\tglobal_load_dwordx4 %1, %3, %4 sc0 sc1\n\ts_waitcnt vmcnt(0)"
                     : "=&v"(r0), "=&v"(r1) : "v"(off(0)), "v"(off(1)), "s"(stg) : "memory");
        const u32x4 e = (r0 ^ tag) | (r1 ^ tag);
        if (__all(((e[0] | e[1] | e[2] | e[3]) & 0x80808080u) == 0)) break;
        if (now() - t0 > DEADLINE) bad = true;
        __builtin_amdgcn_s_sleep(CFG_PSLP);
      }
      *(LAS u32x4*)(size_t)(lb + cl(0) * 16) = r0;
      *(LAS u32x4*)(size_t)(lb + cl(1) * 16) = r1;
    }
    asm volatile("" ::: "memory");
    u32 d[8];
#pragma unroll
    for (int k = 0; k < 8; k++) {
      const LAS u32* q = (const LAS u32*)(size_t)ad[k];
      d[k] = __builtin_amdgcn_perm(q[1], q[0], sel[k]) & keep[k];
    }
    if (bad) {
#pragma unroll
      for (int k = 0; k < 8; k++) { d[k] = 0x80808080u; asm volatile("" : "+v"(d[k])); }
    }
    const char* op = mine + (h >> 1) * 32 * NCOL + (h & 1) * 64;
#pragma unroll
    for (int m = 0; m < 2; m++) {
      const u32x4 v = u32x4{d[4 * m], d[4 * m + 1], d[4 * m + 2], d[4 * m + 3]};
      st16_so<0>(op + m * 16 * NCOL, li * NCOL + g * 16, v);
    }
    const int cell = (lane >> 3) * 64 + (use & 7) * 8 + (lane & 7);
    if (lane < (early ? 32 : 16) && cell >= len) st16_so<0>(stg, cell_off(cell), tag);
    if (bad && lane == 0) st4_sys(tmo, call);
  };
  if (wave >= 6) {
    {
#pragma unroll 1
      for (int nb = 0; nb < NSTEP; nb++) __builtin_amdgcn_s_barrier();
    }
    const u32 call = __builtin_amdgcn_readfirstlane(sm[2]);
    const u64 t0 = now();
    if (wave == 6) {
      const u32 gm = wait_gmax(ws, b, lane, call, t0, false);
      if (lane == 0) *(LAS u32x2*)(sm + 6) = u32x2{gm, call};
    }
    recv(wave - 4, call, t0);
    return;
  }
  if (wave >= 4) {
    const int lw = wave - 4;
    const int kx = CFG_PH ? mt ^ (ntl >> 1) : (mt ^ ntl) & 3;
    const int ph = (ntl & 1 ? 0 : CFG_PH) + CFG_PX * (xcd >> 2);
    auto rx = __builtin_amdgcn_make_buffer_rsrc((void*)x, 0, MS * KD * 2, 0x00020000);
    auto rw = __builtin_amdgcn_make_buffer_rsrc((void*)w, 0, KD * ND * 2, 0x00020000);
    int vx[2], vw[2];
#pragma unroll
    for (int h = 0; h < 2; h++) {
      vx[h] = (lane >> 3) * 8192 + (((lane & 7) ^ ((((lane >> 3) + 8 * h) >> 1) & 7)) << 4);
      vw[h] = g * 8192 + ((li ^ g ^ (8 * h)) << 4);
    }
    const int sx0 = (mt * 128 + (lw ^ ks) * 64) * 8192 + ks * KD;
    const int sw0 = (ks * (KD / 2) + lw * 32) * 8192 + nt * 256;
    auto issue = [&](int s, int slot) {
      const int sp = ((s + ph) & (NSTEP - 1)) ^ kx;
      LAS char* l = (LAS char*)(smem + slot * BLK + lw * 8192);
#pragma unroll
      for (int i = 0; i < 8; i++) __builtin_amdgcn_raw_ptr_buffer_load_lds(rx, l + XRING + i * 1024, 16, vx[i & 1], sx0 + i * 8 * 8192 + sp * 128, 0, 0);
#pragma unroll
      for (int i = 0; i < 8; i++) __builtin_amdgcn_raw_ptr_buffer_load_lds(rw, l + WRING + i * 1024, 16, vw[(i >> 1) & 1], sw0 + (sp * 64 + i * 4) * 8192, 0, 0);
    };
    issue(0, 0); issue(1, 1);
    wait_vm<16>();
    __builtin_amdgcn_s_barrier();
    issue(2, 2); issue(3, 3);
#pragma unroll 1
    for (int j = 1; j <= NSTEP - NS; j++) {
      wait_vm<32>();
      __builtin_amdgcn_s_barrier();
      issue(j - 1 + NS, (j - 1) & 3);
    }
    wait_vm<32>(); __builtin_amdgcn_s_barrier();
    wait_vm<16>(); __builtin_amdgcn_s_barrier();
    wait_vm<0>(); __builtin_amdgcn_s_barrier();
    const u32 call = __builtin_amdgcn_readfirstlane(sm[2]);
    const u64 t0 = now();
    bool bad = false;
    if (lw == 1 && id == 31) {
      char* pw = ws_all[lane & 7];
      u32 gm = 0;
      const char* pg = lane < 32 ? ws + WS_LMAX + (size_t)(lane * NAGG + xcd) * 128 : (const char*)tmo - 4;
      while (true) {
        const u32x2 r = ld8_sys(pg);
        gm = lane < 32 ? r[0] : 0;
        if (__all(lane >= 32 || r[1] >= call)) break;
        if (__builtin_amdgcn_readlane(r[1], 63) == call || now() - t0 > DEADLINE) { bad = true; break; }
      }
      gm = bad ? 0xffffffffu : wave_max(gm);
      if (lane < NR * REP) st8_sys(pw + WS_GMAX + (size_t)(((call & 1) * REP + (lane >> 3)) * (NR * NAGG) + rank * NAGG + xcd) * 128, u32x2{gm, call});
      if (lane == 0) sm[3] = call;
    }
    recv(lw, call, t0);
    return;
  }
  const int wm = wave >> 1, wn = wave & 1;
  u32* ctr = (u32*)(ws + WS_CTR + b * 128);
  const u32 call = __builtin_amdgcn_readfirstlane(ld4_sys(ctr)) + 1;
  char* const pw = uni(ws_all[dest & 7]);
  const char* pmsk = pw + WS_MSK + (size_t)(((call & 1) * 256 + item) * 4 + wave) * 512;
  const char* pstg = pw + WS_STG + (size_t)(call & 1) * 4 * PLANE + (item * 4 + wave) * SSTR;
  asm volatile("" : "+s"(pmsk), "+s"(pstg));
  if (wave == 0 && lane < 8) sm[lane] = lane == 2 ? call : 0;
  f32x4 acc[4][4];
#pragma unroll
  for (int i = 0; i < 16; i++) acc[i >> 2][i & 3] = f32x4{0.f, 0.f, 0.f, 0.f};
  u32 ax[2], aw[4];
#pragma unroll
  for (int kk = 0; kk < 2; kk++) ax[kk] = (u32)(size_t)smem + XRING + (32 * wm + li) * 128 + (((4 * kk + g) ^ ((li >> 1) & 7)) << 4);
  {
    const int q = li >> 2, p = li & 3;
#pragma unroll
    for (int n = 0; n < 4; n++)
      aw[n] = (u32)(size_t)smem + WRING + (8 * g + q) * 256 + (((8 * wn + 2 * p + (n >> 1)) ^ q ^ ((g & 1) << 3)) << 4) + (((n ^ p) & 1) << 3);
  }
#pragma unroll
  for (int n = 0; n < 4; n++) asm volatile("" : "+v"(aw[n]));
  bf16x8 fx[2][4], fw[2][4];
  auto rdw = [&](int buf, int n, u32 a) {
    const bf16x4 lo = __builtin_amdgcn_ds_read_tr16_b64_v4bf16((LAS bf16x4*)(size_t)a);
    const bf16x4 hi = __builtin_amdgcn_ds_read_tr16_b64_v4bf16((LAS bf16x4*)(size_t)(a + 1024));
    fw[buf][n] = __builtin_shufflevector(lo, hi, 0, 1, 2, 3, 4, 5, 6, 7);
  };
  auto rd = [&](int buf, int slot, int kk) {
    rdw(buf, 0, aw[0] + slot * BLK + kk * 8192);
#pragma unroll
    for (int m = 0; m < 4; m++) fx[buf][m] = *(LAS bf16x8*)(size_t)(ax[kk] + slot * BLK + ((m & 1) * 16 + (m >> 1) * 64) * 128);
#pragma unroll
    for (int n = 1; n < 4; n++) rdw(buf, n, aw[n] + slot * BLK + kk * 8192);
  };
  auto mm = [&](int buf) {
#pragma unroll
    for (int n = 0; n < 4; n++)
#pragma unroll
      for (int i = 0; i < 4; i++) {
        const int m = (n & 1) ? 3 - i : i;
        acc[m][n] = __builtin_amdgcn_mfma_f32_16x16x32_bf16(fw[buf][n], fx[buf][m], acc[m][n], 0, 0, 0);
      }
  };
  auto mix = [&]() {
#pragma unroll
    for (int i = 0; i < 12; i++) { __builtin_amdgcn_sched_group_barrier(0x008, 1, 0); __builtin_amdgcn_sched_group_barrier(0x100, 1, 0); }
    __builtin_amdgcn_sched_group_barrier(0x008, 4, 0);
    __builtin_amdgcn_sched_barrier(0);
  };
  auto sync = [&]() {
    __builtin_amdgcn_s_waitcnt(0xc07f);
    asm volatile("" ::: "memory");
    __builtin_amdgcn_s_barrier();
    __builtin_amdgcn_sched_barrier(0);
  };
  auto step = [&](int u, bool last) {
    __builtin_amdgcn_s_waitcnt(0xc67f);
    __builtin_amdgcn_sched_barrier(0);
    rd(1, u, 1); mm(0); mix();
    if (!last) { sync(); rd(0, (u + 1) % NS, 0); mm(1); mix(); }
    else mm(1);
  };
  sync();
  rd(0, 0, 0);
  __builtin_amdgcn_sched_barrier(0);
#pragma unroll 1
  for (int j0 = 0; j0 < NSTEP - NS; j0 += NS) {
#pragma unroll
    for (int u = 0; u < NS; u++) step(u, false);
  }
#pragma unroll
  for (int u = 0; u < NS; u++) step(u, u == NS - 1);
  bool bad = false;
  const u64 t0 = now();
  {
    const char* dst = ws + WS_EX + (size_t)((b ^ 1) * 4 + wave) * 4096;
    auto pk = [&](int m, int n) { return pk8(acc[m][n], acc[m][n + 1]); };
    st16_so<0>(dst, lane * 16, pk(2, 0));
    st16_so<1024>(dst, lane * 16, pk(2, 2));
    st16_so<2048>(dst, lane * 16, pk(3, 0));
    st16_so<3072>(dst, lane * 16, pk(3, 2));
    const char* src = ws + WS_EX + (size_t)(b * 4 + wave) * 4096;
    u32x4 r[4];
    const u32 one = g == li >> 2 ? 0x3f80u << (li & 1) * 16 : 0;
    u32x4 u0 = u32x4{li & 2 ? 0 : one, li & 2 ? one : 0, 0, 0}, u1 = u32x4{0, 0, u0[0], u0[1]};
    asm volatile("" : "+v"(u0), "+v"(u1));
    const bf16x8 s0 = __builtin_bit_cast(bf16x8, u0), s1 = __builtin_bit_cast(bf16x8, u1);
    {
      wait_vm<0>();
      if (lane == 0) st4_sys(ws + WS_XF + ((b ^ 1) * 4 + wave) * 128, call);
      if (wave == 0 && lane == 63) st4_sys(ctr, call);
      const char* pf = lane == 0 ? ws + WS_XF + (b * 4 + wave) * 128 : (const char*)tmo;
      while (true) {
        const u32 f = ld4_sys(pf);
        if (__builtin_amdgcn_readfirstlane(f) >= call) break;
        if (__builtin_amdgcn_readlane(f, 63) == call || now() - t0 > DEADLINE) { bad = true; break; }
        __builtin_amdgcn_s_sleep(1);
      }
      ld64_sys<1024>(r, src, lane * 16);
    }
#pragma unroll
    for (int i = 0; i < 4; i++) {
      if (CFG_MADD) {
        const bf16x8 v = __builtin_bit_cast(bf16x8, r[i]);
        acc[i >> 1][2 * (i & 1)] = __builtin_amdgcn_mfma_f32_16x16x32_bf16(s0, v, acc[i >> 1][2 * (i & 1)], 0, 0, 0);
        acc[i >> 1][2 * (i & 1) + 1] = __builtin_amdgcn_mfma_f32_16x16x32_bf16(s1, v, acc[i >> 1][2 * (i & 1) + 1], 0, 0, 0);
      } else {
#pragma unroll
        for (int e = 0; e < 4; e++) {
          acc[i >> 1][2 * (i & 1) + (e >> 1)][2 * (e & 1)] += lo16(r[i][e]);
          acc[i >> 1][2 * (i & 1) + (e >> 1)][2 * (e & 1) + 1] += hi16(r[i][e]);
        }
      }
    }
  }
  float mx = 0.f;
#pragma unroll
  for (int i = 0; i < 8; i++)
#pragma unroll
    for (int e = 0; e < 4; e++) mx = fmaxf(mx, acc[i >> 2][i & 3][e]);
  wg_max(bad ? 0xffffffffu : wave_max(__builtin_bit_cast(u32, mx)), sm, ws, b, lane, call);
  __builtin_amdgcn_sched_barrier(0);
  u32 neg = 0;
#pragma unroll
  for (int j = 31; j >= 0; j--) {
    const float v = acc[j >> 4][(j >> 2) & 3][j & 3];
    neg = __builtin_amdgcn_alignbit(neg, __builtin_bit_cast(u32, v), 31);
  }
  auto mask = [&]() {
    if (CFG_M4) {
      const auto two = __builtin_amdgcn_permlane32_swap(neg, neg, false, false);
      if (lane < 32) st16_so<0>(pmsk, lane * 16, u32x4{two[0], call, two[1], call});
    } else st8_sys((char*)pmsk + lane * 8, u32x2{neg, call});
  };
  const bool hold = CFG_AM && id == 31;
  if (!hold) mask();
  const u32 cnt = __builtin_popcount(~neg), inc = wave_scan(cnt);
  const int total = __builtin_amdgcn_readlane(inc, 63);
  const u32 zb = (u32)(size_t)smem + XRING + 512 + wave * 8704;
  {
    u32 p = zb + (inc - cnt) * 4;
    const u32 skip = zb - 512 + lane * 4;
#pragma unroll
    for (int j = 0; j < 32; j++) {
      const float v = acc[j >> 4][(j >> 2) & 3][j & 3];
      u32 t = (u32)(__builtin_bit_cast(int, v) >> 31);
      asm("" : "+v"(t));
      *(LAS float*)(size_t)(((t & skip) | (~t & p)) + 4 * j) = v;
      p += t << 2;
      asm("" : "+v"(p));
    }
  }
  asm volatile("" ::: "memory");
  const char* dst = pstg;
  const u32x2 tg = tags((call + 1) >> 1, early);
  if (early) {
    u32x4 c[4];
#pragma unroll
    for (int i = 0; i < 4; i++) {
      const LAS f32x4* p = (const LAS f32x4*)(size_t)(zb + i * 2048 + lane * 32);
      c[i] = (pk8(p[0], p[1]) & 0x7fff7fffu) | u32x4{tg[0], tg[1], tg[0], tg[1]};
    }
    if (CFG_EDLY) { const u64 t1 = now(); while (now() - t1 < (u32)(CFG_EOFF + CFG_EDLY * (((id & ~1) + ks) * 4 + wave) / (CFG_NE * 4))) __builtin_amdgcn_s_sleep(1); }
    if (!bad) {
      if (lane * 8 < total) st16_so<0>(dst, lane * 16, c[0]);
      if (lane * 8 + 512 < total) st16_so<0>(dst + cell_off(64), lane * 16, c[1]);
      if (lane * 8 + 1024 < total) st16_so<0>(dst + cell_off(128), lane * 16, c[2]);
      if (lane * 8 + 1536 < total) st16_so<0>(dst + cell_off(192), lane * 16, c[3]);
    } else if (lane == 0) st4_sys(tmo, call);
  } else {
    f32x4 y[2][4];
#pragma unroll
    for (int i = 0; i < 8; i++) y[i >> 2][i & 3] = *(LAS f32x4*)(size_t)(zb + (i >> 2) * 4096 + lane * 64 + (i & 3) * 16);
    if (hold) {
      while (__builtin_amdgcn_readfirstlane(lds_ld8(sm + 2)[1]) != call && now() - t0 < 2 * DEADLINE) __builtin_amdgcn_s_sleep(1);
      mask();
    }
    const u32 gm = get_gmax(call, t0, bad, [&]() {});
    if (gm != 0xffffffffu) {
      const float inv = inv_scale(gm);
      auto q16 = [&](const f32x4* v) {
        return (u32x4{quant4(v[0], inv), quant4(v[1], inv), quant4(v[2], inv), quant4(v[3], inv)} & 0x7f7f7f7fu) | tg[0];
      };
      if (lane * 16 < total) st16_so<0>(dst, lane * 16, q16(y[0]));
      if (lane * 16 + 1024 < total) st16_so<0>(dst + cell_off(64), lane * 16, q16(y[1]));
    }
  }
}

static bool launch_setup() {
  hipError_t e = hipFuncSetAttribute((const void*)gemm_a2a, hipFuncAttributeMaxDynamicSharedMemorySize, LDS_BYTES);
  if (e != hipSuccess) { fprintf(stderr, "HIP error %s\n", hipGetErrorString(e)); return false; }
  return true;
}
static void launch_impl(int rank, hipStream_t stream, const void* x, const void* w, char* const* out_dev, char* const* ws_dev,
                        char* const* out_host, char* const* ws_host) {
  gemm_a2a<<<256, 512, LDS_BYTES, stream>>>((const char*)x, (const char*)w, out_dev, ws_dev, ws_host[rank], out_host[rank], rank);
}
extern "C" void kernel_launch(int rank, int nranks, ncclComm_t comm, hipStream_t stream, void* const* d_in, const int* in_sizes, int n_in,
                              void* d_out, int out_size, const int* in_dtypes, int out_dtype, void* const* d_out_all,
                              void* const* d_ws_all, size_t ws_bytes) {
  if (nranks != NR || (unsigned)rank >= (unsigned)NR || n_in < 2 || in_sizes[0] != MS * KD || in_sizes[1] != KD * ND ||
      out_size != NR * MS * NCOL || ws_bytes < WS_END) {
    fprintf(stderr, "unexpected problem shape\n");
    return;
  }
  if (!launch_setup()) return;
  launch_impl(rank, stream, d_in[0], d_in[1], (char* const*)d_out_all, (char* const*)d_ws_all, (char* const*)d_out_all, (char* const*)d_ws_all);
}
```

```cpp
#include <hip/hip_runtime.h>
#include <rccl/rccl.h>
#include <cstdio>

typedef unsigned u32;
typedef unsigned long long u64;
typedef u32 u32x4 __attribute__((ext_vector_type(4)));
typedef u32 u32x2 __attribute__((ext_vector_type(2)));
typedef float f32x4 __attribute__((ext_vector_type(4)));
typedef __bf16 bf16x4 __attribute__((ext_vector_type(4)));
typedef __bf16 bf16x8 __attribute__((ext_vector_type(8)));
typedef __bf16 bf16x2 __attribute__((ext_vector_type(2)));
#define LAS __attribute__((address_space(3)))

#define CFG_PSLP 2
#define CFG_PDLY 220
#define CFG_PH 2
#define CFG_PX 0
#define CFG_KSF 0
#define CFG_REP 8
#define CFG_NE 10
#define CFG_EDLY 180
#define CFG_EOFF 0
#define CFG_AM 1
#define CFG_MADD 1
#define CFG_M4 1
#define CFG_EX 16
#define CFG_D 4
#define CFG_XD 1
#define CFG_NW 3
#define CFG_ASLP 0
#define CFG_GSLP 1
#define CFG_GDLY 200
#define CFG_MZ 0
#define CFG_GSTR 8
#define CFG_LSTR 8
#define CFG_AVM 0
#define CFG_GVM 0
#define CFG_ADLY 0
#define CFG_GH 1
#define CFG_EP 0
#define CFG_RV 30
#define CFG_SR 0
#define CFG_LD 130
#define CFG_RD 100
#define CFG_ESLP 2
#define CFG_XS 0x2211
#define CFG_XP 0
#define CFG_XDLY 0
#define CFG_XSLP 1
#define CFG_DM 2
constexpr int NAGG = 8, REP = CFG_REP, DEF = CFG_D;
constexpr int NR = 8, MS = 512, KD = 4096, ND = 4096, NCOL = 512;
constexpr int NSTEP = KD / 2 / 64, NS = 4, BLK = 16384, NBAR = NSTEP + DEF, XSTR = 6144;
constexpr int XRING = 0, WRING = NS * BLK, SMALL = 2 * NS * BLK, LDS_BYTES = SMALL + 256;
constexpr u64 DEADLINE = 100000000ull;

constexpr size_t WS_CTR = 0;
constexpr size_t WS_TMO = 32768 + 4;
constexpr size_t WS_LMAX = 98304;
constexpr size_t WS_GMAX = 524288;
constexpr size_t WS_TS = 786432;
constexpr size_t WS_EX = 1u << 20;
constexpr size_t WS_STG = WS_EX + (size_t)1024 * XSTR;
#define CFG_LAY 0
constexpr u32 PLANE = 1u << 20, SSTR = CFG_LAY == 0 ? 1024 : CFG_LAY == 1 ? 2048 : 4096;
__device__ __forceinline__ u32 cell_off(u32 c) { return CFG_LAY == 0 ? (c >> 6) * PLANE + (c & 63) * 16 : CFG_LAY == 1 ? (c >> 7) * 2 * PLANE + (c & 127) * 16 : c * 16; }
constexpr size_t WS_MSK = WS_STG + (size_t)2 * 1024 * 4096;
constexpr size_t WS_END = WS_MSK + (size_t)2 * 1024 * 512;

template <int N> __device__ __forceinline__ void wait_vm() { asm volatile("s_waitcnt vmcnt(%0)" ::"i"(N) : "memory"); }
constexpr int nld(int s) { return s >= NSTEP ? 0 : CFG_XD && s >= NSTEP - DEF ? 12 : 16; }
constexpr int xat(int i) { return NSTEP - NS + (CFG_XS >> 4 * i & 15); }
constexpr int left(int j) {
  int n = nld(j + 1) + nld(j + 2);
  for (int i = 0; i < DEF; i++) if (CFG_XD && xat(i) < j && i > j - NSTEP) n += 4;
  return n;
}
template <int V> struct Num { static constexpr int v = V; };
template <int J, class F> __device__ __forceinline__ void tail(F&& f) { if constexpr (J < NBAR) { f(Num<J>{}); tail<J + 1>(f); } }
__device__ __forceinline__ void st16_sys(void* p, u32x4 v) { asm volatile("global_store_dwordx4 %0, %1, off sc0 sc1\n\ts_nop 1" ::"v"(p), "v"(v) : "memory"); }
template <int O> __device__ __forceinline__ void st16_so(const char* base, u32 off, u32x4 v) {
  asm volatile("global_store_dwordx4 %0, %1, %2 offset:%3 sc0 sc1\n\ts_nop 1" ::"v"(off), "v"(v), "s"(base), "i"(O) : "memory");
}
#define CFG_XST "sc0 sc1"
#define CFG_XLD 17
template <int O> __device__ __forceinline__ void st16_x(const char* base, u32 off, u32x4 v) {
  asm volatile("global_store_dwordx4 %0, %1, %2 offset:%3 " CFG_XST "\n\ts_nop 1" ::"v"(off), "v"(v), "s"(base), "i"(O) : "memory");
}
__device__ __forceinline__ void st8_sys(void* p, u32x2 v) { asm volatile("global_store_dwordx2 %0, %1, off sc0 sc1\n\ts_nop 1" ::"v"(p), "v"(v) : "memory"); }
__device__ __forceinline__ void st4_sys(void* p, u32 v) { asm volatile("global_store_dword %0, %1, off sc0 sc1\n\ts_nop 1" ::"v"(p), "v"(v) : "memory"); }
__device__ __forceinline__ u32 ld4_sys(const void* p) { u32 v; asm volatile("global_load_dword %0, %1, off sc0 sc1\n\ts_waitcnt vmcnt(0)" : "=&v"(v) : "v"(p) : "memory"); return v; }
__device__ __forceinline__ u32x2 ld8_sys(const void* p) { u32x2 v; asm volatile("global_load_dwordx2 %0, %1, off sc0 sc1\n\ts_waitcnt vmcnt(0)" : "=&v"(v) : "v"(p) : "memory"); return v; }
__device__ __forceinline__ u64 now() { return __builtin_amdgcn_s_memrealtime(); }
__device__ __forceinline__ u32 vm_peek() { u32 v; asm volatile("s_getreg_b32 %0, hwreg(HW_REG_IB_STS, 0, 4)" : "=s"(v) :: "memory"); return v; }
__device__ __forceinline__ char* uni(char* p) {
  const u64 v = (u64)p;
  return (char*)(((u64)(u32)__builtin_amdgcn_readfirstlane((u32)(v >> 32)) << 32) | (u32)__builtin_amdgcn_readfirstlane((u32)v));
}
__device__ __forceinline__ u32 clk() { return (u32)__builtin_amdgcn_s_memtime(); }
__device__ __forceinline__ u32 pk2(float a, float b) { return __builtin_bit_cast(u32, bf16x2{(__bf16)a, (__bf16)b}); }
__device__ __forceinline__ u32x4 pk8(f32x4 a, f32x4 c) { return u32x4{pk2(a[0], a[1]), pk2(a[2], a[3]), pk2(c[0], c[1]), pk2(c[2], c[3])}; }
__device__ __forceinline__ float lo16(u32 v) { return __builtin_bit_cast(float, v << 16); }
__device__ __forceinline__ float hi16(u32 v) { return __builtin_bit_cast(float, v & 0xffff0000u); }

__device__ __forceinline__ u32 wave_max(u32 v) {
  v = max(v, (u32)__builtin_amdgcn_update_dpp(0, (int)v, 0xb1, 0xf, 0xf, true));
  v = max(v, (u32)__builtin_amdgcn_update_dpp(0, (int)v, 0x4e, 0xf, 0xf, true));
  v = max(v, (u32)__builtin_amdgcn_update_dpp(0, (int)v, 0x141, 0xf, 0xf, true));
  v = max(v, (u32)__builtin_amdgcn_update_dpp(0, (int)v, 0x140, 0xf, 0xf, true));
  return max(max((u32)__builtin_amdgcn_readlane(v, 0), (u32)__builtin_amdgcn_readlane(v, 16)),
             max((u32)__builtin_amdgcn_readlane(v, 32), (u32)__builtin_amdgcn_readlane(v, 48)));
}
template <int SLP, int VM> __device__ __forceinline__ u32 poll(const char* p, u32 call, u64 t0, bool& bad, u32x2& r) {
  u32 v, late;
  u64 ex;
  asm volatile("s_mov_b64 %2, exec\n\tglobal_load_dwordx2 v[100:101], %4, off sc0 sc1\n\ts_waitcnt vmcnt(0)\n\ts_cmp_eq_u32 0, 1\n\ts_branch 3f\n"
               "1:\n\ts_sleep %8\n\tglobal_load_dwordx2 v[100:101], %4, off sc0 sc1\n\t"
               "s_memrealtime s[60:61]\n\ts_waitcnt vmcnt(%9) lgkmcnt(0)\n\t"
               "s_sub_u32 %1, s60, %6\n\ts_cmp_gt_u32 %1, %7\n\ts_cbranch_scc1 2f\n\t"
               "3:\n\tv_xor_b32 %0, v100, v101\n\tv_cmpx_ne_u32 vcc, %5, %0\n\ts_cbranch_execnz 1b\n"
               "2:\n\ts_cselect_b32 %1, 1, 0\n\ts_mov_b64 exec, %2\n\tv_mov_b32 %0, v100\n\ts_nop 1"
               : "=&v"(v), "=&s"(late), "=&s"(ex), "=&{v[100:101]}"(r)
               : "v"(p), "s"(call), "s"((u32)t0), "s"((u32)DEADLINE), "i"(SLP), "i"(VM) : "memory", "vcc", "scc", "s60", "s61");
  bad |= __builtin_amdgcn_readfirstlane(late) != 0;
  return v;
}
__device__ __forceinline__ void settle(u32x2& r) { asm volatile("s_waitcnt vmcnt(0)" : "+{v[100:101]}"(r) :: "memory"); }
__device__ __forceinline__ void wg_max(u32 wmx, LAS u32* sm, char* ws, int b, int lane, u32 call) {
  if (lane == 0) {
    u32 tk, m4;
    asm volatile("ds_max_u32 %2, %3\n\tds_add_rtn_u32 %0, %2, %4 offset:4\n\tds_read_b32 %1, %2\n\ts_waitcnt lgkmcnt(0)"
                 : "=&v"(tk), "=&v"(m4) : "v"((u32)(size_t)sm), "v"(wmx), "v"(1u) : "memory");
    if (tk == 3) st8_sys(ws + WS_LMAX + (b & 7) * 4096 + (b >> 3) * CFG_LSTR, u32x2{m4, m4 ^ call});
  }
}
__device__ __forceinline__ u32 wait_gmax(char* ws, int b, int lane, u32 call, u64 t0, bool bad) {
  const char* pg = ws + WS_GMAX + (size_t)((call & 1) * REP + b % REP) * 8192 + lane * CFG_GSTR;
  u32 gm;
  while (true) {
    const u32x2 r = ld8_sys(pg);
    gm = r[0];
    if (__all((r[0] ^ r[1]) == call)) break;
    if (now() - t0 > DEADLINE) { bad = true; break; }
    __builtin_amdgcn_s_sleep(1);
  }
  if (bad && lane == 0) st4_sys(ws + WS_TMO, call);
  return bad ? 0xffffffffu : wave_max(gm);
}
__device__ __forceinline__ float inv_scale(u32 gm) {
  const float scale = __builtin_bit_cast(float, gm) / 127.0f;
  return scale > 0.f ? 1.0f / scale : 0.f;
}
__device__ __forceinline__ u32 quant4(f32x4 y, float inv) {
  u32 q[4];
#pragma unroll
  for (int e = 0; e < 4; e++) q[e] = __builtin_bit_cast(u32, fminf(__builtin_fmaf(y[e], inv, 8388608.f), 8388735.f));
  return __builtin_amdgcn_perm(__builtin_amdgcn_perm(q[3], q[2], 0x0c0c0400u), __builtin_amdgcn_perm(q[1], q[0], 0x0c0c0400u), 0x05040100u);
}
__device__ __forceinline__ u32 spread4(u32 nb) { return __builtin_amdgcn_ubfe(nb, 0, 4) * 0x204081u & 0x01010101u; }
__device__ __forceinline__ u32x2 tags(u32 use, bool wide) {
  const u32 t = spread4(use) << 7;
  return wide ? u32x2{(use & 1) << 15 | (use & 2) << 30, (use & 4) << 13 | (use & 8) << 28} : u32x2{t, t};
}
__device__ __forceinline__ u32 wave_scan(u32 v) {
  v += (u32)__builtin_amdgcn_update_dpp(0, (int)v, 0x111, 0xf, 0xf, false);
  v += (u32)__builtin_amdgcn_update_dpp(0, (int)v, 0x112, 0xf, 0xf, false);
  v += (u32)__builtin_amdgcn_update_dpp(0, (int)v, 0x114, 0xf, 0xf, false);
  v += (u32)__builtin_amdgcn_update_dpp(0, (int)v, 0x118, 0xf, 0xf, false);
  v += (u32)__builtin_amdgcn_update_dpp(0, (int)v, 0x142, 0xa, 0xf, false);
  v += (u32)__builtin_amdgcn_update_dpp(0, (int)v, 0x143, 0xc, 0xf, false);
  return v;
}

template <int STR> __device__ __forceinline__ void ld64_sys(u32x4* r, const char* base, u32 off) {
  asm volatile("global_load_dwordx4 %0, %4, %5 sc0 sc1\n\tglobal_load_dwordx4 %1, %4, %6 sc0 sc1\n\t"
               "global_load_dwordx4 %2, %4, %7 sc0 sc1\n\tglobal_load_dwordx4 %3, %4, %8 sc0 sc1\n\ts_waitcnt vmcnt(0)"
               : "=&v"(r[0]), "=&v"(r[1]), "=&v"(r[2]), "=&v"(r[3])
               : "v"(off), "s"(base), "s"(base + STR), "s"(base + 2 * STR), "s"(base + 3 * STR) : "memory");
}
__device__ __forceinline__ u32x2 lds_ld8(LAS u32* p) { u32x2 v; asm volatile("ds_read_b64 %0, %1\n\ts_waitcnt lgkmcnt(0)" : "=&v"(v) : "v"((u32)(size_t)p) : "memory"); return v; }

__global__ __launch_bounds__(512, 2) void gemm_a2a(const char* __restrict__ x, const char* __restrict__ w,
                                                   char* const* out_all, char* const* ws_all, char* ws, char* out, int rank) {
  extern __shared__ __attribute__((aligned(16))) char smem[];
  const int b = blockIdx.x, lane = threadIdx.x & 63;
  const int wave = __builtin_amdgcn_readfirstlane(threadIdx.x >> 6);
  const int xcd = b & 7, idx = b >> 3;
  const int id = idx ^ CFG_EX;
  const int ks = (xcd ^ CFG_KSF) & 1, mt = (id >> 1) & 3, ntl = (id & 1) * 4 + (id >> 3), nt = (xcd >> 1) * 8 + ntl;
  const int g = lane >> 4, li = lane & 15;
  const int rid = id ^ CFG_RV, src = (xcd + CFG_SR * (id >> 1)) & 7, pos = ((id ^ CFG_EP) & ~1) + ks, rpos = rid ^ CFG_EP;
  const int dest = nt >> 2, item = rank * 32 + (id & ~1) + ks, mitem = src * 32 + rid;
  const bool aggr = (id ^ CFG_EP) == 31;
  char* const mine = out + (size_t)(src * MS + (rid & 7) * 64) * NCOL + (rid >> 3) * 128;
  LAS u32* sm = (LAS u32*)(smem + SMALL);
  u32* tmo = (u32*)(ws + WS_TMO);
  auto get_gmax = [&](u32 call, u64 t0, bool bad, auto&& idle) {
    u32 gm;
    while (true) {
      const u32x2 v = lds_ld8(sm + 6);
      gm = __builtin_amdgcn_readfirstlane(v[0]);
      if (__builtin_amdgcn_readfirstlane(v[1]) == call) break;
      if (now() - t0 > 2 * DEADLINE) { bad = true; break; }
      idle();
    }
    if (bad && lane == 0) st4_sys(tmo, call);
    return bad ? 0xffffffffu : gm;
  };
  auto recv = [&](int h, u32 call, u64 t0) {
    const bool early = rpos < CFG_NE;
    const u32 gm = get_gmax(call, t0, false, [&]() { __builtin_amdgcn_s_sleep(2); });
    bool bad = gm == 0xffffffffu;
    t0 = now();
    const char* pm = ws + WS_MSK + (size_t)(((call & 1) * 256 + mitem) * 4 + h) * 512 + (CFG_M4 ? (lane & 31) * 16 + (lane >> 5) * 8 : lane * 8);
    u32 km = 0;
    while (!bad) {
      const u32x2 r = ld8_sys(pm);
      km = ~r[0];
      if (__all(r[1] >= call)) break;
      if (now() - t0 > DEADLINE) bad = true;
      __builtin_amdgcn_s_sleep(2);
    }
    const u32 cnt = __builtin_popcount(km), inc = wave_scan(cnt);
    const int len = early ? (__builtin_amdgcn_readlane(inc, 63) + 7) >> 3 : (__builtin_amdgcn_readlane(inc, 63) + 15) >> 4;
    const u32 lb = (u32)(size_t)smem + WRING + h * 4096;
    u32 ad[8], sel[8], keep[8];
#pragma unroll
    for (int k = 0; k < 8; k++) {
      const u32 sh = 4 * (k ^ (g & 1));
      const u32 o = inc - cnt + __builtin_popcount(km & ((1u << sh) - 1)), sp = spread4(km >> sh);
      ad[k] = lb + (o & ~3u);
      sel[k] = (sp << 8) + (sp << 16) + (sp << 24) + (o & 3) * 0x01010101u;
      keep[k] = sp * 127;
      asm volatile("" : "+v"(ad[k]), "+v"(sel[k]), "+v"(keep[k]));
    }
    const char* stg = ws + WS_STG + (size_t)(call & 1) * 4 * PLANE + (mitem * 4 + h) * SSTR;
    const u32 use = (call + 1) >> 1;
    const u32x2 tg = tags(use, early);
    const u32x4 tag = u32x4{tg[0], tg[1], tg[0], tg[1]};
    auto cl = [&](int i) { return (u32)max(min(lane + 64 * i, len - 1), 0); };
    auto off = [&](int i) { return cell_off(cl(i)); };
    if (early) {
      u32x4 r[4];
      while (!bad && len > 0) {
        asm volatile("global_load_dwordx4 %0, %4, %8 sc0 sc1\n\tglobal_load_dwordx4 %1, %5, %8 sc0 sc1\n\t"
                     "global_load_dwordx4 %2, %6, %8 sc0 sc1\n\tglobal_load_dwordx4 %3, %7, %8 sc0 sc1\n\ts_waitcnt vmcnt(0)"
                     : "=&v"(r[0]), "=&v"(r[1]), "=&v"(r[2]), "=&v"(r[3]) : "v"(off(0)), "v"(off(1)), "v"(off(2)), "v"(off(3)), "s"(stg) : "memory");
        const u32x4 e = (r[0] ^ tag) | (r[1] ^ tag) | (r[2] ^ tag) | (r[3] ^ tag);
        if (__all(((e[0] | e[1] | e[2] | e[3]) & 0x80008000u) == 0)) break;
        if (now() - t0 > DEADLINE) bad = true;
        __builtin_amdgcn_s_sleep(CFG_ESLP);
      }
      const float inv = inv_scale(gm);
#pragma unroll
      for (int i = 0; i < 4; i++) {
        if (i * 64 >= len) break;
        f32x4 y[2];
#pragma unroll
        for (int e = 0; e < 4; e++) {
          const u32 v = r[i][e];
          y[e >> 1][2 * (e & 1)] = __builtin_fabsf(lo16(v));
          y[e >> 1][2 * (e & 1) + 1] = __builtin_bit_cast(float, v & 0x7fff0000u);
        }
        *(LAS u32x2*)(size_t)(lb + cl(i) * 8) = u32x2{quant4(y[0], inv), quant4(y[1], inv)};
      }
    } else {
      const u32 dly = CFG_PDLY - CFG_RD * ((32 - rpos) * 4 - 1 - h) / ((32 - CFG_NE) * 4);
      while (now() - t0 < dly) __builtin_amdgcn_s_sleep(1);
      u32x4 r0, r1;
      while (!bad && len > 0) {
        asm volatile("global_load_dwordx4 %0, %2, %4 sc0 sc1\n\tglobal_load_dwordx4 %1, %3, %4 sc0 sc1\n\ts_waitcnt vmcnt(0)"
                     : "=&v"(r0), "=&v"(r1) : "v"(off(0)), "v"(off(1)), "s"(stg) : "memory");
        const u32x4 e = (r0 ^ tag) | (r1 ^ tag);
        if (__all(((e[0] | e[1] | e[2] | e[3]) & 0x80808080u) == 0)) break;
        if (now() - t0 > DEADLINE) bad = true;
        __builtin_amdgcn_s_sleep(CFG_PSLP);
      }
      *(LAS u32x4*)(size_t)(lb + cl(0) * 16) = r0;
      *(LAS u32x4*)(size_t)(lb + cl(1) * 16) = r1;
    }
    asm volatile("" ::: "memory");
    u32 d[8];
#pragma unroll
    for (int k = 0; k < 8; k++) {
      const LAS u32* q = (const LAS u32*)(size_t)ad[k];
      d[k] = __builtin_amdgcn_perm(q[1], q[0], sel[k]) & keep[k];
    }
    if (bad) {
#pragma unroll
      for (int k = 0; k < 8; k++) { d[k] = 0x80808080u; asm volatile("" : "+v"(d[k])); }
    }
    const char* op = mine + (h >> 1) * 32 * NCOL + (h & 1) * 64;
#pragma unroll
    for (int m = 0; m < 2; m++) {
      const u32x4 v = u32x4{d[4 * m], d[4 * m + 1], d[4 * m + 2], d[4 * m + 3]};
      st16_so<0>(op + m * 16 * NCOL, li * NCOL + g * 16, v);
    }
    const int cell = (lane >> 3) * 64 + (use & 7) * 8 + (lane & 7);
    if (lane < (early ? 32 : 16) && cell >= len) st16_so<0>(stg, cell_off(cell), tag);
    if (bad && lane == 0) st4_sys(tmo, call);
  };
  if (wave >= 6) {
    {
#pragma unroll 1
      for (int nb = 0; nb < NBAR; nb++) __builtin_amdgcn_s_barrier();
    }
    const u32 call = __builtin_amdgcn_readfirstlane(sm[2]);
    const u64 t0 = now();
    if (wave == 6) {
      if (CFG_GDLY) while (now() - t0 < CFG_GDLY) __builtin_amdgcn_s_sleep(2);
      if (CFG_GH && aggr) while (__builtin_amdgcn_readfirstlane(lds_ld8(sm + 2)[1]) != call && now() - t0 < 2 * DEADLINE) __builtin_amdgcn_s_sleep(2);
      if (CFG_NW & 2) {
        bool bad = false;
        u32x2 r;
        u32 gm = poll<CFG_GSLP, CFG_GVM>(ws + WS_GMAX + (size_t)((call & 1) * REP + b % REP) * 8192 + lane * CFG_GSTR, call, t0, bad, r);
        gm = bad ? 0xffffffffu : wave_max(gm);
        if (lane == 0) *(LAS u32x2*)(sm + 6) = u32x2{gm, call};
        settle(r);
        if (bad && lane == 0) st4_sys(tmo, call);
      } else {
        const u32 gm = wait_gmax(ws, b, lane, call, t0, false);
        if (lane == 0) *(LAS u32x2*)(sm + 6) = u32x2{gm, call};
      }
    }
    recv(wave - 4, call, t0);
    return;
  }
  if (wave >= 4) {
    const int lw = wave - 4;
    const int kx = CFG_PH ? mt ^ (ntl >> 1) : (mt ^ ntl) & 3;
    const int ph = (ntl & 1 ? 0 : CFG_PH) + CFG_PX * (xcd >> 2);
    auto rx = __builtin_amdgcn_make_buffer_rsrc((void*)x, 0, MS * KD * 2, 0x00020000);
    auto rw = __builtin_amdgcn_make_buffer_rsrc((void*)w, 0, KD * ND * 2, 0x00020000);
    int vx[2], vw[2];
#pragma unroll
    for (int h = 0; h < 2; h++) {
      vx[h] = (lane >> 3) * 8192 + (((lane & 7) ^ ((((lane >> 3) + 8 * h) >> 1) & 7)) << 4);
      vw[h] = g * 8192 + ((li ^ g ^ (8 * h)) << 4);
    }
    const int sx0 = (mt * 128 + lw * 32) * 8192 + ks * KD;
    const int sw0 = (ks * (KD / 2) + lw * 32) * 8192 + nt * 256;
    auto spos = [&](int s) { return ((s + ph) & (NSTEP - 1)) ^ kx; };
    auto ix = [&](int s, int slot, int hf) {
      LAS char* l = (LAS char*)(smem + XRING + slot * BLK + hf * 8192 + lw * 4096);
      const int o = sx0 + (hf ^ ks) * 64 * 8192 + spos(s) * 128;
#pragma unroll
      for (int i = 0; i < 4; i++) __builtin_amdgcn_raw_ptr_buffer_load_lds(rx, l + i * 1024, 16, vx[i & 1], o + i * 8 * 8192, 0, 0);
    };
    auto issue = [&](int s, int slot, bool all) {
      if (all) ix(s, slot, 0);
      ix(s, slot, 1);
      LAS char* l = (LAS char*)(smem + WRING + slot * BLK + lw * 8192);
      const int sp = spos(s);
#pragma unroll
      for (int i = 0; i < 8; i++) __builtin_amdgcn_raw_ptr_buffer_load_lds(rw, l + i * 1024, 16, vw[(i >> 1) & 1], sw0 + (sp * 64 + i * 4) * 8192, 0, 0);
    };
    issue(0, 0, true); issue(1, 1, true);
    wait_vm<16>();
    __builtin_amdgcn_s_barrier();
    issue(2, 2, true); issue(3, 3, true);
#pragma unroll 1
    for (int j = 1; j <= NSTEP - 2 * NS; j++) {
      wait_vm<32>();
      __builtin_amdgcn_s_barrier();
      issue(j - 1 + NS, (j - 1) & 3, true);
    }
    tail<NSTEP - 2 * NS + 1>([&](auto jc) {
      constexpr int j = jc.v;
      wait_vm<left(j)>();
      __builtin_amdgcn_s_barrier();
      if (j <= NSTEP - NS) {
        int s = j - 1 + NS;
        asm volatile("" : "+s"(s));
        issue(s, s & 3, nld(j - 1 + NS) == 16);
      }
#pragma unroll
      for (int i = 0; i < DEF; i++) {
        if (!CFG_XD || xat(i) != j) continue;
        int s = NSTEP - DEF + i;
        asm volatile("" : "+s"(s));
        ix(s, s & 3, 0);
      }
    });
    const u32 call = __builtin_amdgcn_readfirstlane(sm[2]);
    const u64 t0 = now();
    bool bad = false;
    if (lw == 1 && aggr) {
      char* pw = ws_all[lane & 7];
      u32 gm = 0;
      u32x2 r;
      char* pd = pw + WS_GMAX + (size_t)((call & 1) * REP + (lane >> 3)) * 8192 + (rank * NAGG + xcd) * CFG_GSTR;
      if (CFG_ADLY) while (now() - t0 < CFG_ADLY) __builtin_amdgcn_s_sleep(2);
      asm volatile("" : "+v"(pd));
      if (CFG_NW & 1) gm = poll<CFG_ASLP, CFG_AVM>(ws + WS_LMAX + xcd * 4096 + (lane & 31) * CFG_LSTR, call, t0, bad, r);
      else {
        const char* pg = lane < 32 ? ws + WS_LMAX + xcd * 4096 + lane * CFG_LSTR : (const char*)tmo - 4;
        while (true) {
          r = ld8_sys(pg);
          gm = lane < 32 ? r[0] : 0;
          if (__all(lane >= 32 || (r[0] ^ r[1]) == call)) break;
          if (__builtin_amdgcn_readlane(r[1], 63) == call || now() - t0 > DEADLINE) { bad = true; break; }
        }
      }
      gm = bad ? 0xffffffffu : wave_max(gm);
      if (lane < NR * REP) st8_sys(pd, u32x2{gm, gm ^ call});
      if (lane == 0) sm[3] = call;
      if (CFG_NW & 1) settle(r);
    }
    recv(lw, call, t0);
    return;
  }
  const int wm = wave >> 1, wn = wave & 1;
  u32* ctr = (u32*)(ws + WS_CTR + b * 128);
  const u32 call = __builtin_amdgcn_readfirstlane(ld4_sys(ctr)) + 1;
  char* const pw = uni(ws_all[dest & 7]);
  const char* pmsk = pw + WS_MSK + (size_t)(((call & 1) * 256 + item) * 4 + wave) * 512;
  const char* pstg = pw + WS_STG + (size_t)(call & 1) * 4 * PLANE + (item * 4 + wave) * SSTR;
  const char* xdst = ws + WS_EX + (size_t)((b ^ 1) * 4 + wave) * XSTR + 2048;
  char* xsrc = ws + WS_EX + (size_t)(b * 4 + wave) * XSTR;
  u32 xtag = (0x2000u | (call & 0x1fffu)) << 16, l16 = lane * 16, h16 = (lane & 31) * 16, h8 = h16 + (lane >> 5) * 8;
  asm volatile("" : "+s"(pmsk), "+s"(pstg), "+s"(xdst), "+s"(xsrc), "+s"(xtag));
  asm volatile("" : "+v"(l16), "+v"(h16), "+v"(h8));
  u32x4 u0, u1, u2;
  {
    const u32 e = g == li >> 2 ? li & 3 : 4, one = 0x3f80u << (li & 1) * 16;
    u0 = u32x4{e < 2 ? one : 0, e == 2 ? 0x3f80u : 0, 0, e == 3 ? 0x3f80u : 0}; u1 = u32x4{0, 0, u0[0], 0}; u2 = u32x4{0, 0, (e & 6) == 2 ? one : 0, 0};
    asm volatile("" : "+v"(u0), "+v"(u1), "+v"(u2));
  }
  if (wave == 0 && lane < 8) sm[lane] = lane == 2 ? call : 0;
  f32x4 acc[4][4];
#pragma unroll
  for (int i = 0; i < 16; i++) acc[i >> 2][i & 3] = f32x4{0.f, 0.f, 0.f, 0.f};
  u32 ax[2], aw[4];
#pragma unroll
  for (int kk = 0; kk < 2; kk++) ax[kk] = (u32)(size_t)smem + XRING + (32 * wm + li) * 128 + (((4 * kk + g) ^ ((li >> 1) & 7)) << 4);
  {
    const int q = li >> 2, p = li & 3;
#pragma unroll
    for (int n = 0; n < 4; n++)
      aw[n] = (u32)(size_t)smem + WRING + (8 * g + q) * 256 + (((8 * wn + 2 * p + (n >> 1)) ^ q ^ ((g & 1) << 3)) << 4) + (((n ^ p) & 1) << 3);
  }
#pragma unroll
  for (int n = 0; n < 4; n++) asm volatile("" : "+v"(aw[n]));
  bf16x8 fx[2][4], fw[2][4];
  auto rdw = [&](int buf, int n, u32 a) {
    const bf16x4 lo = __builtin_amdgcn_ds_read_tr16_b64_v4bf16((LAS bf16x4*)(size_t)a);
    const bf16x4 hi = __builtin_amdgcn_ds_read_tr16_b64_v4bf16((LAS bf16x4*)(size_t)(a + 1024));
    fw[buf][n] = __builtin_shufflevector(lo, hi, 0, 1, 2, 3, 4, 5, 6, 7);
  };
  auto rd = [&](int buf, int slot, int kk) {
    rdw(buf, 0, aw[0] + slot * BLK + kk * 8192);
#pragma unroll
    for (int m = 0; m < 4; m++) fx[buf][m] = *(LAS bf16x8*)(size_t)(ax[kk] + slot * BLK + ((m & 1) * 16 + (m >> 1) * 64) * 128);
#pragma unroll
    for (int n = 1; n < 4; n++) rdw(buf, n, aw[n] + slot * BLK + kk * 8192);
  };
  auto mm = [&](int buf) {
#pragma unroll
    for (int n = 0; n < 4; n++)
#pragma unroll
      for (int i = 0; i < 4; i++) {
        const int m = (n & 1) ? 3 - i : i;
        acc[m][n] = __builtin_amdgcn_mfma_f32_16x16x32_bf16(fw[buf][n], fx[buf][m], acc[m][n], 0, 0, 0);
      }
  };
  auto mix = [&]() {
#pragma unroll
    for (int i = 0; i < 12; i++) { __builtin_amdgcn_sched_group_barrier(0x008, 1, 0); __builtin_amdgcn_sched_group_barrier(0x100, 1, 0); }
    __builtin_amdgcn_sched_group_barrier(0x008, 4, 0);
    __builtin_amdgcn_sched_barrier(0);
  };
  auto sync = [&]() {
    __builtin_amdgcn_s_waitcnt(0xc07f);
    asm volatile("" ::: "memory");
    __builtin_amdgcn_s_barrier();
    __builtin_amdgcn_sched_barrier(0);
  };
  auto step = [&](int u, bool last) {
    __builtin_amdgcn_s_waitcnt(0xc67f);
    __builtin_amdgcn_sched_barrier(0);
    rd(1, u, 1); mm(0); mix();
    if (!last) { sync(); rd(0, (u + 1) % NS, 0); mm(1); mix(); }
    else mm(1);
  };
  sync();
  rd(0, 0, 0);
  __builtin_amdgcn_sched_barrier(0);
#pragma unroll 1
  for (int j0 = 0; j0 < NSTEP - NS; j0 += NS) {
#pragma unroll
    for (int u = 0; u < NS; u++) step(u, false);
  }
#pragma unroll
  for (int u = 0; u < NS - DEF; u++) step(u, u == NS - 1);
  auto frag = [&](int hf, int q) -> f32x4& { return acc[2 * hf + ((q ^ q >> 1) & 1)][q >> 1]; };
  auto dump = [&](int j) {
    const float tf = __builtin_bit_cast(float, xtag);
    if (j < 5) {
      const f32x4 f = frag(1, j + 1 + (j >> 1)), h = frag(1, 3 * (j >> 1));
      const u32x4 c = u32x4{pk2(f[0], f[1]), pk2(f[2], tf), pk2(h[2 * (j & 1)], h[2 * (j & 1) + 1]), pk2(f[3], tf)};
      if (j == 0) st16_x<-2048>(xdst, l16, c);
      if (j == 1) st16_x<-1024>(xdst, l16, c);
      if (j == 2) st16_x<0>(xdst, l16, c);
      if (j == 3) st16_x<1024>(xdst, l16, c);
      if (j == 4) st16_x<2048>(xdst, l16, c);
    } else {
      const f32x4 h = frag(1, 6);
      const u32 v = pk2(h[2], h[3]);
      const auto two = __builtin_amdgcn_permlane32_swap(v, v, false, false);
      st16_x<3072>(xdst, h16, u32x4{two[0], xtag, two[1], xtag});
    }
  };
  auto rs = __builtin_amdgcn_make_buffer_rsrc(xsrc, 0, XSTR, 0x00020000);
  u32x4 r[5];
  u32x2 r5;
  auto fetch = [&](int j) {
    if (j < 5) r[j] = __builtin_amdgcn_raw_buffer_load_b128(rs, l16 + (j & 3) * 1024, (j >> 2) * 4096, CFG_XLD);
    else r5 = __builtin_amdgcn_raw_buffer_load_b64(rs, h8 + 1024, 4096, CFG_XLD);
  };
  if (DEF) {
    bf16x4 wl[4];
#pragma unroll
    for (int t = 0; t < 4 * DEF; t++) {
      const int hf = t < 2 * DEF, n = t + 1, nh = n < 2 * DEF, slot = (NSTEP - DEF + (n % (2 * DEF)) / 2) & 3, o = slot * BLK + (n & 1) * 8192;
      if ((t & 1) && n < 4 * DEF) {
        if (n == 4 * DEF - 2) __builtin_amdgcn_s_waitcnt(0xc07f);
        asm volatile("" ::: "memory");
        __builtin_amdgcn_s_barrier();
        asm volatile("" ::: "memory");
        __builtin_amdgcn_sched_barrier(0);
      }
      auto lo = [&](int i) { wl[i] = __builtin_amdgcn_ds_read_tr16_b64_v4bf16((LAS bf16x4*)(size_t)(aw[i] + o)); };
      auto hi = [&](int i) {
        const bf16x4 h = __builtin_amdgcn_ds_read_tr16_b64_v4bf16((LAS bf16x4*)(size_t)(aw[i] + o + 1024));
        fw[n & 1][i] = __builtin_shufflevector(wl[i], h, 0, 1, 2, 3, 4, 5, 6, 7);
      };
      auto xr = [&](int m) { fx[n & 1][2 * nh + m] = *(LAS bf16x8*)(size_t)(ax[n & 1] + slot * BLK + (m * 16 + nh * 64) * 128); };
      if (CFG_DM == 0 && t == 2 * DEF) {
#pragma unroll
        for (int j = 0; j < 6; j++) dump(j);
        __builtin_amdgcn_sched_barrier(0);
      }
#pragma unroll
      for (int q = 0; q < 8; q++) {
        frag(hf, q) = __builtin_amdgcn_mfma_f32_16x16x32_bf16(fw[t & 1][q >> 1], fx[t & 1][2 * hf + ((q ^ q >> 1) & 1)], frag(hf, q), 0, 0, 0);
        __builtin_amdgcn_sched_barrier(0);
        if (n < 4 * DEF) {
          if (q == 0) { lo(0); hi(0); }
          if (q == 1) xr(0);
          if (q == 2) xr(1);
          if (q == 3) { lo(1); hi(1); }
          if (q == 4) lo(2);
          if (q == 5) hi(2);
          if (q == 6) lo(3);
          if (q == 7) hi(3);
        }
        const int z = (t - 2 * DEF) * 8 + q;
        if (CFG_DM == 1) {
          if (z == -4) dump(0);
          if (z == -2) dump(1);
          if (z == -1) dump(2);
          if (z == 1) dump(3);
          if (z == 2) dump(4);
          if (z == 4) dump(5);
        }
        if (CFG_DM >= 3 && z >= -4 && (z + 4) % (CFG_DM - 1) == 0 && (z + 4) / (CFG_DM - 1) < 6) dump((z + 4) / (CFG_DM - 1));
        if (CFG_DM == 2) {
          if (z == -4) dump(0);
          if (z == -3) dump(1);
          if (z == -2) dump(2);
          if (z == -1) dump(3);
          if (z == 0) dump(5);
          if (z == 1) dump(4);
        }
        if (t == 4 * DEF - CFG_XP && q >= 1 && q < 7) fetch(q - 1);
        __builtin_amdgcn_sched_barrier(0);
      }
    }
  }
  bool bad = false;
  const u64 t0 = now();
  {
    if (!DEF) {
#pragma unroll
      for (int j = 0; j < 6; j++) dump(j);
      __builtin_amdgcn_sched_barrier(0);
    }
    if (!DEF || !CFG_XP) {
      if (CFG_XDLY) while (now() - t0 < CFG_XDLY) {}
#pragma unroll
      for (int j = 0; j < 6; j++) fetch(j);
    }
    while (true) {
      u32 d = r5[1] ^ xtag;
#pragma unroll
      for (int j = 0; j < 5; j++) d |= (r[j][1] ^ xtag) | (r[j][3] ^ xtag);
      if (__all(d < 65536)) break;
      if (now() - t0 > DEADLINE) { bad = true; break; }
      if (CFG_XSLP) __builtin_amdgcn_s_sleep(CFG_XSLP);
      asm volatile("" ::: "memory");
#pragma unroll
      for (int j = 0; j < 6; j++) fetch(j);
    }
    frag(0, 6)[2] += lo16(r5[0]);
    frag(0, 6)[3] += hi16(r5[0]);
#pragma unroll
    for (int j = 0; j < 5; j++) {
      const bf16x8 v = __builtin_bit_cast(bf16x8, r[j]);
      f32x4 &f = frag(0, j + 1 + (j >> 1)), &h = frag(0, 3 * (j >> 1));
      h = __builtin_amdgcn_mfma_f32_16x16x32_bf16(__builtin_bit_cast(bf16x8, j & 1 ? u2 : u1), v, h, 0, 0, 0);
      f = __builtin_amdgcn_mfma_f32_16x16x32_bf16(__builtin_bit_cast(bf16x8, u0), v, f, 0, 0, 0);
    }
  }
  float mx = 0.f;
#pragma unroll
  for (int i = 0; i < 8; i++)
#pragma unroll
    for (int e = 0; e < 4; e++) mx = fmaxf(mx, acc[i >> 2][i & 3][e]);
  wg_max(bad ? 0xffffffffu : wave_max(__builtin_bit_cast(u32, mx)), sm, ws, b, lane, call);
  __builtin_amdgcn_sched_barrier(0);
  u32 neg = 0;
#pragma unroll
  for (int j = 31; j >= 0; j--) {
    const float v = acc[j >> 4][(j >> 2) & 3][j & 3];
    neg = __builtin_amdgcn_alignbit(neg, __builtin_bit_cast(u32, v), 31);
  }
  auto mask = [&]() {
    if (CFG_M4) {
      const auto two = __builtin_amdgcn_permlane32_swap(neg, neg, false, false);
      if (lane < 32) st16_so<0>(pmsk, lane * 16, u32x4{two[0], call, two[1], call});
    } else st8_sys((char*)pmsk + lane * 8, u32x2{neg, call});
  };
  const bool hold = CFG_AM && aggr, early = pos < CFG_NE;
  if (!hold && !CFG_MZ) mask();
  const u32 cnt = __builtin_popcount(~neg), inc = wave_scan(cnt);
  const int total = __builtin_amdgcn_readlane(inc, 63);
  const u32 zb = (u32)(size_t)smem + XRING + 512 + wave * 8704;
  {
    u32 p = zb + (inc - cnt) * 4;
    const u32 skip = zb - 512 + lane * 4;
#pragma unroll
    for (int j = 0; j < 32; j++) {
      const float v = acc[j >> 4][(j >> 2) & 3][j & 3];
      u32 t = (u32)(__builtin_bit_cast(int, v) >> 31);
      asm("" : "+v"(t));
      *(LAS float*)(size_t)(((t & skip) | (~t & p)) + 4 * j) = v;
      p += t << 2;
      asm("" : "+v"(p));
    }
  }
  asm volatile("" ::: "memory");
  if (!hold && CFG_MZ) mask();
  const char* dst = pstg;
  const u32x2 tg = tags((call + 1) >> 1, early);
  if (early) {
    u32x4 c[4];
#pragma unroll
    for (int i = 0; i < 4; i++) {
      const LAS f32x4* p = (const LAS f32x4*)(size_t)(zb + i * 2048 + lane * 32);
      c[i] = (pk8(p[0], p[1]) & 0x7fff7fffu) | u32x4{tg[0], tg[1], tg[0], tg[1]};
    }
    if (CFG_EDLY) { const u64 t1 = now(); while (now() - t1 < (u32)(CFG_EOFF + CFG_EDLY * (pos * 4 + wave) / (CFG_NE * 4))) __builtin_amdgcn_s_sleep(1); }
    if (!bad) {
      if (lane * 8 < total) st16_so<0>(dst, lane * 16, c[0]);
      if (lane * 8 + 512 < total) st16_so<0>(dst + cell_off(64), lane * 16, c[1]);
      if (lane * 8 + 1024 < total) st16_so<0>(dst + cell_off(128), lane * 16, c[2]);
      if (lane * 8 + 1536 < total) st16_so<0>(dst + cell_off(192), lane * 16, c[3]);
    } else if (lane == 0) st4_sys(tmo, call);
  } else {
    f32x4 y[2][4];
#pragma unroll
    for (int i = 0; i < 8; i++) y[i >> 2][i & 3] = *(LAS f32x4*)(size_t)(zb + (i >> 2) * 4096 + lane * 64 + (i & 3) * 16);
    if (hold) {
      while (__builtin_amdgcn_readfirstlane(lds_ld8(sm + 2)[1]) != call && now() - t0 < 2 * DEADLINE) __builtin_amdgcn_s_sleep(1);
      mask();
    }
    const u32 gm = get_gmax(call, t0, bad, [&]() {});
    if (gm != 0xffffffffu) {
      const float inv = inv_scale(gm);
      auto q16 = [&](const f32x4* v) {
        return (u32x4{quant4(v[0], inv), quant4(v[1], inv), quant4(v[2], inv), quant4(v[3], inv)} & 0x7f7f7f7fu) | tg[0];
      };
      if (CFG_LD) {
        const u64 t1 = now();
        u32x4 c0 = q16(y[0]), c1 = q16(y[1]);
        asm volatile("" : "+v"(c0), "+v"(c1));
        while (now() - t1 < (u32)(CFG_LD * ((pos - CFG_NE) * 4 + wave) / ((32 - CFG_NE) * 4))) {}
        if (lane * 16 < total) st16_so<0>(dst, lane * 16, c0);
        if (lane * 16 + 1024 < total) st16_so<0>(dst + cell_off(64), lane * 16, c1);
      } else {
        if (lane * 16 < total) st16_so<0>(dst, lane * 16, q16(y[0]));
        if (lane * 16 + 1024 < total) st16_so<0>(dst + cell_off(64), lane * 16, q16(y[1]));
      }
    }
  }
  if (wave == 0 && lane == 63) st4_sys(ctr, call);
}

static bool launch_setup() {
  hipError_t e = hipFuncSetAttribute((const void*)gemm_a2a, hipFuncAttributeMaxDynamicSharedMemorySize, LDS_BYTES);
  if (e != hipSuccess) { fprintf(stderr, "HIP error %s\n", hipGetErrorString(e)); return false; }
  return true;
}
static void launch_impl(int rank, hipStream_t stream, const void* x, const void* w, char* const* out_dev, char* const* ws_dev,
                        char* const* out_host, char* const* ws_host) {
  gemm_a2a<<<256, 512, LDS_BYTES, stream>>>((const char*)x, (const char*)w, out_dev, ws_dev, ws_host[rank], out_host[rank], rank);
}
extern "C" void kernel_launch(int rank, int nranks, ncclComm_t comm, hipStream_t stream, void* const* d_in, const int* in_sizes, int n_in,
                              void* d_out, int out_size, const int* in_dtypes, int out_dtype, void* const* d_out_all,
                              void* const* d_ws_all, size_t ws_bytes) {
  if (nranks != NR || (unsigned)rank >= (unsigned)NR || n_in < 2 || in_sizes[0] != MS * KD || in_sizes[1] != KD * ND ||
      out_size != NR * MS * NCOL || ws_bytes < WS_END) {
    fprintf(stderr, "unexpected problem shape\n");
    return;
  }
  if (!launch_setup()) return;
  launch_impl(rank, stream, d_in[0], d_in[1], (char* const*)d_out_all, (char* const*)d_ws_all, (char* const*)d_out_all, (char* const*)d_ws_all);
}
```

```cpp
#include <hip/hip_runtime.h>
#include <rccl/rccl.h>
#include <cstdio>

typedef unsigned u32;
typedef unsigned long long u64;
typedef u32 u32x4 __attribute__((ext_vector_type(4)));
typedef u32 u32x2 __attribute__((ext_vector_type(2)));
typedef float f32x4 __attribute__((ext_vector_type(4)));
typedef __bf16 bf16x4 __attribute__((ext_vector_type(4)));
typedef __bf16 bf16x8 __attribute__((ext_vector_type(8)));
typedef __bf16 bf16x2 __attribute__((ext_vector_type(2)));
#define LAS __attribute__((address_space(3)))

#define CFG_PSLP 2
#define CFG_PDLY 220
#define CFG_PH 2
#define CFG_PX 0
#define CFG_KSF 0
#define CFG_REP 8
#define CFG_NE 10
#define CFG_EDLY 180
#define CFG_EOFF 0
#define CFG_AM 1
#define CFG_MADD 1
#define CFG_M4 1
#define CFG_EX 16
#define CFG_D 4
#define CFG_XD 1
#define CFG_NW 3
#define CFG_ASLP 0
#define CFG_GSLP 1
#define CFG_GDLY 200
#define CFG_MZ 0
#define CFG_GSTR 8
#define CFG_LSTR 8
#define CFG_AVM 0
#define CFG_GVM 0
#define CFG_ADLY 0
#define CFG_GH 1
#define CFG_EP 0
#define CFG_RV 30
#define CFG_SR 0
#define CFG_LD 130
#define CFG_RD 100
#define CFG_ESLP 2
#define CFG_LF 1
#define CFG_PRE 3
#define CFG_XS 0x2211
#define CFG_XP 0
#define CFG_XDLY 0
#define CFG_XSLP 1
#define CFG_DM 2
constexpr int NAGG = 8, REP = CFG_REP, DEF = CFG_D;
constexpr int NR = 8, MS = 512, KD = 4096, ND = 4096, NCOL = 512;
constexpr int NSTEP = KD / 2 / 64, NS = 4, BLK = 16384, NBAR = NSTEP + DEF, XSTR = 6144;
constexpr int XRING = 0, WRING = NS * BLK, SMALL = 2 * NS * BLK, LDS_BYTES = SMALL + 256;
constexpr u64 DEADLINE = 100000000ull;

constexpr size_t WS_CTR = 0;
constexpr size_t WS_TMO = 32768 + 4;
constexpr size_t WS_LMAX = 98304;
constexpr size_t WS_GMAX = 524288;
constexpr size_t WS_TS = 786432;
constexpr size_t WS_EX = 1u << 20;
constexpr size_t WS_STG = WS_EX + (size_t)1024 * XSTR;
#define CFG_LAY 0
constexpr u32 PLANE = 1u << 20, SSTR = CFG_LAY == 0 ? 1024 : CFG_LAY == 1 ? 2048 : 4096;
__device__ __forceinline__ u32 cell_off(u32 c) { return CFG_LAY == 0 ? (c >> 6) * PLANE + (c & 63) * 16 : CFG_LAY == 1 ? (c >> 7) * 2 * PLANE + (c & 127) * 16 : c * 16; }
constexpr size_t WS_MSK = WS_STG + (size_t)2 * 1024 * 4096;
constexpr size_t WS_END = WS_MSK + (size_t)2 * 1024 * 512;

template <int N> __device__ __forceinline__ void wait_vm() { asm volatile("s_waitcnt vmcnt(%0)" ::"i"(N) : "memory"); }
constexpr int nld(int s) { return s >= NSTEP ? 0 : CFG_XD && s >= NSTEP - DEF ? 12 : 16; }
constexpr int xat(int i) { return NSTEP - NS + (CFG_XS >> 4 * i & 15); }
constexpr int left(int j) {
  int n = nld(j + 1) + nld(j + 2);
  for (int i = 0; i < DEF; i++) if (CFG_XD && xat(i) < j && i > j - NSTEP) n += 4;
  return n;
}
template <int V> struct Num { static constexpr int v = V; };
template <int J, class F> __device__ __forceinline__ void tail(F&& f) { if constexpr (J < NBAR) { f(Num<J>{}); tail<J + 1>(f); } }
__device__ __forceinline__ void st16_sys(void* p, u32x4 v) { asm volatile("global_store_dwordx4 %0, %1, off sc0 sc1\n\ts_nop 1" ::"v"(p), "v"(v) : "memory"); }
template <int O> __device__ __forceinline__ void st16_so(const char* base, u32 off, u32x4 v) {
  asm volatile("global_store_dwordx4 %0, %1, %2 offset:%3 sc0 sc1\n\ts_nop 1" ::"v"(off), "v"(v), "s"(base), "i"(O) : "memory");
}
#define CFG_XST "sc0 sc1"
#define CFG_XLD 17
template <int O> __device__ __forceinline__ void st16_x(const char* base, u32 off, u32x4 v) {
  asm volatile("global_store_dwordx4 %0, %1, %2 offset:%3 " CFG_XST "\n\ts_nop 1" ::"v"(off), "v"(v), "s"(base), "i"(O) : "memory");
}
__device__ __forceinline__ void st8_sys(void* p, u32x2 v) { asm volatile("global_store_dwordx2 %0, %1, off sc0 sc1\n\ts_nop 1" ::"v"(p), "v"(v) : "memory"); }
__device__ __forceinline__ void st4_sys(void* p, u32 v) { asm volatile("global_store_dword %0, %1, off sc0 sc1\n\ts_nop 1" ::"v"(p), "v"(v) : "memory"); }
__device__ __forceinline__ u32 ld4_sys(const void* p) { u32 v; asm volatile("global_load_dword %0, %1, off sc0 sc1\n\ts_waitcnt vmcnt(0)" : "=&v"(v) : "v"(p) : "memory"); return v; }
__device__ __forceinline__ u32x2 ld8_sys(const void* p) { u32x2 v; asm volatile("global_load_dwordx2 %0, %1, off sc0 sc1\n\ts_waitcnt vmcnt(0)" : "=&v"(v) : "v"(p) : "memory"); return v; }
__device__ __forceinline__ u64 now() { return __builtin_amdgcn_s_memrealtime(); }
__device__ __forceinline__ u32 vm_peek() { u32 v; asm volatile("s_getreg_b32 %0, hwreg(HW_REG_IB_STS, 0, 4)" : "=s"(v) :: "memory"); return v; }
__device__ __forceinline__ char* uni(char* p) {
  const u64 v = (u64)p;
  return (char*)(((u64)(u32)__builtin_amdgcn_readfirstlane((u32)(v >> 32)) << 32) | (u32)__builtin_amdgcn_readfirstlane((u32)v));
}
__device__ __forceinline__ u32 clk() { return (u32)__builtin_amdgcn_s_memtime(); }
__device__ __forceinline__ u32 pk2(float a, float b) { return __builtin_bit_cast(u32, bf16x2{(__bf16)a, (__bf16)b}); }
__device__ __forceinline__ u32x4 pk8(f32x4 a, f32x4 c) { return u32x4{pk2(a[0], a[1]), pk2(a[2], a[3]), pk2(c[0], c[1]), pk2(c[2], c[3])}; }
__device__ __forceinline__ float lo16(u32 v) { return __builtin_bit_cast(float, v << 16); }
__device__ __forceinline__ float hi16(u32 v) { return __builtin_bit_cast(float, v & 0xffff0000u); }

__device__ __forceinline__ u32 wave_max(u32 v) {
  v = max(v, (u32)__builtin_amdgcn_update_dpp(0, (int)v, 0xb1, 0xf, 0xf, true));
  v = max(v, (u32)__builtin_amdgcn_update_dpp(0, (int)v, 0x4e, 0xf, 0xf, true));
  v = max(v, (u32)__builtin_amdgcn_update_dpp(0, (int)v, 0x141, 0xf, 0xf, true));
  v = max(v, (u32)__builtin_amdgcn_update_dpp(0, (int)v, 0x140, 0xf, 0xf, true));
  return max(max((u32)__builtin_amdgcn_readlane(v, 0), (u32)__builtin_amdgcn_readlane(v, 16)),
             max((u32)__builtin_amdgcn_readlane(v, 32), (u32)__builtin_amdgcn_readlane(v, 48)));
}
template <int SLP, int VM> __device__ __forceinline__ u32 poll(const char* p, u32 call, u64 t0, bool& bad, u32x2& r) {
  u32 v, late;
  u64 ex;
  asm volatile("s_mov_b64 %2, exec\n\tglobal_load_dwordx2 v[100:101], %4, off sc0 sc1\n\ts_waitcnt vmcnt(0)\n\ts_cmp_eq_u32 0, 1\n\ts_branch 3f\n"
               "1:\n\ts_sleep %8\n\tglobal_load_dwordx2 v[100:101], %4, off sc0 sc1\n\t"
               "s_memrealtime s[60:61]\n\ts_waitcnt vmcnt(%9) lgkmcnt(0)\n\t"
               "s_sub_u32 %1, s60, %6\n\ts_cmp_gt_u32 %1, %7\n\ts_cbranch_scc1 2f\n\t"
               "3:\n\tv_xor_b32 %0, v100, v101\n\tv_cmpx_ne_u32 vcc, %5, %0\n\ts_cbranch_execnz 1b\n"
               "2:\n\ts_cselect_b32 %1, 1, 0\n\ts_mov_b64 exec, %2\n\tv_mov_b32 %0, v100\n\ts_nop 1"
               : "=&v"(v), "=&s"(late), "=&s"(ex), "=&{v[100:101]}"(r)
               : "v"(p), "s"(call), "s"((u32)t0), "s"((u32)DEADLINE), "i"(SLP), "i"(VM) : "memory", "vcc", "scc", "s60", "s61");
  bad |= __builtin_amdgcn_readfirstlane(late) != 0;
  return v;
}
__device__ __forceinline__ void settle(u32x2& r) { asm volatile("s_waitcnt vmcnt(0)" : "+{v[100:101]}"(r) :: "memory"); }
__device__ __forceinline__ void wg_max(u32 wmx, LAS u32* sm, char* ws, int b, int lane, u32 call) {
  if (lane == 0) {
    u32 tk, m4;
    asm volatile("ds_max_u32 %2, %3\n\tds_add_rtn_u32 %0, %2, %4 offset:4\n\tds_read_b32 %1, %2\n\ts_waitcnt lgkmcnt(0)"
                 : "=&v"(tk), "=&v"(m4) : "v"((u32)(size_t)sm), "v"(wmx), "v"(1u) : "memory");
    if (tk == 3) st8_sys(ws + WS_LMAX + (b & 7) * 4096 + (b >> 3) * CFG_LSTR, u32x2{m4, m4 ^ call});
  }
}
__device__ __forceinline__ u32 wait_gmax(char* ws, int b, int lane, u32 call, u64 t0, bool bad) {
  const char* pg = ws + WS_GMAX + (size_t)((call & 1) * REP + b % REP) * 8192 + lane * CFG_GSTR;
  u32 gm;
  while (true) {
    const u32x2 r = ld8_sys(pg);
    gm = r[0];
    if (__all((r[0] ^ r[1]) == call)) break;
    if (now() - t0 > DEADLINE) { bad = true; break; }
    __builtin_amdgcn_s_sleep(1);
  }
  if (bad && lane == 0) st4_sys(ws + WS_TMO, call);
  return bad ? 0xffffffffu : wave_max(gm);
}
__device__ __forceinline__ float inv_scale(u32 gm) {
  const float scale = __builtin_bit_cast(float, gm) / 127.0f;
  return scale > 0.f ? 1.0f / scale : 0.f;
}
__device__ __forceinline__ u32 quant4(f32x4 y, float inv) {
  u32 q[4];
#pragma unroll
  for (int e = 0; e < 4; e++) q[e] = __builtin_bit_cast(u32, fminf(__builtin_fmaf(y[e], inv, 8388608.f), 8388735.f));
  return __builtin_amdgcn_perm(__builtin_amdgcn_perm(q[3], q[2], 0x0c0c0400u), __builtin_amdgcn_perm(q[1], q[0], 0x0c0c0400u), 0x05040100u);
}
__device__ __forceinline__ u32 spread4(u32 nb) { return __builtin_amdgcn_ubfe(nb, 0, 4) * 0x204081u & 0x01010101u; }
__device__ __forceinline__ u32x2 tags(u32 use, bool wide) {
  const u32 t = spread4(use) << 7;
  return wide ? u32x2{(use & 1) << 15 | (use & 2) << 30, (use & 4) << 13 | (use & 8) << 28} : u32x2{t, t};
}
__device__ __forceinline__ u32 wave_scan(u32 v) {
  v += (u32)__builtin_amdgcn_update_dpp(0, (int)v, 0x111, 0xf, 0xf, false);
  v += (u32)__builtin_amdgcn_update_dpp(0, (int)v, 0x112, 0xf, 0xf, false);
  v += (u32)__builtin_amdgcn_update_dpp(0, (int)v, 0x114, 0xf, 0xf, false);
  v += (u32)__builtin_amdgcn_update_dpp(0, (int)v, 0x118, 0xf, 0xf, false);
  v += (u32)__builtin_amdgcn_update_dpp(0, (int)v, 0x142, 0xa, 0xf, false);
  v += (u32)__builtin_amdgcn_update_dpp(0, (int)v, 0x143, 0xc, 0xf, false);
  return v;
}

template <int STR> __device__ __forceinline__ void ld64_sys(u32x4* r, const char* base, u32 off) {
  asm volatile("global_load_dwordx4 %0, %4, %5 sc0 sc1\n\tglobal_load_dwordx4 %1, %4, %6 sc0 sc1\n\t"
               "global_load_dwordx4 %2, %4, %7 sc0 sc1\n\tglobal_load_dwordx4 %3, %4, %8 sc0 sc1\n\ts_waitcnt vmcnt(0)"
               : "=&v"(r[0]), "=&v"(r[1]), "=&v"(r[2]), "=&v"(r[3])
               : "v"(off), "s"(base), "s"(base + STR), "s"(base + 2 * STR), "s"(base + 3 * STR) : "memory");
}
__device__ __forceinline__ u32x2 lds_ld8(LAS u32* p) { u32x2 v; asm volatile("ds_read_b64 %0, %1\n\ts_waitcnt lgkmcnt(0)" : "=&v"(v) : "v"((u32)(size_t)p) : "memory"); return v; }

struct Peers { char* ws[NR]; };
__global__ __launch_bounds__(512, 2) void gemm_a2a(const char* __restrict__ x, const char* __restrict__ w, char* ws, char* out,
                                                   char* const* ws_all, int rank, Peers peers) {
  extern __shared__ __attribute__((aligned(16))) char smem[];
  if (CFG_LF) asm volatile("" : "+s"(x), "+s"(w), "+s"(ws), "+s"(out));
  const int b = blockIdx.x, lane = threadIdx.x & 63;
  const int wave = __builtin_amdgcn_readfirstlane(threadIdx.x >> 6);
  const int xcd = b & 7, idx = b >> 3;
  const int id = idx ^ CFG_EX;
  const int ks = (xcd ^ CFG_KSF) & 1, mt = (id >> 1) & 3, ntl = (id & 1) * 4 + (id >> 3), nt = (xcd >> 1) * 8 + ntl;
  const int g = lane >> 4, li = lane & 15;
  const int rid = id ^ CFG_RV, src = (xcd + CFG_SR * (id >> 1)) & 7, pos = ((id ^ CFG_EP) & ~1) + ks, rpos = rid ^ CFG_EP;
  const int dest = nt >> 2, item = rank * 32 + (id & ~1) + ks, mitem = src * 32 + rid;
  const bool aggr = (id ^ CFG_EP) == 31;
  char* const mine = out + (size_t)(src * MS + (rid & 7) * 64) * NCOL + (rid >> 3) * 128;
  LAS u32* sm = (LAS u32*)(smem + SMALL);
  u32* tmo = (u32*)(ws + WS_TMO);
  auto get_gmax = [&](u32 call, u64 t0, bool bad, auto&& idle) {
    u32 gm;
    while (true) {
      const u32x2 v = lds_ld8(sm + 6);
      gm = __builtin_amdgcn_readfirstlane(v[0]);
      if (__builtin_amdgcn_readfirstlane(v[1]) == call) break;
      if (now() - t0 > 2 * DEADLINE) { bad = true; break; }
      idle();
    }
    if (bad && lane == 0) st4_sys(tmo, call);
    return bad ? 0xffffffffu : gm;
  };
  auto recv = [&](int h, u32 call, u64 t0) {
    const bool early = rpos < CFG_NE;
    const u32 gm = get_gmax(call, t0, false, [&]() { __builtin_amdgcn_s_sleep(2); });
    bool bad = gm == 0xffffffffu;
    t0 = now();
    const char* pm = ws + WS_MSK + (size_t)(((call & 1) * 256 + mitem) * 4 + h) * 512 + (CFG_M4 ? (lane & 31) * 16 + (lane >> 5) * 8 : lane * 8);
    u32 km = 0;
    while (!bad) {
      const u32x2 r = ld8_sys(pm);
      km = ~r[0];
      if (__all(r[1] >= call)) break;
      if (now() - t0 > DEADLINE) bad = true;
      __builtin_amdgcn_s_sleep(2);
    }
    const u32 cnt = __builtin_popcount(km), inc = wave_scan(cnt);
    const int len = early ? (__builtin_amdgcn_readlane(inc, 63) + 7) >> 3 : (__builtin_amdgcn_readlane(inc, 63) + 15) >> 4;
    const u32 lb = (u32)(size_t)smem + WRING + h * 4096;
    u32 ad[8], sel[8], keep[8];
#pragma unroll
    for (int k = 0; k < 8; k++) {
      const u32 sh = 4 * (k ^ (g & 1));
      const u32 o = inc - cnt + __builtin_popcount(km & ((1u << sh) - 1)), sp = spread4(km >> sh);
      ad[k] = lb + (o & ~3u);
      sel[k] = (sp << 8) + (sp << 16) + (sp << 24) + (o & 3) * 0x01010101u;
      keep[k] = sp * 127;
      asm volatile("" : "+v"(ad[k]), "+v"(sel[k]), "+v"(keep[k]));
    }
    const char* stg = ws + WS_STG + (size_t)(call & 1) * 4 * PLANE + (mitem * 4 + h) * SSTR;
    const u32 use = (call + 1) >> 1;
    const u32x2 tg = tags(use, early);
    const u32x4 tag = u32x4{tg[0], tg[1], tg[0], tg[1]};
    auto cl = [&](int i) { return (u32)max(min(lane + 64 * i, len - 1), 0); };
    auto off = [&](int i) { return cell_off(cl(i)); };
    if (early) {
      u32x4 r[4];
      while (!bad && len > 0) {
        asm volatile("global_load_dwordx4 %0, %4, %8 sc0 sc1\n\tglobal_load_dwordx4 %1, %5, %8 sc0 sc1\n\t"
                     "global_load_dwordx4 %2, %6, %8 sc0 sc1\n\tglobal_load_dwordx4 %3, %7, %8 sc0 sc1\n\ts_waitcnt vmcnt(0)"
                     : "=&v"(r[0]), "=&v"(r[1]), "=&v"(r[2]), "=&v"(r[3]) : "v"(off(0)), "v"(off(1)), "v"(off(2)), "v"(off(3)), "s"(stg) : "memory");
        const u32x4 e = (r[0] ^ tag) | (r[1] ^ tag) | (r[2] ^ tag) | (r[3] ^ tag);
        if (__all(((e[0] | e[1] | e[2] | e[3]) & 0x80008000u) == 0)) break;
        if (now() - t0 > DEADLINE) bad = true;
        __builtin_amdgcn_s_sleep(CFG_ESLP);
      }
      const float inv = inv_scale(gm);
#pragma unroll
      for (int i = 0; i < 4; i++) {
        if (i * 64 >= len) break;
        f32x4 y[2];
#pragma unroll
        for (int e = 0; e < 4; e++) {
          const u32 v = r[i][e];
          y[e >> 1][2 * (e & 1)] = __builtin_fabsf(lo16(v));
          y[e >> 1][2 * (e & 1) + 1] = __builtin_bit_cast(float, v & 0x7fff0000u);
        }
        *(LAS u32x2*)(size_t)(lb + cl(i) * 8) = u32x2{quant4(y[0], inv), quant4(y[1], inv)};
      }
    } else {
      const u32 dly = CFG_PDLY - CFG_RD * ((32 - rpos) * 4 - 1 - h) / ((32 - CFG_NE) * 4);
      while (now() - t0 < dly) __builtin_amdgcn_s_sleep(1);
      u32x4 r0, r1;
      while (!bad && len > 0) {
        asm volatile("global_load_dwordx4 %0, %2, %4 sc0 sc1\n\tglobal_load_dwordx4 %1, %3, %4 sc0 sc1\n\ts_waitcnt vmcnt(0)"
                     : "=&v"(r0), "=&v"(r1) : "v"(off(0)), "v"(off(1)), "s"(stg) : "memory");
        const u32x4 e = (r0 ^ tag) | (r1 ^ tag);
        if (__all(((e[0] | e[1] | e[2] | e[3]) & 0x80808080u) == 0)) break;
        if (now() - t0 > DEADLINE) bad = true;
        __builtin_amdgcn_s_sleep(CFG_PSLP);
      }
      *(LAS u32x4*)(size_t)(lb + cl(0) * 16) = r0;
      *(LAS u32x4*)(size_t)(lb + cl(1) * 16) = r1;
    }
    asm volatile("" ::: "memory");
    u32 d[8];
#pragma unroll
    for (int k = 0; k < 8; k++) {
      const LAS u32* q = (const LAS u32*)(size_t)ad[k];
      d[k] = __builtin_amdgcn_perm(q[1], q[0], sel[k]) & keep[k];
    }
    if (bad) {
#pragma unroll
      for (int k = 0; k < 8; k++) { d[k] = 0x80808080u; asm volatile("" : "+v"(d[k])); }
    }
    const char* op = mine + (h >> 1) * 32 * NCOL + (h & 1) * 64;
#pragma unroll
    for (int m = 0; m < 2; m++) {
      const u32x4 v = u32x4{d[4 * m], d[4 * m + 1], d[4 * m + 2], d[4 * m + 3]};
      st16_so<0>(op + m * 16 * NCOL, li * NCOL + g * 16, v);
    }
    const int cell = (lane >> 3) * 64 + (use & 7) * 8 + (lane & 7);
    if (lane < (early ? 32 : 16) && cell >= len) st16_so<0>(stg, cell_off(cell), tag);
    if (bad && lane == 0) st4_sys(tmo, call);
  };
  if (__builtin_expect((wave & 6) == 4, CFG_LF)) {
    const int lw = wave - 4;
    const int kx = CFG_PH ? mt ^ (ntl >> 1) : (mt ^ ntl) & 3;
    const int ph = (ntl & 1 ? 0 : CFG_PH) + CFG_PX * (xcd >> 2);
    auto rx = __builtin_amdgcn_make_buffer_rsrc((void*)x, 0, MS * KD * 2, 0x00020000);
    auto rw = __builtin_amdgcn_make_buffer_rsrc((void*)w, 0, KD * ND * 2, 0x00020000);
    int vx[2], vw[2];
#pragma unroll
    for (int h = 0; h < 2; h++) {
      vx[h] = (lane >> 3) * 8192 + (((lane & 7) ^ ((((lane >> 3) + 8 * h) >> 1) & 7)) << 4);
      vw[h] = g * 8192 + ((li ^ g ^ (8 * h)) << 4);
    }
    const int sx0 = (mt * 128 + lw * 32) * 8192 + ks * KD;
    const int sw0 = (ks * (KD / 2) + lw * 32) * 8192 + nt * 256;
    auto spos = [&](int s) { return ((s + ph) & (NSTEP - 1)) ^ kx; };
    auto ix = [&](int s, int slot, int hf) {
      LAS char* l = (LAS char*)(smem + XRING + slot * BLK + hf * 8192 + lw * 4096);
      const int o = sx0 + (hf ^ ks) * 64 * 8192 + spos(s) * 128;
#pragma unroll
      for (int i = 0; i < 4; i++) __builtin_amdgcn_raw_ptr_buffer_load_lds(rx, l + i * 1024, 16, vx[i & 1], o + i * 8 * 8192, 0, 0);
    };
    auto issue = [&](int s, int slot, bool all) {
      if (all) ix(s, slot, 0);
      ix(s, slot, 1);
      LAS char* l = (LAS char*)(smem + WRING + slot * BLK + lw * 8192);
      const int sp = spos(s);
#pragma unroll
      for (int i = 0; i < 8; i++) __builtin_amdgcn_raw_ptr_buffer_load_lds(rw, l + i * 1024, 16, vw[(i >> 1) & 1], sw0 + (sp * 64 + i * 4) * 8192, 0, 0);
    };
#pragma unroll
    for (int i = 0; i < CFG_PRE; i++) issue(i, i, true);
    wait_vm<16 * (CFG_PRE - 1)>();
    __builtin_amdgcn_s_barrier();
#pragma unroll
    for (int i = CFG_PRE; i < NS; i++) issue(i, i, true);
#pragma unroll 1
    for (int j = 1; j <= NSTEP - 2 * NS; j++) {
      wait_vm<32>();
      __builtin_amdgcn_s_barrier();
      issue(j - 1 + NS, (j - 1) & 3, true);
    }
    tail<NSTEP - 2 * NS + 1>([&](auto jc) {
      constexpr int j = jc.v;
      wait_vm<left(j)>();
      __builtin_amdgcn_s_barrier();
      if (j <= NSTEP - NS) {
        int s = j - 1 + NS;
        asm volatile("" : "+s"(s));
        issue(s, s & 3, nld(j - 1 + NS) == 16);
      }
#pragma unroll
      for (int i = 0; i < DEF; i++) {
        if (!CFG_XD || xat(i) != j) continue;
        int s = NSTEP - DEF + i;
        asm volatile("" : "+s"(s));
        ix(s, s & 3, 0);
      }
    });
    const u32 call = __builtin_amdgcn_readfirstlane(sm[2]);
    const u64 t0 = now();
    bool bad = false;
    if (lw == 1 && aggr) {
      char* pw = ws_all[lane & 7];
      u32 gm = 0;
      u32x2 r;
      char* pd = pw + WS_GMAX + (size_t)((call & 1) * REP + (lane >> 3)) * 8192 + (rank * NAGG + xcd) * CFG_GSTR;
      if (CFG_ADLY) while (now() - t0 < CFG_ADLY) __builtin_amdgcn_s_sleep(2);
      asm volatile("" : "+v"(pd));
      if (CFG_NW & 1) gm = poll<CFG_ASLP, CFG_AVM>(ws + WS_LMAX + xcd * 4096 + (lane & 31) * CFG_LSTR, call, t0, bad, r);
      else {
        const char* pg = lane < 32 ? ws + WS_LMAX + xcd * 4096 + lane * CFG_LSTR : (const char*)tmo - 4;
        while (true) {
          r = ld8_sys(pg);
          gm = lane < 32 ? r[0] : 0;
          if (__all(lane >= 32 || (r[0] ^ r[1]) == call)) break;
          if (__builtin_amdgcn_readlane(r[1], 63) == call || now() - t0 > DEADLINE) { bad = true; break; }
        }
      }
      gm = bad ? 0xffffffffu : wave_max(gm);
      if (lane < NR * REP) st8_sys(pd, u32x2{gm, gm ^ call});
      if (lane == 0) sm[3] = call;
      if (CFG_NW & 1) settle(r);
    }
    recv(lw, call, t0);
    return;
  }
  if (wave >= 6) {
    {
#pragma unroll 1
      for (int nb = 0; nb < NBAR; nb++) __builtin_amdgcn_s_barrier();
    }
    const u32 call = __builtin_amdgcn_readfirstlane(sm[2]);
    const u64 t0 = now();
    if (wave == 6) {
      if (CFG_GDLY) while (now() - t0 < CFG_GDLY) __builtin_amdgcn_s_sleep(2);
      if (CFG_GH && aggr) while (__builtin_amdgcn_readfirstlane(lds_ld8(sm + 2)[1]) != call && now() - t0 < 2 * DEADLINE) __builtin_amdgcn_s_sleep(2);
      if (CFG_NW & 2) {
        bool bad = false;
        u32x2 r;
        u32 gm = poll<CFG_GSLP, CFG_GVM>(ws + WS_GMAX + (size_t)((call & 1) * REP + b % REP) * 8192 + lane * CFG_GSTR, call, t0, bad, r);
        gm = bad ? 0xffffffffu : wave_max(gm);
        if (lane == 0) *(LAS u32x2*)(sm + 6) = u32x2{gm, call};
        settle(r);
        if (bad && lane == 0) st4_sys(tmo, call);
      } else {
        const u32 gm = wait_gmax(ws, b, lane, call, t0, false);
        if (lane == 0) *(LAS u32x2*)(sm + 6) = u32x2{gm, call};
      }
    }
    recv(wave - 4, call, t0);
    return;
  }
  const int wm = wave >> 1, wn = wave & 1;
  u32* ctr = (u32*)(ws + WS_CTR + b * 128);
  const u32 call = *(const __attribute__((address_space(4))) u32*)(size_t)ctr + 1;
  char* const pw = peers.ws[dest & 7];
  const char* pmsk = pw + WS_MSK + (size_t)(((call & 1) * 256 + item) * 4 + wave) * 512;
  const char* pstg = pw + WS_STG + (size_t)(call & 1) * 4 * PLANE + (item * 4 + wave) * SSTR;
  const char* xdst = ws + WS_EX + (size_t)((b ^ 1) * 4 + wave) * XSTR + 2048;
  char* xsrc = ws + WS_EX + (size_t)(b * 4 + wave) * XSTR;
  u32 xtag = (0x2000u | (call & 0x1fffu)) << 16;
  u32 l16 = lane * 16, h16 = (lane & 31) * 16, h8 = h16 + (lane >> 5) * 8;
  asm volatile("" : "+v"(l16), "+v"(h16), "+v"(h8));
  u32x4 u0, u1, u2;
  {
    const u32 e = g == li >> 2 ? li & 3 : 4, one = 0x3f80u << (li & 1) * 16;
    u0 = u32x4{e < 2 ? one : 0, e == 2 ? 0x3f80u : 0, 0, e == 3 ? 0x3f80u : 0}; u1 = u32x4{0, 0, u0[0], 0}; u2 = u32x4{0, 0, (e & 6) == 2 ? one : 0, 0};
    asm volatile("" : "+v"(u0), "+v"(u1), "+v"(u2));
  }
  f32x4 acc[4][4];
#pragma unroll
  for (int i = 0; i < 16; i++) acc[i >> 2][i & 3] = f32x4{0.f, 0.f, 0.f, 0.f};
  u32 ax[2], aw[4];
#pragma unroll
  for (int kk = 0; kk < 2; kk++) ax[kk] = (u32)(size_t)smem + XRING + (32 * wm + li) * 128 + (((4 * kk + g) ^ ((li >> 1) & 7)) << 4);
  {
    const int q = li >> 2, p = li & 3;
#pragma unroll
    for (int n = 0; n < 4; n++)
      aw[n] = (u32)(size_t)smem + WRING + (8 * g + q) * 256 + (((8 * wn + 2 * p + (n >> 1)) ^ q ^ ((g & 1) << 3)) << 4) + (((n ^ p) & 1) << 3);
  }
#pragma unroll
  for (int n = 0; n < 4; n++) asm volatile("" : "+v"(aw[n]));
  bf16x8 fx[2][4], fw[2][4];
  auto rdw = [&](int buf, int n, u32 a) {
    const bf16x4 lo = __builtin_amdgcn_ds_read_tr16_b64_v4bf16((LAS bf16x4*)(size_t)a);
    const bf16x4 hi = __builtin_amdgcn_ds_read_tr16_b64_v4bf16((LAS bf16x4*)(size_t)(a + 1024));
    fw[buf][n] = __builtin_shufflevector(lo, hi, 0, 1, 2, 3, 4, 5, 6, 7);
  };
  auto rd = [&](int buf, int slot, int kk) {
    rdw(buf, 0, aw[0] + slot * BLK + kk * 8192);
#pragma unroll
    for (int m = 0; m < 4; m++) fx[buf][m] = *(LAS bf16x8*)(size_t)(ax[kk] + slot * BLK + ((m & 1) * 16 + (m >> 1) * 64) * 128);
#pragma unroll
    for (int n = 1; n < 4; n++) rdw(buf, n, aw[n] + slot * BLK + kk * 8192);
  };
  auto mm = [&](int buf) {
#pragma unroll
    for (int n = 0; n < 4; n++)
#pragma unroll
      for (int i = 0; i < 4; i++) {
        const int m = (n & 1) ? 3 - i : i;
        acc[m][n] = __builtin_amdgcn_mfma_f32_16x16x32_bf16(fw[buf][n], fx[buf][m], acc[m][n], 0, 0, 0);
      }
  };
  auto mix = [&]() {
#pragma unroll
    for (int i = 0; i < 12; i++) { __builtin_amdgcn_sched_group_barrier(0x008, 1, 0); __builtin_amdgcn_sched_group_barrier(0x100, 1, 0); }
    __builtin_amdgcn_sched_group_barrier(0x008, 4, 0);
    __builtin_amdgcn_sched_barrier(0);
  };
  auto sync = [&]() {
    __builtin_amdgcn_s_waitcnt(0xc07f);
    asm volatile("" ::: "memory");
    __builtin_amdgcn_s_barrier();
    __builtin_amdgcn_sched_barrier(0);
  };
  auto step = [&](int u, bool last) {
    __builtin_amdgcn_s_waitcnt(0xc67f);
    __builtin_amdgcn_sched_barrier(0);
    rd(1, u, 1); mm(0); mix();
    if (!last) { sync(); rd(0, (u + 1) % NS, 0); mm(1); mix(); }
    else mm(1);
  };
#pragma unroll
  for (int i = 0; i < 16; i++) asm volatile("" : "+v"(acc[i >> 2][i & 3]));
  asm volatile("" : "+s"(pmsk), "+s"(pstg), "+s"(xdst), "+s"(xsrc), "+s"(xtag));
  if (wave == 0 && lane < 8) sm[lane] = lane == 2 ? call : 0;
  sync();
  rd(0, 0, 0);
  __builtin_amdgcn_sched_barrier(0);
#pragma unroll 1
  for (int j0 = 0; j0 < NSTEP - NS; j0 += NS) {
#pragma unroll
    for (int u = 0; u < NS; u++) step(u, false);
  }
#pragma unroll
  for (int u = 0; u < NS - DEF; u++) step(u, u == NS - 1);
  auto frag = [&](int hf, int q) -> f32x4& { return acc[2 * hf + ((q ^ q >> 1) & 1)][q >> 1]; };
  auto dump = [&](int j) {
    const float tf = __builtin_bit_cast(float, xtag);
    if (j < 5) {
      const f32x4 f = frag(1, j + 1 + (j >> 1)), h = frag(1, 3 * (j >> 1));
      const u32x4 c = u32x4{pk2(f[0], f[1]), pk2(f[2], tf), pk2(h[2 * (j & 1)], h[2 * (j & 1) + 1]), pk2(f[3], tf)};
      if (j == 0) st16_x<-2048>(xdst, l16, c);
      if (j == 1) st16_x<-1024>(xdst, l16, c);
      if (j == 2) st16_x<0>(xdst, l16, c);
      if (j == 3) st16_x<1024>(xdst, l16, c);
      if (j == 4) st16_x<2048>(xdst, l16, c);
    } else {
      const f32x4 h = frag(1, 6);
      const u32 v = pk2(h[2], h[3]);
      const auto two = __builtin_amdgcn_permlane32_swap(v, v, false, false);
      st16_x<3072>(xdst, h16, u32x4{two[0], xtag, two[1], xtag});
    }
  };
  auto rs = __builtin_amdgcn_make_buffer_rsrc(xsrc, 0, XSTR, 0x00020000);
  u32x4 r[5];
  u32x2 r5;
  auto fetch = [&](int j) {
    if (j < 5) r[j] = __builtin_amdgcn_raw_buffer_load_b128(rs, l16 + (j & 3) * 1024, (j >> 2) * 4096, CFG_XLD);
    else r5 = __builtin_amdgcn_raw_buffer_load_b64(rs, h8 + 1024, 4096, CFG_XLD);
  };
  if (DEF) {
    bf16x4 wl[4];
#pragma unroll
    for (int t = 0; t < 4 * DEF; t++) {
      const int hf = t < 2 * DEF, n = t + 1, nh = n < 2 * DEF, slot = (NSTEP - DEF + (n % (2 * DEF)) / 2) & 3, o = slot * BLK + (n & 1) * 8192;
      if ((t & 1) && n < 4 * DEF) {
        if (n == 4 * DEF - 2) __builtin_amdgcn_s_waitcnt(0xc07f);
        asm volatile("" ::: "memory");
        __builtin_amdgcn_s_barrier();
        asm volatile("" ::: "memory");
        __builtin_amdgcn_sched_barrier(0);
      }
      auto lo = [&](int i) { wl[i] = __builtin_amdgcn_ds_read_tr16_b64_v4bf16((LAS bf16x4*)(size_t)(aw[i] + o)); };
      auto hi = [&](int i) {
        const bf16x4 h = __builtin_amdgcn_ds_read_tr16_b64_v4bf16((LAS bf16x4*)(size_t)(aw[i] + o + 1024));
        fw[n & 1][i] = __builtin_shufflevector(wl[i], h, 0, 1, 2, 3, 4, 5, 6, 7);
      };
      auto xr = [&](int m) { fx[n & 1][2 * nh + m] = *(LAS bf16x8*)(size_t)(ax[n & 1] + slot * BLK + (m * 16 + nh * 64) * 128); };
      if (CFG_DM == 0 && t == 2 * DEF) {
#pragma unroll
        for (int j = 0; j < 6; j++) dump(j);
        __builtin_amdgcn_sched_barrier(0);
      }
#pragma unroll
      for (int q = 0; q < 8; q++) {
        frag(hf, q) = __builtin_amdgcn_mfma_f32_16x16x32_bf16(fw[t & 1][q >> 1], fx[t & 1][2 * hf + ((q ^ q >> 1) & 1)], frag(hf, q), 0, 0, 0);
        __builtin_amdgcn_sched_barrier(0);
        if (n < 4 * DEF) {
          if (q == 0) { lo(0); hi(0); }
          if (q == 1) xr(0);
          if (q == 2) xr(1);
          if (q == 3) { lo(1); hi(1); }
          if (q == 4) lo(2);
          if (q == 5) hi(2);
          if (q == 6) lo(3);
          if (q == 7) hi(3);
        }
        const int z = (t - 2 * DEF) * 8 + q;
        if (CFG_DM == 1) {
          if (z == -4) dump(0);
          if (z == -2) dump(1);
          if (z == -1) dump(2);
          if (z == 1) dump(3);
          if (z == 2) dump(4);
          if (z == 4) dump(5);
        }
        if (CFG_DM >= 3 && z >= -4 && (z + 4) % (CFG_DM - 1) == 0 && (z + 4) / (CFG_DM - 1) < 6) dump((z + 4) / (CFG_DM - 1));
        if (CFG_DM == 2) {
          if (z == -4) dump(0);
          if (z == -3) dump(1);
          if (z == -2) dump(2);
          if (z == -1) dump(3);
          if (z == 0) dump(5);
          if (z == 1) dump(4);
        }
        if (t == 4 * DEF - CFG_XP && q >= 1 && q < 7) fetch(q - 1);
        __builtin_amdgcn_sched_barrier(0);
      }
    }
  }
  bool bad = false;
  const u64 t0 = now();
  {
    if (!DEF) {
#pragma unroll
      for (int j = 0; j < 6; j++) dump(j);
      __builtin_amdgcn_sched_barrier(0);
    }
    if (!DEF || !CFG_XP) {
      if (CFG_XDLY) while (now() - t0 < CFG_XDLY) {}
#pragma unroll
      for (int j = 0; j < 6; j++) fetch(j);
    }
    while (true) {
      u32 d = r5[1] ^ xtag;
#pragma unroll
      for (int j = 0; j < 5; j++) d |= (r[j][1] ^ xtag) | (r[j][3] ^ xtag);
      if (__all(d < 65536)) break;
      if (now() - t0 > DEADLINE) { bad = true; break; }
      if (CFG_XSLP) __builtin_amdgcn_s_sleep(CFG_XSLP);
      asm volatile("" ::: "memory");
#pragma unroll
      for (int j = 0; j < 6; j++) fetch(j);
    }
    frag(0, 6)[2] += lo16(r5[0]);
    frag(0, 6)[3] += hi16(r5[0]);
#pragma unroll
    for (int j = 0; j < 5; j++) {
      const bf16x8 v = __builtin_bit_cast(bf16x8, r[j]);
      f32x4 &f = frag(0, j + 1 + (j >> 1)), &h = frag(0, 3 * (j >> 1));
      h = __builtin_amdgcn_mfma_f32_16x16x32_bf16(__builtin_bit_cast(bf16x8, j & 1 ? u2 : u1), v, h, 0, 0, 0);
      f = __builtin_amdgcn_mfma_f32_16x16x32_bf16(__builtin_bit_cast(bf16x8, u0), v, f, 0, 0, 0);
    }
  }
  float mx = 0.f;
#pragma unroll
  for (int i = 0; i < 8; i++)
#pragma unroll
    for (int e = 0; e < 4; e++) mx = fmaxf(mx, acc[i >> 2][i & 3][e]);
  wg_max(bad ? 0xffffffffu : wave_max(__builtin_bit_cast(u32, mx)), sm, ws, b, lane, call);
  __builtin_amdgcn_sched_barrier(0);
  u32 neg = 0;
#pragma unroll
  for (int j = 31; j >= 0; j--) {
    const float v = acc[j >> 4][(j >> 2) & 3][j & 3];
    neg = __builtin_amdgcn_alignbit(neg, __builtin_bit_cast(u32, v), 31);
  }
  auto mask = [&]() {
    if (CFG_M4) {
      const auto two = __builtin_amdgcn_permlane32_swap(neg, neg, false, false);
      if (lane < 32) st16_so<0>(pmsk, lane * 16, u32x4{two[0], call, two[1], call});
    } else st8_sys((char*)pmsk + lane * 8, u32x2{neg, call});
  };
  const bool hold = CFG_AM && aggr, early = pos < CFG_NE;
  if (!hold && !CFG_MZ) mask();
  const u32 cnt = __builtin_popcount(~neg), inc = wave_scan(cnt);
  const int total = __builtin_amdgcn_readlane(inc, 63);
  const u32 zb = (u32)(size_t)smem + XRING + 512 + wave * 8704;
  {
    u32 p = zb + (inc - cnt) * 4;
    const u32 skip = zb - 512 + lane * 4;
#pragma unroll
    for (int j = 0; j < 32; j++) {
      const float v = acc[j >> 4][(j >> 2) & 3][j & 3];
      u32 t = (u32)(__builtin_bit_cast(int, v) >> 31);
      asm("" : "+v"(t));
      *(LAS float*)(size_t)(((t & skip) | (~t & p)) + 4 * j) = v;
      p += t << 2;
      asm("" : "+v"(p));
    }
  }
  asm volatile("" ::: "memory");
  if (!hold && CFG_MZ) mask();
  const char* dst = pstg;
  const u32x2 tg = tags((call + 1) >> 1, early);
  if (early) {
    u32x4 c[4];
#pragma unroll
    for (int i = 0; i < 4; i++) {
      const LAS f32x4* p = (const LAS f32x4*)(size_t)(zb + i * 2048 + lane * 32);
      c[i] = (pk8(p[0], p[1]) & 0x7fff7fffu) | u32x4{tg[0], tg[1], tg[0], tg[1]};
    }
    if (CFG_EDLY) { const u64 t1 = now(); while (now() - t1 < (u32)(CFG_EOFF + CFG_EDLY * (pos * 4 + wave) / (CFG_NE * 4))) __builtin_amdgcn_s_sleep(1); }
    if (!bad) {
      if (lane * 8 < total) st16_so<0>(dst, lane * 16, c[0]);
      if (lane * 8 + 512 < total) st16_so<0>(dst + cell_off(64), lane * 16, c[1]);
      if (lane * 8 + 1024 < total) st16_so<0>(dst + cell_off(128), lane * 16, c[2]);
      if (lane * 8 + 1536 < total) st16_so<0>(dst + cell_off(192), lane * 16, c[3]);
    } else if (lane == 0) st4_sys(tmo, call);
  } else {
    f32x4 y[2][4];
#pragma unroll
    for (int i = 0; i < 8; i++) y[i >> 2][i & 3] = *(LAS f32x4*)(size_t)(zb + (i >> 2) * 4096 + lane * 64 + (i & 3) * 16);
    if (hold) {
      while (__builtin_amdgcn_readfirstlane(lds_ld8(sm + 2)[1]) != call && now() - t0 < 2 * DEADLINE) __builtin_amdgcn_s_sleep(1);
      mask();
    }
    const u32 gm = get_gmax(call, t0, bad, [&]() {});
    if (gm != 0xffffffffu) {
      const float inv = inv_scale(gm);
      auto q16 = [&](const f32x4* v) {
        return (u32x4{quant4(v[0], inv), quant4(v[1], inv), quant4(v[2], inv), quant4(v[3], inv)} & 0x7f7f7f7fu) | tg[0];
      };
      if (CFG_LD) {
        const u64 t1 = now();
        u32x4 c0 = q16(y[0]), c1 = q16(y[1]);
        asm volatile("" : "+v"(c0), "+v"(c1));
        while (now() - t1 < (u32)(CFG_LD * ((pos - CFG_NE) * 4 + wave) / ((32 - CFG_NE) * 4))) {}
        if (lane * 16 < total) st16_so<0>(dst, lane * 16, c0);
        if (lane * 16 + 1024 < total) st16_so<0>(dst + cell_off(64), lane * 16, c1);
      } else {
        if (lane * 16 < total) st16_so<0>(dst, lane * 16, q16(y[0]));
        if (lane * 16 + 1024 < total) st16_so<0>(dst + cell_off(64), lane * 16, q16(y[1]));
      }
    }
  }
  if (wave == 0 && lane == 63) st4_sys(ctr, call);
}

static bool launch_setup() {
  hipError_t e = hipFuncSetAttribute((const void*)gemm_a2a, hipFuncAttributeMaxDynamicSharedMemorySize, LDS_BYTES);
  if (e != hipSuccess) { fprintf(stderr, "HIP error %s\n", hipGetErrorString(e)); return false; }
  return true;
}
static void launch_impl(int rank, hipStream_t stream, const void* x, const void* w, char* const* out_dev, char* const* ws_dev,
                        char* const* out_host, char* const* ws_host) {
  Peers peers;
  for (int p = 0; p < NR; p++) peers.ws[p] = ws_host[p];
  gemm_a2a<<<256, 512, LDS_BYTES, stream>>>((const char*)x, (const char*)w, ws_host[rank], out_host[rank], ws_dev, rank, peers);
}
extern "C" void kernel_launch(int rank, int nranks, ncclComm_t comm, hipStream_t stream, void* const* d_in, const int* in_sizes, int n_in,
                              void* d_out, int out_size, const int* in_dtypes, int out_dtype, void* const* d_out_all,
                              void* const* d_ws_all, size_t ws_bytes) {
  if (nranks != NR || (unsigned)rank >= (unsigned)NR || n_in < 2 || in_sizes[0] != MS * KD || in_sizes[1] != KD * ND ||
      out_size != NR * MS * NCOL || ws_bytes < WS_END) {
    fprintf(stderr, "unexpected problem shape\n");
    return;
  }
  if (!launch_setup()) return;
  launch_impl(rank, stream, d_in[0], d_in[1], (char* const*)d_out_all, (char* const*)d_ws_all, (char* const*)d_out_all, (char* const*)d_ws_all);
}
```

```cpp
#define CFG_EB 1
#define CFG_MC 1
#define CFG_TC 35
#define CFG_TM 3
#define CFG_PDLY 258
#define CFG_JM 1
#define CFG_ND 0
#define CFG_NE 8
#define CFG_EDLY 140
#define CFG_LD 175
#define CFG_RD 135
#include <hip/hip_runtime.h>
#include <rccl/rccl.h>
#include <cstdio>

typedef unsigned u32;
typedef unsigned long long u64;
typedef u32 u32x4 __attribute__((ext_vector_type(4)));
typedef u32 u32x2 __attribute__((ext_vector_type(2)));
typedef float f32x4 __attribute__((ext_vector_type(4)));
typedef __bf16 bf16x4 __attribute__((ext_vector_type(4)));
typedef __bf16 bf16x8 __attribute__((ext_vector_type(8)));
typedef __bf16 bf16x2 __attribute__((ext_vector_type(2)));
#define LAS __attribute__((address_space(3)))

#define CFG_PSLP 2
#define CFG_PH 2
#define CFG_PX 0
#define CFG_KSF 0
#define CFG_REP 8
#define CFG_EOFF 0
#define CFG_AM 1
#define CFG_MADD 1
#define CFG_M4 1
#define CFG_EX 16
#define CFG_D 4
#define CFG_XD 1
#define CFG_NW 3
#define CFG_ASLP 0
#define CFG_GSLP 1
#define CFG_GDLY 200
#define CFG_MZ 0
#define CFG_GSTR 8
#define CFG_LSTR 8
#define CFG_AVM 0
#define CFG_GVM 0
#define CFG_ADLY 0
#define CFG_GH 1
#define CFG_EP 0
#define CFG_RV 30
#define CFG_SR 0
#define CFG_ESLP 2
#define CFG_LF 1
#define CFG_PRE 3
#define CFG_XS 0x2211
#define CFG_XP 0
#define CFG_XDLY 0
#define CFG_XSLP 1
#define CFG_DM 2
#define CFG_DW 200
#define CFG_DRW 200
#define CFG_DOFF 0
#define CFG_DSO 0
#define CFG_DSLP 2
#define CFG_DL 0
#define CFG_ERD 0
__host__ __device__ constexpr int rmap(int id) {
  return CFG_RV >= 0 ? id ^ CFG_RV : CFG_RV == -2 ? (id + CFG_NE) & 31 : id < CFG_NE ? 2 * CFG_NE - 1 - id : id < 32 - CFG_NE ? 31 + CFG_NE - id : id ^ 30;
}
#define HAVE_RMAP 1
#define CFG_RA 0
#define CFG_GK 1
#define CFG_GO 1664
#define CFG_GPH 0
#define CFG_AK 1
#define CFG_AO 896
#define CFG_AGAP 4
#define CFG_TW 10
#define CFG_TID 28
#define CFG_MSTR 512
__host__ __device__ constexpr int jobA(int id) {
  return id < CFG_NE ? rmap(id) : id >= 32 - CFG_NE ? (CFG_JM & 1 ? -1 : rmap(id)) : CFG_JM & 2 ? (id < 16 ? id : -1) : rmap(id);
}
__host__ __device__ constexpr int jobC(int id) {
  return id < CFG_NE ? (CFG_JM & 1 ? (CFG_JM & 4 ? CFG_NE - 1 - id : id) : -1) : CFG_JM & 2 && id < 16 ? id + 16 - CFG_NE : -1;
}
constexpr int P0 = 32 - CFG_ND;
constexpr int NAGG = 8, REP = CFG_REP, DEF = CFG_D;
constexpr int NR = 8, MS = 512, KD = 4096, ND = 4096, NCOL = 512;
constexpr int NSTEP = KD / 2 / 64, NS = 4, BLK = 16384, NBAR = NSTEP + DEF, XSTR = 6144;
constexpr int XRING = 0, WRING = NS * BLK, SMALL = 2 * NS * BLK, LDS_BYTES = SMALL + 256;
constexpr u64 DEADLINE = 100000000ull;

constexpr size_t WS_CTR = 0;
constexpr size_t WS_TMO = 32768 + 4;
constexpr size_t WS_LMAX = 98304;
constexpr size_t WS_TREF = 65536;
constexpr size_t WS_GMAX = 524288;
constexpr size_t WS_TS = 786432;
constexpr size_t WS_EX = 1u << 20;
constexpr size_t WS_STG = WS_EX + (size_t)1024 * XSTR;
#define CFG_LAY 0
constexpr u32 PLANE = 1u << 20, SSTR = CFG_LAY == 0 ? 1024 : CFG_LAY == 1 ? 2048 : 4096;
__device__ __forceinline__ u32 cell_off(u32 c) { return CFG_LAY == 0 ? (c >> 6) * PLANE + (c & 63) * 16 : CFG_LAY == 1 ? (c >> 7) * 2 * PLANE + (c & 127) * 16 : c * 16; }
constexpr size_t WS_MSK = WS_STG + (size_t)2 * 1024 * 4096;
constexpr size_t WS_END = WS_MSK + (size_t)2 * 1024 * 512;

template <int N> __device__ __forceinline__ void wait_vm() { asm volatile("s_waitcnt vmcnt(%0)" ::"i"(N) : "memory"); }
constexpr int nld(int s) { return s >= NSTEP ? 0 : CFG_XD && s >= NSTEP - DEF ? 12 : 16; }
constexpr int xat(int i) { return NSTEP - NS + (CFG_XS >> 4 * i & 15); }
constexpr int left(int j) {
  int n = nld(j + 1) + nld(j + 2);
  for (int i = 0; i < DEF; i++) if (CFG_XD && xat(i) < j && i > j - NSTEP) n += 4;
  return n;
}
template <int V> struct Num { static constexpr int v = V; };
template <int J, class F> __device__ __forceinline__ void tail(F&& f) { if constexpr (J < NBAR) { f(Num<J>{}); tail<J + 1>(f); } }
__device__ __forceinline__ void st16_sys(void* p, u32x4 v) { asm volatile("global_store_dwordx4 %0, %1, off sc0 sc1\n\ts_nop 1" ::"v"(p), "v"(v) : "memory"); }
template <int O> __device__ __forceinline__ void st16_so(const char* base, u32 off, u32x4 v) {
  asm volatile("global_store_dwordx4 %0, %1, %2 offset:%3 sc0 sc1\n\ts_nop 1" ::"v"(off), "v"(v), "s"(base), "i"(O) : "memory");
}
#define CFG_XST "sc0 sc1"
#define CFG_XLD 17
template <int O> __device__ __forceinline__ void st16_x(const char* base, u32 off, u32x4 v) {
  asm volatile("global_store_dwordx4 %0, %1, %2 offset:%3 " CFG_XST "\n\ts_nop 1" ::"v"(off), "v"(v), "s"(base), "i"(O) : "memory");
}
__device__ __forceinline__ void st8_sys(void* p, u32x2 v) { asm volatile("global_store_dwordx2 %0, %1, off sc0 sc1\n\ts_nop 1" ::"v"(p), "v"(v) : "memory"); }
__device__ __forceinline__ void st4_sys(void* p, u32 v) { asm volatile("global_store_dword %0, %1, off sc0 sc1\n\ts_nop 1" ::"v"(p), "v"(v) : "memory"); }
__device__ __forceinline__ u32 ld4_sys(const void* p) { u32 v; asm volatile("global_load_dword %0, %1, off sc0 sc1\n\ts_waitcnt vmcnt(0)" : "=&v"(v) : "v"(p) : "memory"); return v; }
__device__ __forceinline__ u32x2 ld8_sys(const void* p) { u32x2 v; asm volatile("global_load_dwordx2 %0, %1, off sc0 sc1\n\ts_waitcnt vmcnt(0)" : "=&v"(v) : "v"(p) : "memory"); return v; }
__device__ __forceinline__ u64 now() { return __builtin_amdgcn_s_memrealtime(); }
__device__ __forceinline__ u32 vm_peek() { u32 v; asm volatile("s_getreg_b32 %0, hwreg(HW_REG_IB_STS, 0, 4)" : "=s"(v) :: "memory"); return v; }
__device__ __forceinline__ char* uni(char* p) {
  const u64 v = (u64)p;
  return (char*)(((u64)(u32)__builtin_amdgcn_readfirstlane((u32)(v >> 32)) << 32) | (u32)__builtin_amdgcn_readfirstlane((u32)v));
}
__device__ __forceinline__ u32 clk() { return (u32)__builtin_amdgcn_s_memtime(); }
__device__ __forceinline__ u32 pk2(float a, float b) { return __builtin_bit_cast(u32, bf16x2{(__bf16)a, (__bf16)b}); }
__device__ __forceinline__ u32x4 pk8(f32x4 a, f32x4 c) { return u32x4{pk2(a[0], a[1]), pk2(a[2], a[3]), pk2(c[0], c[1]), pk2(c[2], c[3])}; }
__device__ __forceinline__ float lo16(u32 v) { return __builtin_bit_cast(float, v << 16); }
__device__ __forceinline__ float hi16(u32 v) { return __builtin_bit_cast(float, v & 0xffff0000u); }

__device__ __forceinline__ u32 wave_max(u32 v) {
  v = max(v, (u32)__builtin_amdgcn_update_dpp(0, (int)v, 0xb1, 0xf, 0xf, true));
  v = max(v, (u32)__builtin_amdgcn_update_dpp(0, (int)v, 0x4e, 0xf, 0xf, true));
  v = max(v, (u32)__builtin_amdgcn_update_dpp(0, (int)v, 0x141, 0xf, 0xf, true));
  v = max(v, (u32)__builtin_amdgcn_update_dpp(0, (int)v, 0x140, 0xf, 0xf, true));
  return max(max((u32)__builtin_amdgcn_readlane(v, 0), (u32)__builtin_amdgcn_readlane(v, 16)),
             max((u32)__builtin_amdgcn_readlane(v, 32), (u32)__builtin_amdgcn_readlane(v, 48)));
}
template <int SLP, int VM> __device__ __forceinline__ u32 poll(const char* p, u32 call, u64 t0, bool& bad, u32x2& r) {
  u32 v, late;
  u64 ex;
  asm volatile("s_mov_b64 %2, exec\n\tglobal_load_dwordx2 v[100:101], %4, off sc0 sc1\n\ts_waitcnt vmcnt(0)\n\ts_cmp_eq_u32 0, 1\n\ts_branch 3f\n"
               "1:\n\ts_sleep %8\n\tglobal_load_dwordx2 v[100:101], %4, off sc0 sc1\n\t"
               "s_memrealtime s[60:61]\n\ts_waitcnt vmcnt(%9) lgkmcnt(0)\n\t"
               "s_sub_u32 %1, s60, %6\n\ts_cmp_gt_u32 %1, %7\n\ts_cbranch_scc1 2f\n\t"
               "3:\n\tv_xor_b32 %0, v100, v101\n\tv_cmpx_ne_u32 vcc, %5, %0\n\ts_cbranch_execnz 1b\n"
               "2:\n\ts_cselect_b32 %1, 1, 0\n\ts_mov_b64 exec, %2\n\tv_mov_b32 %0, v100\n\ts_nop 1"
               : "=&v"(v), "=&s"(late), "=&s"(ex), "=&{v[100:101]}"(r)
               : "v"(p), "s"(call), "s"((u32)t0), "s"((u32)DEADLINE), "i"(SLP), "i"(VM) : "memory", "vcc", "scc", "s60", "s61");
  bad |= __builtin_amdgcn_readfirstlane(late) != 0;
  return v;
}
__device__ __forceinline__ void settle(u32x2& r) { asm volatile("s_waitcnt vmcnt(0)" : "+{v[100:101]}"(r) :: "memory"); }
__device__ __forceinline__ void wg_max(u32 wmx, LAS u32* sm, char* ws, int b, int lane, u32 call) {
  if (lane == 0) {
    u32 tk, m4;
    asm volatile("ds_max_u32 %2, %3\n\tds_add_rtn_u32 %0, %2, %4 offset:4\n\tds_read_b32 %1, %2\n\ts_waitcnt lgkmcnt(0)"
                 : "=&v"(tk), "=&v"(m4) : "v"((u32)(size_t)sm), "v"(wmx), "v"(1u) : "memory");
    if (tk == 3) st8_sys(ws + WS_LMAX + (b & 7) * 4096 + (b >> 3) * CFG_LSTR, u32x2{m4, m4 ^ call});
  }
}
__device__ __forceinline__ u32 wait_gmax(char* ws, int b, int lane, u32 call, u64 t0, bool bad) {
  const char* pg = ws + WS_GMAX + (size_t)((call & 1) * REP + b % REP) * 8192 + lane * CFG_GSTR;
  u32 gm;
  while (true) {
    const u32x2 r = ld8_sys(pg);
    gm = r[0];
    if (__all((r[0] ^ r[1]) == call)) break;
    if (now() - t0 > DEADLINE) { bad = true; break; }
    __builtin_amdgcn_s_sleep(1);
  }
  if (bad && lane == 0) st4_sys(ws + WS_TMO, call);
  return bad ? 0xffffffffu : wave_max(gm);
}
__device__ __forceinline__ float inv_scale(u32 gm) {
  const float scale = __builtin_bit_cast(float, gm) / 127.0f;
  return scale > 0.f ? 1.0f / scale : 0.f;
}
__device__ __forceinline__ u32 quant4(f32x4 y, float inv) {
  u32 q[4];
#pragma unroll
  for (int e = 0; e < 4; e++) q[e] = __builtin_bit_cast(u32, fminf(__builtin_fmaf(y[e], inv, 8388608.f), 8388735.f));
  return __builtin_amdgcn_perm(__builtin_amdgcn_perm(q[3], q[2], 0x0c0c0400u), __builtin_amdgcn_perm(q[1], q[0], 0x0c0c0400u), 0x05040100u);
}
__device__ __forceinline__ u32 spread4(u32 nb) { return __builtin_amdgcn_ubfe(nb, 0, 4) * 0x204081u & 0x01010101u; }
__device__ __forceinline__ u32x2 tags(u32 use, bool wide) {
  const u32 t = spread4(use) << 7;
  return wide ? u32x2{(use & 1) << 15 | (use & 2) << 30, (use & 4) << 13 | (use & 8) << 28} : u32x2{t, t};
}
__device__ __forceinline__ u32 wave_scan(u32 v) {
  v += (u32)__builtin_amdgcn_update_dpp(0, (int)v, 0x111, 0xf, 0xf, false);
  v += (u32)__builtin_amdgcn_update_dpp(0, (int)v, 0x112, 0xf, 0xf, false);
  v += (u32)__builtin_amdgcn_update_dpp(0, (int)v, 0x114, 0xf, 0xf, false);
  v += (u32)__builtin_amdgcn_update_dpp(0, (int)v, 0x118, 0xf, 0xf, false);
  v += (u32)__builtin_amdgcn_update_dpp(0, (int)v, 0x142, 0xa, 0xf, false);
  v += (u32)__builtin_amdgcn_update_dpp(0, (int)v, 0x143, 0xc, 0xf, false);
  return v;
}

template <int STR> __device__ __forceinline__ void ld64_sys(u32x4* r, const char* base, u32 off) {
  asm volatile("global_load_dwordx4 %0, %4, %5 sc0 sc1\n\tglobal_load_dwordx4 %1, %4, %6 sc0 sc1\n\t"
               "global_load_dwordx4 %2, %4, %7 sc0 sc1\n\tglobal_load_dwordx4 %3, %4, %8 sc0 sc1\n\ts_waitcnt vmcnt(0)"
               : "=&v"(r[0]), "=&v"(r[1]), "=&v"(r[2]), "=&v"(r[3])
               : "v"(off), "s"(base), "s"(base + STR), "s"(base + 2 * STR), "s"(base + 3 * STR) : "memory");
}
__device__ __forceinline__ u32x2 lds_ld8(LAS u32* p) { u32x2 v; asm volatile("ds_read_b64 %0, %1\n\ts_waitcnt lgkmcnt(0)" : "=&v"(v) : "v"((u32)(size_t)p) : "memory"); return v; }

struct Peers { char* ws[NR]; char* out[NR]; };
__global__ __launch_bounds__(512, 2) void gemm_a2a(const char* __restrict__ x, const char* __restrict__ w, char* ws, char* out,
                                                   char* const* ws_all, int rank, Peers peers) {
  extern __shared__ __attribute__((aligned(16))) char smem[];
  if (CFG_LF) asm volatile("" : "+s"(x), "+s"(w), "+s"(ws), "+s"(out));
  const int b = blockIdx.x, lane = threadIdx.x & 63;
  const int wave = __builtin_amdgcn_readfirstlane(threadIdx.x >> 6);
  const int xcd = b & 7, idx = b >> 3;
  const int id = idx ^ CFG_EX;
  const int ks = (xcd ^ CFG_KSF) & 1, mt = (id >> 1) & 3, ntl = (id & 1) * 4 + (id >> 3), nt = (xcd >> 1) * 8 + ntl;
  const int g = lane >> 4, li = lane & 15;
  const int pos = (id & ~1) + ks, dest = nt >> 2, item = rank * 32 + pos;
  const bool aggr = id == 31;
  auto place = [&](int j) { return out + (size_t)(xcd * MS + (j & 7) * 64) * NCOL + (j >> 3) * 128; };
  LAS u32* sm = (LAS u32*)(smem + SMALL);
  u32* tmo = (u32*)(ws + WS_TMO);
  auto get_gmax = [&](u32 call, u64 t0, bool bad, auto&& idle) {
    u32 gm;
    while (true) {
      const u32x2 v = lds_ld8(sm + 6);
      gm = __builtin_amdgcn_readfirstlane(v[0]);
      if (__builtin_amdgcn_readfirstlane(v[1]) == call) break;
      if (now() - t0 > 2 * DEADLINE) { bad = true; break; }
      idle();
    }
    if (bad && lane == 0) st4_sys(tmo, call);
    return bad ? 0xffffffffu : gm;
  };
  u32 tr1 = 0;
  auto look = [&](int c, u32 call, u64 t0) {
    while (now() - t0 < (u32)(CFG_GDLY + c * CFG_GPH)) __builtin_amdgcn_s_sleep(2);
    if (CFG_GH && aggr) while (__builtin_amdgcn_readfirstlane(lds_ld8(sm + 2)[1]) != call && now() - t0 < 2 * DEADLINE) __builtin_amdgcn_s_sleep(2);
    bool bad = false;
    u32x2 r;
    u32 gm = poll<CFG_GSLP, CFG_GVM>(ws + WS_GMAX + (size_t)((call & 1) * REP + (b >> CFG_RA) % REP) * 8192 + c * CFG_GO + lane * CFG_GSTR, call, t0, bad, r);
    const u32 t1 = (u32)now();
    gm = bad ? 0xffffffffu : wave_max(gm);
    if (lane == 0) asm volatile("ds_write_b32 %0, %1 offset:32\n\tds_write_b64 %0, %2 offset:24" :: "v"((u32)(size_t)sm), "v"(t1), "v"(u32x2{gm, call}) : "memory");
    settle(r);
    if (bad && lane == 0) st4_sys(tmo, call);
    if (CFG_TC) {
      if (id == CFG_TID && lane < 8) st8_sys(ws + WS_TREF + lane * 1152 + xcd * 8, u32x2{t1, call});
      while ((u32)now() - t1 < CFG_TW) {}
      asm volatile("global_load_dwordx2 v[100:101], %0, off sc0 sc1" :: "v"(ws + WS_TREF + xcd * 1152 + (lane & 7) * 8) : "memory", "v100", "v101");
      tr1 = t1;
    }
  };
  auto lateness = [&](u32 call) {
    u32x2 tr;
    asm volatile("s_waitcnt vmcnt(0)\n\tv_mov_b64 %0, v[100:101]" : "=v"(tr) :: "memory");
    const int d = tr[1] == call ? (int)(tr1 - tr[0]) - CFG_TC : 0;
    const u32 m = wave_max(min(max(d, 0), 100));
    if (lane == 0) sm[9] = m;
  };
  auto pace = [&](u64 t, u32 dly, bool on, bool nap) {
    while ((u32)(now() - t) + (CFG_TC && on ? __builtin_amdgcn_readfirstlane(lds_ld8(sm + 8)[1]) : 0) < dly) if (nap) __builtin_amdgcn_s_sleep(1);
  };
  auto recv = [&](int rpos, u32 call, u64 t0) {
    const int h = wave & 3, mitem = xcd * 32 + rpos;
    char* const mine = place(rpos);
    const bool early = rpos < CFG_NE;
    const u32 gm = get_gmax(call, t0, false, [&]() { __builtin_amdgcn_s_sleep(2); });
    bool bad = gm == 0xffffffffu;
    t0 = now();
    t0 -= (u32)t0 - __builtin_amdgcn_readfirstlane(lds_ld8(sm + 8)[0]);
    if (CFG_ND && rpos >= P0) {
      const char* op = CFG_DL ? mine + h * 16 * NCOL : mine + (h >> 1) * 32 * NCOL + (h & 1) * 64;
      const u32 o0 = CFG_DL ? (lane >> 3) * NCOL + (lane & 7) * 16 : li * NCOL + g * 16, o1 = o0 + (CFG_DL ? 8 : 16) * NCOL;
      const u32 dly = CFG_PDLY + CFG_DOFF - CFG_RD * (CFG_ND * 400 - 100 - CFG_DRW * ((rpos - P0) * 4 + h)) / ((32 - CFG_NE) * 400);
      if (CFG_TC && wave == 6) lateness(call);
      pace(t0, dly, CFG_TM & 2, true);
      while (!bad) {
        u32x4 r0, r1;
        asm volatile("global_load_dwordx4 %0, %2, %4 sc0 sc1\n\tglobal_load_dwordx4 %1, %3, %4 sc0 sc1\n\ts_waitcnt vmcnt(0)"
                     : "=&v"(r0), "=&v"(r1) : "v"(o0), "v"(o1), "s"(op) : "memory");
        const u32x4 e = r0 | r1;
        if (__all(((e[0] | e[1] | e[2] | e[3]) & 0x80808080u) == 0)) break;
        if (now() - t0 > DEADLINE) bad = true;
        __builtin_amdgcn_s_sleep(CFG_DSLP);
      }
      if (bad) {
        st16_so<0>(op, o0, u32x4{0x80808080u, 0x80808080u, 0x80808080u, 0x80808080u});
        if (lane == 0) st4_sys(tmo, call);
      }
      return;
    }
    const char* pm = ws + WS_MSK + (size_t)(((call & 1) * 256 + mitem) * 4 + h) * CFG_MSTR + (CFG_MC ? lane * 4 : CFG_M4 ? (lane & 31) * 16 + (lane >> 5) * 8 : lane * 8);
    u32 km = 0;
    while (!bad) {
      if (CFG_MC) {
        const u32 r = ld4_sys(pm);
        km = ~r | 0x80000000u;
        if (__all((int)(r ^ (call + 1) << 30) >= 0)) break;
      } else {
        const u32x2 r = ld8_sys(pm);
        km = ~r[0];
        if (__all(r[1] >= call)) break;
      }
      if (now() - t0 > DEADLINE) bad = true;
      __builtin_amdgcn_s_sleep(2);
    }
    if (CFG_TC && wave == 6) lateness(call);
    const u32 cnt = __builtin_popcount(km), inc = wave_scan(cnt);
    const int len = early ? (__builtin_amdgcn_readlane(inc, 63) + 7) >> 3 : (__builtin_amdgcn_readlane(inc, 63) + 15) >> 4;
    const u32 lb = (u32)(size_t)smem + WRING + wave * 4096;
    u32 ad[8], sel[8], keep[8];
#pragma unroll
    for (int k = 0; k < 8; k++) {
      const u32 sh = 4 * (k ^ (g & 1));
      const u32 o = inc - cnt + __builtin_popcount(km & ((1u << sh) - 1)), sp = spread4(km >> sh);
      ad[k] = lb + (o & ~3u);
      sel[k] = (sp << 8) + (sp << 16) + (sp << 24) + (o & 3) * 0x01010101u;
      keep[k] = sp * 127;
      asm volatile("" : "+v"(ad[k]), "+v"(sel[k]), "+v"(keep[k]));
    }
    const char* stg = ws + WS_STG + (size_t)(call & 1) * 4 * PLANE + (mitem * 4 + h) * SSTR;
    const u32 use = (call + 1) >> 1;
    const u32x2 tg = tags(use, early);
    const u32x4 tag = u32x4{tg[0], tg[1], tg[0], tg[1]};
    auto cl = [&](int i) { return (u32)max(min(lane + 64 * i, len - 1), 0); };
    auto off = [&](int i) { return cell_off(cl(i)); };
    if (early) {
      u32x4 r[4];
      while (!bad && len > 0) {
        asm volatile("global_load_dwordx4 %0, %4, %8 sc0 sc1\n\tglobal_load_dwordx4 %1, %5, %8 sc0 sc1\n\t"
                     "global_load_dwordx4 %2, %6, %8 sc0 sc1\n\tglobal_load_dwordx4 %3, %7, %8 sc0 sc1\n\ts_waitcnt vmcnt(0)"
                     : "=&v"(r[0]), "=&v"(r[1]), "=&v"(r[2]), "=&v"(r[3]) : "v"(off(0)), "v"(off(1)), "v"(off(2)), "v"(off(3)), "s"(stg) : "memory");
        const u32x4 e = (r[0] ^ tag) | (r[1] ^ tag) | (r[2] ^ tag) | (r[3] ^ tag);
        if (__all(((e[0] | e[1] | e[2] | e[3]) & 0x80008000u) == 0)) break;
        if (now() - t0 > DEADLINE) bad = true;
        __builtin_amdgcn_s_sleep(CFG_ESLP);
      }
      const float inv = inv_scale(gm);
#pragma unroll
      for (int i = 0; i < 4; i++) {
        if (i * 64 >= len) break;
        f32x4 y[2];
#pragma unroll
        for (int e = 0; e < 4; e++) {
          const u32 v = r[i][e];
          y[e >> 1][2 * (e & 1)] = __builtin_fabsf(lo16(v));
          y[e >> 1][2 * (e & 1) + 1] = __builtin_bit_cast(float, v & 0x7fff0000u);
        }
        *(LAS u32x2*)(size_t)(lb + cl(i) * 8) = u32x2{quant4(y[0], inv), quant4(y[1], inv)};
      }
    } else {
      const u32 dly = CFG_PDLY - CFG_RD * ((32 - rpos) * 4 - 1 - h) / ((32 - CFG_NE) * 4) - (pos < CFG_NE ? CFG_ERD : 0);
      pace(t0, dly, CFG_TM & 2, true);
      u32x4 r0, r1;
      while (!bad && len > 0) {
        asm volatile("global_load_dwordx4 %0, %2, %4 sc0 sc1\n\tglobal_load_dwordx4 %1, %3, %4 sc0 sc1\n\ts_waitcnt vmcnt(0)"
                     : "=&v"(r0), "=&v"(r1) : "v"(off(0)), "v"(off(1)), "s"(stg) : "memory");
        const u32x4 e = (r0 ^ tag) | (r1 ^ tag);
        if (__all(((e[0] | e[1] | e[2] | e[3]) & 0x80808080u) == 0)) break;
        if (now() - t0 > DEADLINE) bad = true;
        __builtin_amdgcn_s_sleep(CFG_PSLP);
      }
      *(LAS u32x4*)(size_t)(lb + cl(0) * 16) = r0;
      *(LAS u32x4*)(size_t)(lb + cl(1) * 16) = r1;
    }
    asm volatile("" ::: "memory");
    u32 d[8];
#pragma unroll
    for (int k = 0; k < 8; k++) {
      const LAS u32* q = (const LAS u32*)(size_t)ad[k];
      d[k] = __builtin_amdgcn_perm(q[1], q[0], sel[k]) & keep[k];
    }
    if (bad) {
#pragma unroll
      for (int k = 0; k < 8; k++) { d[k] = 0x80808080u; asm volatile("" : "+v"(d[k])); }
    }
    const char* op = mine + (h >> 1) * 32 * NCOL + (h & 1) * 64;
#pragma unroll
    for (int m = 0; m < 2; m++) {
      const u32x4 v = u32x4{d[4 * m], d[4 * m + 1], d[4 * m + 2], d[4 * m + 3]};
      st16_so<0>(op + m * 16 * NCOL, li * NCOL + g * 16, v);
    }
    const int cell = (lane >> 3) * 64 + (use & 7) * 8 + (lane & 7);
    if (lane < (early ? 32 : 16) && cell >= len) st16_so<0>(stg, cell_off(cell), tag);
    if (bad && lane == 0) st4_sys(tmo, call);
  };
  u32 call;
  u64 t0;
  if (__builtin_expect((wave & 6) == 4, CFG_LF)) {
    const int lw = wave - 4;
    const int kx = CFG_PH ? mt ^ (ntl >> 1) : (mt ^ ntl) & 3;
    const int ph = (ntl & 1 ? 0 : CFG_PH) + CFG_PX * (xcd >> 2);
    auto rx = __builtin_amdgcn_make_buffer_rsrc((void*)x, 0, MS * KD * 2, 0x00020000);
    auto rw = __builtin_amdgcn_make_buffer_rsrc((void*)w, 0, KD * ND * 2, 0x00020000);
    int vx[2], vw[2];
#pragma unroll
    for (int h = 0; h < 2; h++) {
      vx[h] = (lane >> 3) * 8192 + (((lane & 7) ^ ((((lane >> 3) + 8 * h) >> 1) & 7)) << 4);
      vw[h] = g * 8192 + ((li ^ g ^ (8 * h)) << 4);
    }
    const int sx0 = (mt * 128 + lw * 32) * 8192 + ks * KD;
    const int sw0 = (ks * (KD / 2) + lw * 32) * 8192 + nt * 256;
    auto spos = [&](int s) { return ((s + ph) & (NSTEP - 1)) ^ kx; };
    auto ix = [&](int s, int slot, int hf) {
      LAS char* l = (LAS char*)(smem + XRING + slot * BLK + hf * 8192 + lw * 4096);
      const int o = sx0 + (hf ^ ks) * 64 * 8192 + spos(s) * 128;
#pragma unroll
      for (int i = 0; i < 4; i++) __builtin_amdgcn_raw_ptr_buffer_load_lds(rx, l + i * 1024, 16, vx[i & 1], o + i * 8 * 8192, 0, 0);
    };
    auto issue = [&](int s, int slot, bool all) {
      if (all) ix(s, slot, 0);
      ix(s, slot, 1);
      LAS char* l = (LAS char*)(smem + WRING + slot * BLK + lw * 8192);
      const int sp = spos(s);
#pragma unroll
      for (int i = 0; i < 8; i++) __builtin_amdgcn_raw_ptr_buffer_load_lds(rw, l + i * 1024, 16, vw[(i >> 1) & 1], sw0 + (sp * 64 + i * 4) * 8192, 0, 0);
    };
#pragma unroll
    for (int i = 0; i < CFG_PRE; i++) issue(i, i, true);
    wait_vm<16 * (CFG_PRE - 1)>();
    __builtin_amdgcn_s_barrier();
#pragma unroll
    for (int i = CFG_PRE; i < NS; i++) issue(i, i, true);
#pragma unroll 1
    for (int j = 1; j <= NSTEP - 2 * NS; j++) {
      wait_vm<32>();
      __builtin_amdgcn_s_barrier();
      issue(j - 1 + NS, (j - 1) & 3, true);
    }
    tail<NSTEP - 2 * NS + 1>([&](auto jc) {
      constexpr int j = jc.v;
      wait_vm<left(j)>();
      __builtin_amdgcn_s_barrier();
      if (j <= NSTEP - NS) {
        int s = j - 1 + NS;
        asm volatile("" : "+s"(s));
        issue(s, s & 3, nld(j - 1 + NS) == 16);
      }
#pragma unroll
      for (int i = 0; i < DEF; i++) {
        if (!CFG_XD || xat(i) != j) continue;
        int s = NSTEP - DEF + i;
        asm volatile("" : "+s"(s));
        ix(s, s & 3, 0);
      }
    });
    call = __builtin_amdgcn_readfirstlane(sm[2]);
    t0 = now();
    bool bad = false;
    if (lw == 1 && aggr) {
      char* pw = ws_all[lane & 7];
      u32 gm = 0;
      u32x2 r;
      char* pd = pw + WS_GMAX + (size_t)((call & 1) * REP + (lane >> 3)) * 8192 + (rank * NAGG + xcd) * CFG_GSTR;
      if (CFG_ADLY) while (now() - t0 < CFG_ADLY) __builtin_amdgcn_s_sleep(2);
      asm volatile("" : "+v"(pd));
      if (CFG_NW & 1) gm = poll<CFG_ASLP, CFG_AVM>(ws + WS_LMAX + xcd * 4096 + (lane & 31) * CFG_LSTR, call, t0, bad, r);
      else {
        const char* pg = lane < 32 ? ws + WS_LMAX + xcd * 4096 + lane * CFG_LSTR : (const char*)tmo - 4;
        while (true) {
          r = ld8_sys(pg);
          gm = lane < 32 ? r[0] : 0;
          if (__all(lane >= 32 || (r[0] ^ r[1]) == call)) break;
          if (__builtin_amdgcn_readlane(r[1], 63) == call || now() - t0 > DEADLINE) { bad = true; break; }
        }
      }
      gm = bad ? 0xffffffffu : wave_max(gm);
      if (lane < NR * REP) st8_sys(pd, u32x2{gm, gm ^ call});
#pragma unroll
      for (int c = 1; c < CFG_GK; c++) st8_sys(pd + c * CFG_GO, u32x2{gm, gm ^ call});
      if (lane == 0) sm[3] = call;
      if (CFG_NW & 1) settle(r);
    }
  } else if (wave >= 6) {
    {
#pragma unroll 1
      for (int nb = 0; nb < 4; nb++) __builtin_amdgcn_s_barrier();
      if (CFG_ND && jobA(id) >= P0) {
#pragma unroll
        for (int i = 0; i < 4; i++)
          st16_so<0>(place(jobA(id)) + ((wave & 1) * 32 + i * 8) * NCOL, (lane >> 3) * NCOL + (lane & 7) * 16, u32x4{0x80808080u, 0x80808080u, 0x80808080u, 0x80808080u});
      }
#pragma unroll 1
      for (int nb = 4; nb < 16; nb++) __builtin_amdgcn_s_barrier();
      wait_vm<0>();
#pragma unroll 1
      for (int nb = 16; nb < NBAR; nb++) __builtin_amdgcn_s_barrier();
    }
    call = __builtin_amdgcn_readfirstlane(sm[2]);
    t0 = now();
    if (wave == 6) {
      look(0, call, t0);
      if (CFG_TC && jobA(id) < 0) lateness(call);
    }
  } else {
  const int wm = wave >> 1, wn = wave & 1;
  u32* ctr = (u32*)(ws + WS_CTR + b * 128);
  call = *(const __attribute__((address_space(4))) u32*)(size_t)ctr + 1;
  char* const pw = peers.ws[dest & 7];
  const char* pmsk = pw + WS_MSK + (size_t)(((call & 1) * 256 + item) * 4 + wave) * CFG_MSTR;
  const bool direct = CFG_ND && pos >= P0;
  const char* pstg = direct ? peers.out[dest & 7] + (size_t)(rank * MS + (item & 7) * 64 + (CFG_DL ? wave * 16 : wm * 32)) * NCOL + (item >> 3 & 3) * 128 + (CFG_DL ? 0 : wn * 64)
                            : pw + WS_STG + (size_t)(call & 1) * 4 * PLANE + (item * 4 + wave) * SSTR;
  const char* xdst = ws + WS_EX + (size_t)((b ^ 1) * 4 + wave) * XSTR + 2048;
  char* xsrc = ws + WS_EX + (size_t)(b * 4 + wave) * XSTR;
  u32 xtag = (0x2000u | (call & 0x1fffu)) << 16;
  u32 l16 = lane * 16, h16 = (lane & 31) * 16, h8 = h16 + (lane >> 5) * 8;
  asm volatile("" : "+v"(l16), "+v"(h16), "+v"(h8));
  u32x4 u0, u1, u2;
  {
    const u32 e = g == li >> 2 ? li & 3 : 4, one = 0x3f80u << (li & 1) * 16;
    u0 = u32x4{e < 2 ? one : 0, e == 2 ? 0x3f80u : 0, 0, e == 3 ? 0x3f80u : 0}; u1 = u32x4{0, 0, u0[0], 0}; u2 = u32x4{0, 0, (e & 6) == 2 ? one : 0, 0};
    asm volatile("" : "+v"(u0), "+v"(u1), "+v"(u2));
  }
  f32x4 acc[4][4];
#pragma unroll
  for (int i = 0; i < 16; i++) acc[i >> 2][i & 3] = f32x4{0.f, 0.f, 0.f, 0.f};
  u32 ax[2], aw[4];
#pragma unroll
  for (int kk = 0; kk < 2; kk++) ax[kk] = (u32)(size_t)smem + XRING + (32 * wm + li) * 128 + (((4 * kk + g) ^ ((li >> 1) & 7)) << 4);
  {
    const int q = li >> 2, p = li & 3;
#pragma unroll
    for (int n = 0; n < 4; n++)
      aw[n] = (u32)(size_t)smem + WRING + (8 * g + q) * 256 + (((8 * wn + 2 * p + (n >> 1)) ^ q ^ ((g & 1) << 3)) << 4) + (((n ^ p) & 1) << 3);
  }
#pragma unroll
  for (int n = 0; n < 4; n++) asm volatile("" : "+v"(aw[n]));
  bf16x8 fx[2][4], fw[2][4];
  auto rdw = [&](int buf, int n, u32 a) {
    const bf16x4 lo = __builtin_amdgcn_ds_read_tr16_b64_v4bf16((LAS bf16x4*)(size_t)a);
    const bf16x4 hi = __builtin_amdgcn_ds_read_tr16_b64_v4bf16((LAS bf16x4*)(size_t)(a + 1024));
    fw[buf][n] = __builtin_shufflevector(lo, hi, 0, 1, 2, 3, 4, 5, 6, 7);
  };
  auto rd = [&](int buf, int slot, int kk) {
    rdw(buf, 0, aw[0] + slot * BLK + kk * 8192);
#pragma unroll
    for (int m = 0; m < 4; m++) fx[buf][m] = *(LAS bf16x8*)(size_t)(ax[kk] + slot * BLK + ((m & 1) * 16 + (m >> 1) * 64) * 128);
#pragma unroll
    for (int n = 1; n < 4; n++) rdw(buf, n, aw[n] + slot * BLK + kk * 8192);
  };
  auto mm = [&](int buf) {
#pragma unroll
    for (int n = 0; n < 4; n++)
#pragma unroll
      for (int i = 0; i < 4; i++) {
        const int m = (n & 1) ? 3 - i : i;
        acc[m][n] = __builtin_amdgcn_mfma_f32_16x16x32_bf16(fw[buf][n], fx[buf][m], acc[m][n], 0, 0, 0);
      }
  };
  auto mix = [&]() {
#pragma unroll
    for (int i = 0; i < 12; i++) { __builtin_amdgcn_sched_group_barrier(0x008, 1, 0); __builtin_amdgcn_sched_group_barrier(0x100, 1, 0); }
    __builtin_amdgcn_sched_group_barrier(0x008, 4, 0);
    __builtin_amdgcn_sched_barrier(0);
  };
  auto sync = [&]() {
    __builtin_amdgcn_s_waitcnt(0xc07f);
    asm volatile("" ::: "memory");
    __builtin_amdgcn_s_barrier();
    __builtin_amdgcn_sched_barrier(0);
  };
  auto step = [&](int u, bool last) {
    __builtin_amdgcn_s_waitcnt(0xc67f);
    __builtin_amdgcn_sched_barrier(0);
    rd(1, u, 1); mm(0); mix();
    if (!last) { sync(); rd(0, (u + 1) % NS, 0); mm(1); mix(); }
    else mm(1);
  };
#pragma unroll
  for (int i = 0; i < 16; i++) asm volatile("" : "+v"(acc[i >> 2][i & 3]));
  asm volatile("" : "+s"(pmsk), "+s"(pstg), "+s"(xdst), "+s"(xsrc), "+s"(xtag));
  if (wave == 0 && lane < 10) sm[lane] = lane == 2 ? call : 0;
  sync();
  rd(0, 0, 0);
  __builtin_amdgcn_sched_barrier(0);
#pragma unroll 1
  for (int j0 = 0; j0 < NSTEP - NS; j0 += NS) {
#pragma unroll
    for (int u = 0; u < NS; u++) step(u, false);
  }
#pragma unroll
  for (int u = 0; u < NS - DEF; u++) step(u, u == NS - 1);
  auto frag = [&](int hf, int q) -> f32x4& { return acc[2 * hf + ((q ^ q >> 1) & 1)][q >> 1]; };
  auto dump = [&](int j) {
    const float tf = __builtin_bit_cast(float, xtag);
    if (j < 5) {
      const f32x4 f = frag(1, j + 1 + (j >> 1)), h = frag(1, 3 * (j >> 1));
      const u32x4 c = u32x4{pk2(f[0], f[1]), pk2(f[2], tf), pk2(h[2 * (j & 1)], h[2 * (j & 1) + 1]), pk2(f[3], tf)};
      if (j == 0) st16_x<-2048>(xdst, l16, c);
      if (j == 1) st16_x<-1024>(xdst, l16, c);
      if (j == 2) st16_x<0>(xdst, l16, c);
      if (j == 3) st16_x<1024>(xdst, l16, c);
      if (j == 4) st16_x<2048>(xdst, l16, c);
    } else {
      const f32x4 h = frag(1, 6);
      const u32 v = pk2(h[2], h[3]);
      const auto two = __builtin_amdgcn_permlane32_swap(v, v, false, false);
      st16_x<3072>(xdst, h16, u32x4{two[0], xtag, two[1], xtag});
    }
  };
  auto rs = __builtin_amdgcn_make_buffer_rsrc(xsrc, 0, XSTR, 0x00020000);
  u32x4 r[5];
  u32x2 r5;
  auto fetch = [&](int j) {
    if (j < 5) r[j] = __builtin_amdgcn_raw_buffer_load_b128(rs, l16 + (j & 3) * 1024, (j >> 2) * 4096, CFG_XLD);
    else r5 = __builtin_amdgcn_raw_buffer_load_b64(rs, h8 + 1024, 4096, CFG_XLD);
  };
  if (DEF) {
    bf16x4 wl[4];
#pragma unroll
    for (int t = 0; t < 4 * DEF; t++) {
      const int hf = t < 2 * DEF, n = t + 1, nh = n < 2 * DEF, slot = (NSTEP - DEF + (n % (2 * DEF)) / 2) & 3, o = slot * BLK + (n & 1) * 8192;
      if ((t & 1) && n < 4 * DEF) {
        if (n == 4 * DEF - 2) __builtin_amdgcn_s_waitcnt(0xc07f);
        asm volatile("" ::: "memory");
        __builtin_amdgcn_s_barrier();
        asm volatile("" ::: "memory");
        __builtin_amdgcn_sched_barrier(0);
      }
      auto lo = [&](int i) { wl[i] = __builtin_amdgcn_ds_read_tr16_b64_v4bf16((LAS bf16x4*)(size_t)(aw[i] + o)); };
      auto hi = [&](int i) {
        const bf16x4 h = __builtin_amdgcn_ds_read_tr16_b64_v4bf16((LAS bf16x4*)(size_t)(aw[i] + o + 1024));
        fw[n & 1][i] = __builtin_shufflevector(wl[i], h, 0, 1, 2, 3, 4, 5, 6, 7);
      };
      auto xr = [&](int m) { fx[n & 1][2 * nh + m] = *(LAS bf16x8*)(size_t)(ax[n & 1] + slot * BLK + (m * 16 + nh * 64) * 128); };
      if (CFG_DM == 0 && t == 2 * DEF) {
#pragma unroll
        for (int j = 0; j < 6; j++) dump(j);
        __builtin_amdgcn_sched_barrier(0);
      }
#pragma unroll
      for (int q = 0; q < 8; q++) {
        frag(hf, q) = __builtin_amdgcn_mfma_f32_16x16x32_bf16(fw[t & 1][q >> 1], fx[t & 1][2 * hf + ((q ^ q >> 1) & 1)], frag(hf, q), 0, 0, 0);
        __builtin_amdgcn_sched_barrier(0);
        if (n < 4 * DEF) {
          if (q == 0) { lo(0); hi(0); }
          if (q == 1) xr(0);
          if (q == 2) xr(1);
          if (q == 3) { lo(1); hi(1); }
          if (q == 4) lo(2);
          if (q == 5) hi(2);
          if (q == 6) lo(3);
          if (q == 7) hi(3);
        }
        const int z = (t - 2 * DEF) * 8 + q;
        if (CFG_DM == 1) {
          if (z == -4) dump(0);
          if (z == -2) dump(1);
          if (z == -1) dump(2);
          if (z == 1) dump(3);
          if (z == 2) dump(4);
          if (z == 4) dump(5);
        }
        if (CFG_DM >= 3 && z >= -4 && (z + 4) % (CFG_DM - 1) == 0 && (z + 4) / (CFG_DM - 1) < 6) dump((z + 4) / (CFG_DM - 1));
        if (CFG_DM == 2) {
          if (z == -4) dump(0);
          if (z == -3) dump(1);
          if (z == -2) dump(2);
          if (z == -1) dump(3);
          if (z == 0) dump(5);
          if (z == 1) dump(4);
        }
        if (t == 4 * DEF - CFG_XP && q >= 1 && q < 7) fetch(q - 1);
        __builtin_amdgcn_sched_barrier(0);
      }
    }
  }
  bool bad = false;
  t0 = now();
  {
    if (!DEF) {
#pragma unroll
      for (int j = 0; j < 6; j++) dump(j);
      __builtin_amdgcn_sched_barrier(0);
    }
    if (!DEF || !CFG_XP) {
      if (CFG_XDLY) while (now() - t0 < CFG_XDLY) {}
#pragma unroll
      for (int j = 0; j < 6; j++) fetch(j);
    }
    while (true) {
      u32 d = r5[1] ^ xtag;
#pragma unroll
      for (int j = 0; j < 5; j++) d |= (r[j][1] ^ xtag) | (r[j][3] ^ xtag);
      if (__all(d < 65536)) break;
      if (now() - t0 > DEADLINE) { bad = true; break; }
      if (CFG_XSLP) __builtin_amdgcn_s_sleep(CFG_XSLP);
      asm volatile("" ::: "memory");
#pragma unroll
      for (int j = 0; j < 6; j++) fetch(j);
    }
    frag(0, 6)[2] += lo16(r5[0]);
    frag(0, 6)[3] += hi16(r5[0]);
#pragma unroll
    for (int j = 0; j < 5; j++) {
      const bf16x8 v = __builtin_bit_cast(bf16x8, r[j]);
      f32x4 &f = frag(0, j + 1 + (j >> 1)), &h = frag(0, 3 * (j >> 1));
      h = __builtin_amdgcn_mfma_f32_16x16x32_bf16(__builtin_bit_cast(bf16x8, j & 1 ? u2 : u1), v, h, 0, 0, 0);
      f = __builtin_amdgcn_mfma_f32_16x16x32_bf16(__builtin_bit_cast(bf16x8, u0), v, f, 0, 0, 0);
    }
  }
  float mx = 0.f;
#pragma unroll
  for (int i = 0; i < 8; i++)
#pragma unroll
    for (int e = 0; e < 4; e++) mx = fmaxf(mx, acc[i >> 2][i & 3][e]);
  wg_max(bad ? 0xffffffffu : wave_max(__builtin_bit_cast(u32, mx)), sm, ws, b, lane, call);
  __builtin_amdgcn_sched_barrier(0);
  if (direct) {
    f32x4 y[2][4];
#pragma unroll
    for (int k = 0; k < 8; k++)
#pragma unroll
      for (int e = 0; e < 4; e++) y[k >> 2][k & 3][e] = fmaxf(g & 1 ? acc[k >> 2][(k & 3) ^ 1][e] : acc[k >> 2][k & 3][e], 0.f);
    u32 o0 = li * NCOL + g * 16;
    if (CFG_DL) {
      const u32 zb = (u32)(size_t)smem + XRING;
#pragma unroll
      for (int k = 0; k < 8; k++) *(LAS f32x4*)(size_t)(zb + (wm * 32 + (k >> 2) * 16 + li) * 528 + (wn * 64 + g * 16 + (k & 3) * 4) * 4) = y[k >> 2][k & 3];
      u32 tk;
      asm volatile("s_waitcnt lgkmcnt(0)\n\tds_add_u32 %0, %1 offset:16" :: "v"((u32)(size_t)sm + wm * 4), "v"(lane == 0 ? 1u : 0u) : "memory");
      while (__builtin_amdgcn_readfirstlane(lds_ld8(sm + 4)[wm]) != 2 && now() - t0 < 2 * DEADLINE) {}
#pragma unroll
      for (int k = 0; k < 8; k++) y[k >> 2][k & 3] = *(LAS f32x4*)(size_t)(zb + (wave * 16 + (k >> 2) * 8 + (lane >> 3)) * 528 + ((lane & 7) * 16 + (k & 3) * 4) * 4);
      o0 = (lane >> 3) * NCOL + (lane & 7) * 16;
    }
#pragma unroll
    for (int k = 0; k < 8; k++) asm volatile("" : "+v"(y[k >> 2][k & 3]));
    const u32 gm = get_gmax(call, t0, bad, [&]() {});
    if (gm != 0xffffffffu) {
      const u64 t1 = now();
      const float inv = inv_scale(gm);
      u32x4 c0 = u32x4{quant4(y[0][0], inv), quant4(y[0][1], inv), quant4(y[0][2], inv), quant4(y[0][3], inv)};
      u32x4 c1 = u32x4{quant4(y[1][0], inv), quant4(y[1][1], inv), quant4(y[1][2], inv), quant4(y[1][3], inv)};
      asm volatile("" : "+v"(c0), "+v"(c1));
      pace(t1, CFG_DSO + CFG_LD * ((P0 - CFG_NE) * 400 + CFG_DW * ((pos - P0) * 4 + wave)) / ((32 - CFG_NE) * 400), CFG_TM & 1, false);
      st16_so<0>(pstg, o0, c0);
      st16_so<0>(pstg + (CFG_DL ? 8 : 16) * NCOL, o0, c1);
    }
    if (wave == 0 && lane == 63) st4_sys(ctr, call);
    return;
  }
  u32 neg = 0;
  if (CFG_MC) { const float v = acc[1][3][3]; acc[1][3][3] = __builtin_bit_cast(float, max(__builtin_bit_cast(int, v), 0)); }
#pragma unroll
  for (int j = 31; j >= 0; j--) {
    const float v = acc[j >> 4][(j >> 2) & 3][j & 3];
    neg = __builtin_amdgcn_alignbit(neg, __builtin_bit_cast(u32, v), 31);
  }
  auto mask = [&]() {
    if (CFG_MC) {
      asm volatile("global_store_dword %0, %1, %2 sc0 sc1\n\ts_nop 1" ::"v"(lane * 4), "v"(neg | ((call + 1) << 30 & 0x80000000u)), "s"(pmsk) : "memory");
    } else if (CFG_M4) {
      const auto two = __builtin_amdgcn_permlane32_swap(neg, neg, false, false);
      if (lane < 32) st16_so<0>(pmsk, lane * 16, u32x4{two[0], call, two[1], call});
    } else st8_sys((char*)pmsk + lane * 8, u32x2{neg, call});
  };
  const bool hold = CFG_AM && aggr, early = pos < CFG_NE;
  if (!hold && !CFG_MZ) mask();
  const u32 cnt = __builtin_popcount(~neg), inc = wave_scan(cnt);
  const int total = __builtin_amdgcn_readlane(inc, 63);
  const u32 zb = (u32)(size_t)smem + XRING + 512 + wave * 8704;
  {
    u32 p = zb + (inc - cnt) * 4;
    const u32 skip = zb - 512 + lane * 4;
#pragma unroll
    for (int j = 0; j < 32; j++) {
      const float v = acc[j >> 4][(j >> 2) & 3][j & 3];
      u32 t = (u32)(__builtin_bit_cast(int, v) >> 31);
      asm("" : "+v"(t));
      *(LAS float*)(size_t)(((t & skip) | (~t & p)) + 4 * j) = v;
      p += t << 2;
      asm("" : "+v"(p));
    }
  }
  asm volatile("" ::: "memory");
  if (!hold && CFG_MZ) mask();
  const char* dst = pstg;
  const u32x2 tg = tags((call + 1) >> 1, early);
  if (early) {
    u32x4 c[4];
#pragma unroll
    for (int i = 0; i < 4; i++) {
      const LAS f32x4* p = (const LAS f32x4*)(size_t)(zb + i * 2048 + lane * 32);
      c[i] = (pk8(p[0], p[1]) & 0x7fff7fffu) | u32x4{tg[0], tg[1], tg[0], tg[1]};
    }
    if (CFG_EDLY) { const u64 t1 = now(); while (now() - t1 < (u32)(CFG_EOFF + CFG_EDLY * (pos * 4 + wave) / (CFG_NE * 4)) &&
                                                   (!CFG_EB || __builtin_amdgcn_readfirstlane(lds_ld8(sm + 6)[1]) != call)) __builtin_amdgcn_s_sleep(1); }
    if (!bad) {
      if (lane * 8 < total) st16_so<0>(dst, lane * 16, c[0]);
      if (lane * 8 + 512 < total) st16_so<0>(dst + cell_off(64), lane * 16, c[1]);
      if (lane * 8 + 1024 < total) st16_so<0>(dst + cell_off(128), lane * 16, c[2]);
      if (lane * 8 + 1536 < total) st16_so<0>(dst + cell_off(192), lane * 16, c[3]);
    } else if (lane == 0) st4_sys(tmo, call);
  } else {
    f32x4 y[2][4];
#pragma unroll
    for (int i = 0; i < 8; i++) y[i >> 2][i & 3] = *(LAS f32x4*)(size_t)(zb + (i >> 2) * 4096 + lane * 64 + (i & 3) * 16);
    if (hold) {
      while (__builtin_amdgcn_readfirstlane(lds_ld8(sm + 2)[1]) != call && now() - t0 < 2 * DEADLINE) __builtin_amdgcn_s_sleep(1);
      mask();
    }
    const u32 gm = get_gmax(call, t0, bad, [&]() {});
    if (gm != 0xffffffffu) {
      const float inv = inv_scale(gm);
      auto q16 = [&](const f32x4* v) {
        return (u32x4{quant4(v[0], inv), quant4(v[1], inv), quant4(v[2], inv), quant4(v[3], inv)} & 0x7f7f7f7fu) | tg[0];
      };
      if (CFG_LD) {
        const u64 t1 = now();
        u32x4 c0 = q16(y[0]), c1 = q16(y[1]);
        asm volatile("" : "+v"(c0), "+v"(c1));
        pace(t1, CFG_LD * ((pos - CFG_NE) * 4 + wave) / ((32 - CFG_NE) * 4), CFG_TM & 1, false);
        if (lane * 16 < total) st16_so<0>(dst, lane * 16, c0);
        if (lane * 16 + 1024 < total) st16_so<0>(dst + cell_off(64), lane * 16, c1);
      } else {
        if (lane * 16 < total) st16_so<0>(dst, lane * 16, q16(y[0]));
        if (lane * 16 + 1024 < total) st16_so<0>(dst + cell_off(64), lane * 16, q16(y[1]));
      }
    }
  }
  if (wave == 0 && lane == 63) st4_sys(ctr, call);
  }
  const int job = wave < 4 ? jobC(id) : jobA(id);
  if (job >= 0) recv(job, call, t0);
}

static bool launch_setup() {
  hipError_t e = hipFuncSetAttribute((const void*)gemm_a2a, hipFuncAttributeMaxDynamicSharedMemorySize, LDS_BYTES);
  if (e != hipSuccess) { fprintf(stderr, "HIP error %s\n", hipGetErrorString(e)); return false; }
  return true;
}
static void launch_impl(int rank, hipStream_t stream, const void* x, const void* w, char* const* out_dev, char* const* ws_dev,
                        char* const* out_host, char* const* ws_host) {
  Peers peers;
  for (int p = 0; p < NR; p++) { peers.ws[p] = ws_host[p]; peers.out[p] = out_host[p]; }
  gemm_a2a<<<256, 512, LDS_BYTES, stream>>>((const char*)x, (const char*)w, ws_host[rank], out_host[rank], ws_dev, rank, peers);
}
extern "C" void kernel_launch(int rank, int nranks, ncclComm_t comm, hipStream_t stream, void* const* d_in, const int* in_sizes, int n_in,
                              void* d_out, int out_size, const int* in_dtypes, int out_dtype, void* const* d_out_all,
                              void* const* d_ws_all, size_t ws_bytes) {
  if (nranks != NR || (unsigned)rank >= (unsigned)NR || n_in < 2 || in_sizes[0] != MS * KD || in_sizes[1] != KD * ND ||
      out_size != NR * MS * NCOL || ws_bytes < WS_END) {
    fprintf(stderr, "unexpected problem shape\n");
    return;
  }
  if (!launch_setup()) return;
  launch_impl(rank, stream, d_in[0], d_in[1], (char* const*)d_out_all, (char* const*)d_ws_all, (char* const*)d_out_all, (char* const*)d_ws_all);
}
```
